```python
import numpy as np

kernel_name = "gpu_a2a_gemm_m4096_k4096_n4096_w8a8_dequant"


ABSMAX_RTOL = 0.001

COMPUTE_DTYPE = "int8"
EPILOGUE = "dequant"
IN_QUANT = "int8"
OUT_QUANT = None
IN_DTYPES = {"x": "int8", "w_mat": "int8",
             "scale_x": "float32", "scale_w": "float32"}
WORLD_SIZE = 8

def setup_inputs():

    x = np.random.randint(-127, 128, size=(4096, 4096)).astype(np.int8)
    w_mat = np.random.randint(-127, 128, size=(4096, 4096)).astype(np.int8)
    scale_x = (0.02 * (1.0 + np.random.rand(1))).astype(np.float32)
    scale_w = (0.02 * (1.0 + np.random.rand(1))).astype(np.float32)
    return {"x": x, "w_mat": w_mat, "scale_x": scale_x, "scale_w": scale_w}

def reference(x, w_mat, scale_x, scale_w):
    acc = x.astype(np.float64) @ w_mat.astype(np.float64)
    y = (acc.astype(np.float32) * (scale_x * scale_w)).astype(np.float32)
    return y
```

```cpp
#include <hip/hip_runtime.h>
#include <rccl/rccl.h>
#include <cstdio>

#define HIP_CHECK(expr) do { hipError_t _e = (expr); if (_e != hipSuccess) { \
    fprintf(stderr, "HIP error %s at %s:%d\n", hipGetErrorString(_e), __FILE__, __LINE__); \
    return; } } while (0)

typedef int      i32x2 __attribute__((ext_vector_type(2)));
typedef int      i32x4 __attribute__((ext_vector_type(4)));
typedef unsigned u32x4 __attribute__((ext_vector_type(4)));
typedef float    f32x4 __attribute__((ext_vector_type(4)));
#define LDS_AS __attribute__((address_space(3)))

namespace {
constexpr int NR = 8;
constexpr int MLOC = 512;
constexpr int ND = 4096;
constexpr int KB = 512;
constexpr int NWG = 256;
constexpr int NCH = 8;
#define SPLIT 2
constexpr int KSS = 8 / SPLIT;

constexpr unsigned WS_CNT  = 0;
constexpr unsigned WS_TMO  = 16384;
constexpr unsigned WS_FLAG = 32768;
constexpr unsigned WS_DBG  = 65536;
constexpr unsigned WS_X    = 1u << 20;
constexpr unsigned WS_NEED = WS_X + 2u * NR * 262144u;
constexpr unsigned long long DEADLINE_TICKS = 30000000ull;

constexpr unsigned LDS_ACC = 0;
constexpr unsigned LDS_CNT = 32768;
constexpr unsigned LDS_SCR = 32768 + 256;
constexpr unsigned LDS_BYTES = LDS_SCR + 8 * 8192;

struct Params {
    const signed char* x;
    const signed char* w;
    const float* sx;
    const float* sw;
    float* out;
    char* ws[NR];
    int rank;
    int pad;
};

#define STAMP_AT(i) do {} while (0)

__device__ __forceinline__ void store16_sys(char* p, u32x4 v) {
    asm volatile("global_store_dwordx4 %0, %1, off sc0 sc1\n\ts_nop 1" :: "v"(p), "v"(v) : "memory");
}
__device__ __forceinline__ void store4_sys(char* p, unsigned v) {
    asm volatile("global_store_dword %0, %1, off sc0 sc1" :: "v"(p), "v"(v) : "memory");
}
template <int N> __device__ __forceinline__ void wait_vm() {
    asm volatile("s_waitcnt vmcnt(%0)" :: "i"(N) : "memory");
}
template <int C> struct SendFlags {
    static __device__ __forceinline__ void run(char* flag, unsigned base, int lane) {
        wait_vm<(NCH - 1 - C) * KSS + C>();
        store4_sys(flag, base + C + 1);
        SendFlags<C + 1>::run(flag, base, lane);
    }
};
template <> struct SendFlags<NCH> {
    static __device__ __forceinline__ void run(char*, unsigned, int) {}
};

__global__ __launch_bounds__(512, 2) void a2a_gemm_kernel(Params P) {
    extern __shared__ __attribute__((aligned(16))) char smem[];
    const int tid = threadIdx.x;
    const int lane = tid & 63;
    const int wave = __builtin_amdgcn_readfirstlane(tid >> 6);
    const int b = blockIdx.x;
    const int rank = P.rank;
    char* const myws = P.ws[rank];
    STAMP_AT(0);

    unsigned* cntp = (unsigned*)(myws + WS_CNT + b * 64);
    unsigned call = *(const __attribute__((address_space(4))) unsigned*)cntp + 1u;
    asm volatile("" : "+s"(call));
    const unsigned par = call & 1u;
    const unsigned fbase = call * NCH;

    const int mb = (b >> 3) & 3;
    const int np = (b & 7) * 8 + (b >> 5);

    {
        u32x4 z = {0, 0, 0, 0};
#pragma unroll
        for (int i = 0; i < 4; ++i) *(u32x4*)(smem + LDS_ACC + i * 8192 + tid * 16) = z;
        if (tid < 16) *(u32x4*)(smem + LDS_CNT + tid * 16) = z;
    }
    asm volatile("s_waitcnt lgkmcnt(0)" ::: "memory");
    __builtin_amdgcn_s_barrier();
    if (tid == 0) __hip_atomic_store(cntp, call, __ATOMIC_RELAXED, __HIP_MEMORY_SCOPE_AGENT);

    const bool sender = (wave == rank) && (b < 32 * SPLIT);
    if (sender) {
        const unsigned dest = (unsigned)(rank + 1 + (b & 7)) & 7u;
        const int smb = (b >> 3) & 3;
        const int sub = b >> 5;
        const signed char* src = P.x + (size_t)(dest * MLOC + smb * 128 + (lane & 15)) * KB
                               + sub * (KSS * 64) + (lane >> 4) * 16;
        u32x4 v[NCH][KSS];
#pragma unroll
        for (int c = 0; c < NCH; ++c)
#pragma unroll
            for (int s = 0; s < KSS; ++s)
                v[c][s] = *(const u32x4*)(src + c * 16 * KB + s * 64);
        char* dws = P.ws[dest];
        char* dst = dws + WS_X + ((par * NR + rank) * 4 + smb) * 65536u + sub * (KSS * 1024) + lane * 16;
        __builtin_amdgcn_s_waitcnt(0x0F70);
        STAMP_AT(1);
#pragma unroll
        for (int c = 0; c < NCH; ++c)
#pragma unroll
            for (int s = 0; s < KSS; ++s)
                store16_sys(dst + c * 8192 + s * 1024, v[c][s]);
        char* flag = dws + WS_FLAG + (rank * 4 + smb) * 128 + sub * 4;
        SendFlags<0>::run(flag, fbase, lane);
        STAMP_AT(2);
    }

    u32x4 wraw[32];
    {
        const signed char* wp = P.w + (size_t)(wave * KB + (lane >> 2)) * ND + np * 64 + (lane & 3) * 16;
#pragma unroll
        for (int i = 0; i < 32; ++i) wraw[i] = *(const u32x4*)(wp + (size_t)i * 16 * ND);
    }
    const float scale = P.sx[0] * P.sw[0];

    i32x4 wf[8][4];
    {
        char* scr = smem + LDS_SCR + wave * 8192;
        const int wrow = lane >> 2;
        const int g = lane >> 4, q = (lane & 15) >> 1;
#pragma unroll
        for (int ks = 0; ks < 8; ++ks) {
            char* sl = scr + (ks & 1) * 4096;
#pragma unroll
            for (int i = 0; i < 4; ++i) {
                const int f = ((wrow >> 2) & 1) | ((i & 1) << 1);
                *(u32x4*)(sl + (i * 16 + wrow) * 64 + (((lane & 3) ^ f) * 16)) = wraw[ks * 4 + i];
            }
#pragma unroll
            for (int t = 0; t < 4; ++t) {
                i32x2 h[2];
#pragma unroll
                for (int j = 0; j < 2; ++j) {
                    const int row = 16 * g + 8 * j + q;
                    const int f = ((row >> 2) & 1) | ((g & 1) << 1);
                    h[j] = __builtin_amdgcn_ds_read_tr8_b64_v2i32(
                        (LDS_AS i32x2*)(sl + row * 64 + ((t ^ f) * 16) + (lane & 1) * 8));
                }
                wf[ks][t] = i32x4{h[0].x, h[0].y, h[1].x, h[1].y};
            }
        }
    }
    STAMP_AT(3);

    const unsigned* flagp = (const unsigned*)(myws + WS_FLAG + (wave * 4 + mb) * 128) + (lane & (SPLIT - 1));
    unsigned* tmo = (unsigned*)(myws + WS_TMO);
    const auto xrs = __builtin_amdgcn_make_buffer_rsrc(
        (void*)(myws + WS_X + ((par * NR + wave) * 4 + mb) * 65536u), 0, 65536, 0x00020000);
    const unsigned long long t0 = __builtin_amdgcn_s_memrealtime();
    const int g = lane >> 4;
    LDS_AS int* accl = (LDS_AS int*)(smem + LDS_ACC) + g * 256 + (lane & 15);
    LDS_AS unsigned* cnt = (LDS_AS unsigned*)(smem + LDS_CNT);

    auto poll_once = [&](unsigned need) -> bool {
        unsigned v = __hip_atomic_load(flagp, __ATOMIC_RELAXED, __HIP_MEMORY_SCOPE_SYSTEM);
        return __all((int)(v - need) >= 0);
    };
    unsigned arrive = 1u;
    auto poll_until = [&](unsigned need) {
        while (!poll_once(need)) {
            if (__hip_atomic_load(tmo, __ATOMIC_RELAXED, __HIP_MEMORY_SCOPE_AGENT) == call ||
                __builtin_amdgcn_s_memrealtime() - t0 > DEADLINE_TICKS) {
                __hip_atomic_store(tmo, call, __ATOMIC_RELAXED, __HIP_MEMORY_SCOPE_AGENT);
                arrive = 0x10001u;
                break;
            }
            __builtin_amdgcn_s_sleep(2);
        }
        asm volatile("" ::: "memory");
    };
    auto load_x = [&](i32x4 (&xa)[8], int c) {
#pragma unroll
        for (int ks = 0; ks < 8; ++ks)
            xa[ks] = __builtin_bit_cast(i32x4, __builtin_amdgcn_raw_buffer_load_b128(
                xrs, lane * 16 + ks * 1024, c * 8192, 17));
    };
    auto compute = [&](i32x4 (&xa)[8], int c) {
        i32x4 acc[4] = {{0, 0, 0, 0}, {0, 0, 0, 0}, {0, 0, 0, 0}, {0, 0, 0, 0}};
#pragma unroll
        for (int ks = 0; ks < 8; ++ks)
#pragma unroll
            for (int t = 0; t < 4; ++t)
                acc[t] = __builtin_amdgcn_mfma_i32_16x16x64_i8(xa[ks], wf[ks][t], acc[t], 0, 0, 0);
        LDS_AS int* a = accl + c * 1024;
#pragma unroll
        for (int t = 0; t < 4; ++t)
#pragma unroll
            for (int r = 0; r < 4; ++r)
                __hip_atomic_fetch_add(a + r * 64 + 16 * (t ^ (g & 1)), acc[t][r], __ATOMIC_RELAXED,
                                       __HIP_MEMORY_SCOPE_WORKGROUP);
        asm volatile("s_waitcnt lgkmcnt(0)" ::: "memory");
        unsigned old = 0;
        if (lane == 0) old = __hip_atomic_fetch_add(cnt + c, arrive, __ATOMIC_RELAXED, __HIP_MEMORY_SCOPE_WORKGROUP);
        old = __builtin_amdgcn_readfirstlane(old) + arrive;
        asm volatile("" ::: "memory");
        if ((old & 0xffffu) == NR) {
            const float sc = (old >> 16) ? __builtin_nanf("") : scale;
            float* op = P.out + (size_t)(mb * 128 + c * 16 + g) * ND + np * 64 + (lane & 15) * 4;
#pragma unroll
            for (int i = 0; i < 4; ++i) {
                i32x4 s = *(LDS_AS i32x4*)((LDS_AS int*)(smem + LDS_ACC) + c * 1024 + (4 * i + g) * 64
                                           + (((lane & 15) * 4) ^ ((i & 1) << 4)));
                f32x4 o = {(float)s.x * sc, (float)s.y * sc, (float)s.z * sc, (float)s.w * sc};
                *(f32x4*)(op + (size_t)i * 4 * ND) = o;
            }
        }
        STAMP_AT(16 + c);
    };

    i32x4 xa[8], xb[8];
    poll_until(fbase + 1);
    STAMP_AT(8);
    load_x(xa, 0);
    auto step = [&](i32x4 (&cur)[8], i32x4 (&nxt)[8], int c) {
        const bool more = c + 1 < NCH;
        const bool pre = more && poll_once(fbase + c + 2);
        asm volatile("" ::: "memory");
        if (pre) { STAMP_AT(8 + c + 1); load_x(nxt, c + 1); }
        compute(cur, c);
        if (more && !pre) { poll_until(fbase + c + 2); STAMP_AT(8 + c + 1); load_x(nxt, c + 1); }
    };
#pragma unroll 1
    for (int c = 0; c < NCH; c += 2) {
        step(xa, xb, c);
        step(xb, xa, c + 1);
    }
}
}

extern "C" void kernel_launch(int rank, int nranks, ncclComm_t comm,
                              hipStream_t stream,
                              void* const* d_in, const int* in_sizes, int n_in,
                              void* d_out, int out_size,
                              const int* in_dtypes, int out_dtype,
                              void* const* d_out_all,
                              void* const* d_ws_all, size_t ws_bytes) {
    if (nranks != NR || (unsigned)rank >= (unsigned)NR || ws_bytes < WS_NEED || n_in < 4 ||
        out_size != MLOC * ND) {
        fprintf(stderr, "unexpected configuration\n");
        return;
    }
    static bool init = false;
    if (!init) {
        HIP_CHECK(hipFuncSetAttribute((const void*)a2a_gemm_kernel,
                                      hipFuncAttributeMaxDynamicSharedMemorySize, LDS_BYTES));
        init = true;
    }
    Params P;
    P.x = (const signed char*)d_in[0];
    P.w = (const signed char*)d_in[1];
    P.sx = (const float*)d_in[2];
    P.sw = (const float*)d_in[3];
    P.out = (float*)d_out;
    for (int p = 0; p < NR; ++p) P.ws[p] = (char*)d_ws_all[p];
    P.rank = rank;
    P.pad = 0;
    hipLaunchKernelGGL(a2a_gemm_kernel, dim3(NWG), dim3(512), LDS_BYTES, stream, P);
    HIP_CHECK(hipGetLastError());
}
```

```cpp
#include <hip/hip_runtime.h>
#include <rccl/rccl.h>
#include <cstdio>

#define HIP_CHECK(expr) do { hipError_t _e = (expr); if (_e != hipSuccess) { \
    fprintf(stderr, "HIP error %s at %s:%d\n", hipGetErrorString(_e), __FILE__, __LINE__); \
    return; } } while (0)

typedef int      i32x2 __attribute__((ext_vector_type(2)));
typedef int      i32x4 __attribute__((ext_vector_type(4)));
typedef unsigned u32x4 __attribute__((ext_vector_type(4)));
typedef float    f32x4 __attribute__((ext_vector_type(4)));
typedef unsigned long long u64;
#define LDS_AS __attribute__((address_space(3)))
typedef __attribute__((address_space(1))) unsigned gu32;
typedef __attribute__((address_space(4))) const unsigned cu32;
typedef __attribute__((address_space(4))) const float cf32;

#define T_FIRST 140
#define T_STEP 64
#define T_NEAR 30
#define W_AUX "nt"
#define X_AUX "nt"
#define W_WIN 8
#define POLL_SLEEP 1
#define CODE_BYTES 14336u
#define IMP_POS 3

namespace {
constexpr int NR = 8;
constexpr int MLOC = 512;
constexpr int ND = 4096;
constexpr int KB = 512;
constexpr int NWG = 256;
constexpr int SYS = 17;
constexpr unsigned UNIT = 4096 + 256;

constexpr unsigned WS_CNT  = 0;
constexpr unsigned WS_TMO  = 16384;
constexpr unsigned WS_X    = 1u << 20;
constexpr unsigned X_PAR   = 512u * UNIT;
constexpr unsigned WS_DBG  = 8u << 20;
constexpr unsigned WS_P    = 10u << 20;
constexpr unsigned WS_NEED = WS_P + NWG * 32768u;
constexpr u64 DEADLINE_TICKS = 30000000ull;

constexpr int BIAS = 1 << 24;
constexpr unsigned LDS_ACC = 0;
constexpr unsigned LDS_SCR = 65536;
constexpr unsigned LDS_CNT = 131072;
constexpr unsigned LDS_BYTES = LDS_CNT + 64;

struct Params {
    const signed char* x;
    const signed char* w;
    char* myws;
    int rank;
    int pad;
    char* ws[NR];
    const float* sx;
    const float* sw;
    float* out;
};

#define STAMP_AT(i) do {} while (0)
#define STAMP_CLK(i) do {} while (0)
#define LDS_EXTRA 0

#define CBAR() asm volatile("" ::: "memory")
#define SB() __builtin_amdgcn_sched_barrier(0)
template <int N> __device__ __forceinline__ void wait_vm() {
    asm volatile("s_waitcnt vmcnt(%0)" :: "i"(N) : "memory");
}
__device__ __forceinline__ __amdgpu_buffer_rsrc_t mk_rsrc(const void* p, unsigned bytes) {
    return __builtin_amdgcn_make_buffer_rsrc((void*)p, 0, bytes, 0x00020000);
}
struct XB { u32x4 f[8]; unsigned e[2]; };

__global__ __launch_bounds__(256, 1) void a2a_gemm_kernel(Params P) {
    extern __shared__ __attribute__((aligned(16))) char smem[];
    const u64 tstart = __builtin_amdgcn_s_memrealtime();
    const int tid = threadIdx.x;
    const int lane = tid & 63;
    const int wave = __builtin_amdgcn_readfirstlane(tid >> 6);
    const int b = blockIdx.x;
    {
        u64 pc;
        asm volatile("s_getpc_b64 %0" : "=s"(pc));
        if (wave >= 2) {
            const u32x4 crs = {(unsigned)pc, (unsigned)(pc >> 32) & 0xffffu, CODE_BYTES, 0x00020000u};
            asm volatile("s_mov_b32 m0, %0\n\ts_nop 0\n\tbuffer_load_dword %1, %2, 0 offen lds"
                         :: "s"((unsigned)(size_t)(LDS_AS char*)smem + LDS_SCR + wave * 16384 + 12288),
                            "v"((tid - 128) * 128), "s"(crs) : "memory");
        }
    }
    int rank = P.rank;
    char* myws = P.myws;
    const signed char* xptr = P.x;
    const signed char* wptr = P.w;
    const float* sxp = P.sx;
    const float* swp = P.sw;
    asm volatile("" : "+s"(rank), "+s"(myws), "+s"(xptr), "+s"(wptr), "+s"(sxp), "+s"(swp));
    STAMP_CLK(62);

    const int mb = (b >> 3) & 3;
    const int kh = b & 1;
    const int np = ((b & 7) >> 1) * 8 + (b >> 5);
    const int src = kh * 4 + wave;
    const int g = lane >> 4, m16 = lane & 15;

    const int spos = (np + (int)((size_t)wptr >> 7) + 5) & 7;
    const unsigned dest = (unsigned)(rank + 1 + (b & 7)) & 7u;
    const int schunk = (spos + (rank & 4)) & 7;
    bool unsent = wave < 2;
    const unsigned tsend = (unsigned)tstart + T_FIRST + (spos > 0 ? spos - 1 : 0) * T_STEP;

    const u32x4 wrs = {(unsigned)(size_t)wptr, (unsigned)((size_t)wptr >> 32) & 0xffffu, (unsigned)ND * ND, 0x00020000u};
    const unsigned wv0 = (lane >> 3) * ND + np * 128 + (((lane & 7) ^ g) * 16);
    const unsigned wv1 = wv0 ^ 64u;
    const unsigned lds0 = (unsigned)(size_t)(LDS_AS char*)smem;
    const unsigned scr = lds0 + LDS_SCR + wave * 16384;
    auto w_piece = [&](int s, int i) {
        const unsigned row0 = src * KB + (((s >> 1) + 2 * mb) & 7) * 64 + (s & 1) * 8;
        asm volatile("s_mov_b32 m0, %0\n\ts_nop 0\n\tbuffer_load_dwordx4 %1, %2, %3 offen " W_AUX " lds"
                     :: "s"(scr + (s & 3) * 4096 + i * 1024), "v"((i & 1) ? wv1 : wv0), "s"(wrs),
                        "s"((row0 + 16 * i) * ND) : "memory");
    };
    u32x4 pay[4], payx;
    if (unsent) {
        const u32x4 xr = {(unsigned)(size_t)xptr, (unsigned)((size_t)xptr >> 32) & 0xffffu, (unsigned)MLOC * NR * KB, 0x00020000u};
        const unsigned so = (dest * MLOC + mb * 128 + schunk * 16) * KB;
#pragma unroll
        for (int s = 0; s < 4; ++s)
            asm volatile("buffer_load_dwordx4 %0, %1, %2, %3 offen " X_AUX
                         : "=v"(pay[s]) : "v"(m16 * KB + g * 16), "s"(xr),
                           "s"(so + ((4 * wave + s + 2 * mb) & 7) * 64) : "memory");
    }
    __builtin_amdgcn_s_barrier();
#pragma unroll
    for (int i = 0; i < 8; ++i) w_piece(i >> 2, i & 3);
    gu32* cntp = (gu32*)(myws + WS_CNT) + (b * 4 + wave);
    unsigned call = *(cu32*)(size_t)cntp + 1u;
    float sxv = *(cf32*)sxp, swv = *(cf32*)swp;
    char* dws = P.ws[dest];
    float* outp = P.out;

    {
        u32x4 z = {0, 0, 0, 0};
#pragma unroll
        for (int i = 0; i < 16; ++i) *(u32x4*)(smem + LDS_ACC + i * 4096 + tid * 16) = z;
        if (tid < 4) *(u32x4*)(smem + LDS_CNT + tid * 16) = z;
    }
    asm volatile("s_waitcnt lgkmcnt(0)" : "+s"(call), "+s"(sxv), "+s"(swv), "+s"(dws), "+s"(outp) :: "memory");
    __builtin_amdgcn_s_barrier();

    const unsigned par = call & 1u;
    const unsigned t4 = call % 15u + 1u;
    const unsigned tag4 = t4 << 28;
    const unsigned tage = t4 * 0x01010101u;
    const unsigned tag5 = (call % 31u + 1u) << 27;

    if (unsent) {
        wait_vm<8>();
        asm volatile("" : "+v"(pay[0]), "+v"(pay[1]), "+v"(pay[2]), "+v"(pay[3]) :: "memory");
        unsigned e = tage;
#pragma unroll
        for (int s = 0; s < 4; ++s) {
            e |= ((pay[s].w ^ tag4) >> 28) << (8 * s + 4);
            pay[s].w = (pay[s].w & 0x0fffffffu) | tag4;
        }
        payx.x = e;
        payx.y = __builtin_amdgcn_ds_bpermute((m16 + 16) * 4, e);
        payx.z = __builtin_amdgcn_ds_bpermute((m16 + 32) * 4, e);
        payx.w = __builtin_amdgcn_ds_bpermute((m16 + 48) * 4, e);
        STAMP_AT(1);
    }
    const auto drs = mk_rsrc(dws + WS_X, 2 * X_PAR);
    const unsigned dso = par * X_PAR + ((((unsigned)rank * 4 + mb) * 8 + spos) * 2 + wave) * UNIT;
    auto try_send = [&](int near) {
        if (unsent) {
            int left = (int)(tsend - (unsigned)__builtin_amdgcn_s_memrealtime());
            if (left <= near) {
                while (left > 0) {
                    __builtin_amdgcn_s_sleep(1);
                    left = (int)(tsend - (unsigned)__builtin_amdgcn_s_memrealtime());
                }
#pragma unroll
                for (int s = 0; s < 4; ++s)
                    __builtin_amdgcn_raw_buffer_store_b128(pay[s], drs, lane * 16 + s * 1024, dso, SYS);
                if (lane < 16) __builtin_amdgcn_raw_buffer_store_b128(payx, drs, lane * 16, dso + 4096, SYS);
                unsent = false;
                CBAR();
                STAMP_AT(2);
            }
        }
    };
    try_send(0);

    i32x4 wf[8][8];
    {
        const int q = m16 >> 1;
        const unsigned rb = scr + (8 * g + q) * 128 + (((4 * g + (q >> 1)) & 7) * 16) + (lane & 1) * 8;
#pragma unroll
        for (int s = 0; s < 16; ++s) {
            if (s < 14) {
#pragma unroll
                for (int i = 0; i < 4; ++i) { wait_vm<W_WIN - 1>(); w_piece(s + 2, i); }
                if (s & 1) try_send(T_NEAR);
                wait_vm<8>();
            } else if (s == 14) {
                wait_vm<4>();
            } else {
                wait_vm<0>();
            }
            asm volatile("s_waitcnt lgkmcnt(0)" ::: "memory");
#pragma unroll
            for (int t = 0; t < 8; ++t) {
                i32x2 v = __builtin_amdgcn_ds_read_tr8_b64_v2i32(
                    (LDS_AS i32x2*)(size_t)((rb ^ (t << 4)) + (s & 3) * 4096));
                if (s & 1) { wf[s >> 1][t].z = v.x; wf[s >> 1][t].w = v.y; }
                else       { wf[s >> 1][t].x = v.x; wf[s >> 1][t].y = v.y; }
            }
        }
    }
    __builtin_amdgcn_s_waitcnt(0xC07F);
    STAMP_AT(3);
    try_send(0x7fffffff);

    gu32* tmo = (gu32*)(myws + WS_TMO);
    const auto xrs = mk_rsrc(myws + WS_X + par * X_PAR + (unsigned)(src * 4 + mb) * 16 * UNIT, 16 * UNIT);
    const size_t pptr = (size_t)(myws + WS_P + (unsigned)(b ^ 1) * 32768u);
    const u32x4 prs = {(unsigned)pptr, (unsigned)(pptr >> 32) & 0xffffu, 32768u, 0x00020000u};
    const auto mrs = mk_rsrc(myws + WS_P + (unsigned)b * 32768u, 32768);
    const auto ors = mk_rsrc(outp, (unsigned)MLOC * ND * 4);
    const float scale = sxv * swv;
    const unsigned pgw = lds0 + LDS_CNT + wave * 4, pgr = lds0 + LDS_CNT + (lane & 3) * 4;
    const unsigned ra = lds0 + LDS_ACC + ((lane & 31) >> 2) * 1024 + (lane & 3) * 128
                      + (((lane >> 5) ^ ((lane & 3) >> 1) ^ (2 * ((lane & 31) >> 2)) ^ (4 * wave)) * 8);
    const unsigned pvo = wave * 2048 + lane * 16;
    const unsigned ovo = (4 * wave + (lane >> 5)) * (ND * 4) + (lane & 31) * 16;
    const unsigned obase = ((mb * 128 - kh * 64) * ND + np * 128) * 4;
    const unsigned park = scr + lane * 16;
    const unsigned xvo = lane * 16, evo = m16 * 16 + g * 4 + 4096;
    constexpr unsigned OOB = 0x40000000u;

    int dead = 0;
    int p = 0, f = 0;
    const i32x4 cinit = {BIAS, 0, BIAS, 0};
    i32x4 pacc = {0, 0, 0, 0};
    unsigned aad = lds0 + LDS_ACC + g * 128 + ((m16 ^ (g >> 1)) * 8);
    unsigned paddr = aad;
    u32x4 fa[2], fq[2];
    int pr;
    unsigned so_pub = OOB, so_out = OOB;

    auto ld_x = [&](XB& B, int i, unsigned so) {
        if (i < 8) B.f[i] = __builtin_amdgcn_raw_buffer_load_b128(xrs, xvo + (i >> 2) * UNIT + (i & 3) * 1024, so, SYS);
        else B.e[i - 8] = __builtin_amdgcn_raw_buffer_load_b32(xrs, evo + (i - 8) * UNIT, so, SYS);
    };
    auto ld_imp = [&](int pos) {
        asm volatile("s_mov_b32 m0, %0\n\ts_nop 0\n\tbuffer_load_dwordx4 %1, %2, %3 offen sc0 sc1 lds\n\t"
                     "buffer_load_dwordx4 %1, %2, %3 offen offset:1024 sc0 sc1 lds"
                     :: "s"(scr + pos * 2048), "v"(pvo), "s"(prs), "s"(((unsigned)(pos - 4) * 8192u) & 0x7fffffffu) : "memory");
    };
    unsigned ck0, ck1;
    auto check_step = [&](XB& B, int st) {
        if (st == 0) {
            asm volatile("" : "+v"(B.f[0]), "+v"(B.f[1]), "+v"(B.f[2]), "+v"(B.f[3]));
            ck0 = (B.f[0].w ^ tag4) | (B.f[1].w ^ tag4);
            ck1 = (B.f[2].w ^ tag4) | (B.f[3].w ^ tag4);
            asm volatile("" : "+v"(ck0), "+v"(ck1));
        } else if (st == 1) {
            asm volatile("" : "+v"(B.f[4]), "+v"(B.f[5]), "+v"(B.f[6]), "+v"(B.f[7]));
            ck0 = ck0 | ck1 | ((B.f[4].w ^ tag4) | (B.f[5].w ^ tag4));
            ck1 = (B.f[6].w ^ tag4) | (B.f[7].w ^ tag4);
            asm volatile("" : "+v"(ck0), "+v"(ck1));
        } else if (st == 2) {
            asm volatile("" : "+v"(B.e[0]), "+v"(B.e[1]));
            B.e[0] ^= tage;
            B.e[1] ^= tage;
            ck0 = (ck0 | ck1) & 0xf0000000u;
            asm volatile("" : "+v"(ck0), "+v"(B.e[0]), "+v"(B.e[1]));
        } else if (st == 3) {
            ck0 |= (B.e[0] | B.e[1]) & 0x0f0f0f0fu;
        } else {
            const int u = (st - 4) >> 1, s0 = 2 * (st - 4);
            unsigned w0 = B.f[s0].w, w1 = B.f[s0 + 1].w;
            if (s0 & 2)
                asm volatile("v_xor_b32_sdwa %0, %0, %2 dst_sel:BYTE_3 dst_unused:UNUSED_PRESERVE src0_sel:BYTE_3 src1_sel:BYTE_2\n\t"
                             "v_xor_b32_sdwa %1, %1, %2 dst_sel:BYTE_3 dst_unused:UNUSED_PRESERVE src0_sel:BYTE_3 src1_sel:BYTE_3"
                             : "+v"(w0), "+v"(w1) : "v"(B.e[u]));
            else
                asm volatile("v_xor_b32_sdwa %0, %0, %2 dst_sel:BYTE_3 dst_unused:UNUSED_PRESERVE src0_sel:BYTE_3 src1_sel:BYTE_0\n\t"
                             "v_xor_b32_sdwa %1, %1, %2 dst_sel:BYTE_3 dst_unused:UNUSED_PRESERVE src0_sel:BYTE_3 src1_sel:BYTE_1"
                             : "+v"(w0), "+v"(w1) : "v"(B.e[u]));
            B.f[s0].w = w0;
            B.f[s0 + 1].w = w1;
        }
    };
    auto lds_add64 = [&](unsigned addr, unsigned lo, unsigned hi) {
        __hip_atomic_fetch_add((LDS_AS u64*)(size_t)addr, ((u64)hi << 32) | lo, __ATOMIC_RELAXED,
                               __HIP_MEMORY_SCOPE_WORKGROUP);
    };
    auto rd_flush = [&](int st) {
        CBAR();
        if (st == 0) pr = *(volatile LDS_AS int*)(size_t)pgr;
        if (st == 1 || st == 2) {
            const unsigned a = (ra + f * 8192) ^ (st == 2 ? 16 : 0);
            const i32x2 lo = *(LDS_AS i32x2*)(size_t)a, hi = *(LDS_AS i32x2*)(size_t)(a + 512);
            fa[st - 1] = u32x4{(unsigned)lo.x, (unsigned)lo.y, (unsigned)hi.x, (unsigned)hi.y};
        }
        if (st >= 3) fq[st - 3] = *(LDS_AS u32x4*)(size_t)(park + f * 2048 + (st - 3) * 1024);
        CBAR();
    };
    auto imp_wrong = [&]() -> unsigned {
        return ((fq[0].y ^ tag5) | (fq[0].w ^ tag5) | (fq[1].y ^ tag5) | (fq[1].w ^ tag5)) >> 27;
    };
    f32x4 fo;
    auto out_step = [&](int i, int st, float sc) {
        if (st == 0) {
            fq[i].x = fa[i].x + fq[i].x - 8 * BIAS;
            fq[i].z = fa[i].z + fq[i].z - 8 * BIAS;
        } else if (st == 1) {
            fq[i].y = (unsigned)((int)(fq[i].y << 5) >> 5);
            fq[i].w = (unsigned)((int)(fq[i].w << 5) >> 5);
        } else if (st == 2) {
            fq[i].y += fa[i].y;
            fq[i].w += fa[i].w;
        } else if (st == 3) {
            fo.x = (float)(int)fq[i].x; fo.y = (float)(int)fq[i].y;
        } else if (st == 4) {
            fo.z = (float)(int)fq[i].z; fo.w = (float)(int)fq[i].w;
        } else if (st == 5) {
            fo.x *= sc; asm volatile("" : "+v"(fo)); fo.y *= sc;
        } else if (st == 6) {
            fo.z *= sc; asm volatile("" : "+v"(fo)); fo.w *= sc;
        } else {
            __builtin_amdgcn_raw_buffer_store_b128(__builtin_bit_cast(u32x4, fo), ors, ovo + i * (2 * ND * 4), so_out, SYS);
        }
        if (st < 3) asm volatile("" : "+v"(fq[i])); else if (st < 7) asm volatile("" : "+v"(fo));
    };
    auto pub_step = [&](int i, int st) {
        if (st == 0) {
            fa[i].y = (fa[i].y & 0x07ffffffu) | tag5;
            fa[i].w = (fa[i].w & 0x07ffffffu) | tag5;
            asm volatile("" : "+v"(fa[i]));
        } else {
            __builtin_amdgcn_raw_buffer_store_b128(fa[i], mrs, pvo + i * 1024, so_pub, SYS);
        }
    };
    auto block = [&](XB& cur, XB& nxt) -> bool {
        STAMP_AT(8 + p);
        unsigned so, at = paddr;
        u64 wrong = 0, nofl = 0;
        SB();
#pragma unroll
        for (int t = 0; t < 8; ++t) {
            i32x4 c;
#pragma unroll
            for (int s = 0; s < 8; ++s) {
                if (s == 0) asm volatile("v_mfma_i32_16x16x64_i8 %0, %1, %2, %3" : "=&v"(c) : "a"(wf[s][t]), "v"(cur.f[s]), "v"(cinit));
                else        asm volatile("v_mfma_i32_16x16x64_i8 %0, %1, %2, %0" : "+v"(c) : "a"(wf[s][t]), "v"(cur.f[s]));
                SB();
                const int gp = t * 8 + s;
                if (t > 0 && s == 0) { at = aad ^ ((t - 1) << 4); asm volatile("" : "+v"(at)); }
                if (s == 5) lds_add64(at + ((t + 7) & 7) * 1024, pacc.x, pacc.y);
                if (s == 7) lds_add64(at + ((t + 7) & 7) * 1024 + 512, pacc.z, pacc.w);
                if (gp == 0) { so = (unsigned)(p + 1) * (2 * UNIT); asm volatile("" : "+s"(so)); ld_imp(p); }
                if (gp >= 1 && gp <= 4) ld_x(nxt, gp - 1, so);
                if (gp == 6) ld_x(nxt, 4, so);
                if (gp >= 9 && gp <= 12) ld_x(nxt, gp - 4, so);
                if (gp == 14) ld_x(nxt, 9, so);
                if (gp == 8) { CBAR(); *(volatile LDS_AS int*)(size_t)pgw = p + dead; }
                if (gp >= 17 && gp <= 20) out_step(0, gp - 17, scale);
                if (gp == 22) out_step(0, 4, scale);
                if (gp >= 25 && gp <= 27) out_step(0, gp - 20, scale);
                if (gp == 28) out_step(1, 0, scale);
                if (gp == 30) out_step(1, 1, scale);
                if (gp >= 33 && gp <= 36) out_step(1, gp - 31, scale);
                if (gp == 38) out_step(1, 6, scale);
                if (gp == 40) out_step(1, 7, scale);
                if (gp >= 41 && gp <= 44) pub_step((gp - 41) >> 1, (gp - 41) & 1);
                if (gp == 46) rd_flush(0);
                if (gp == 48) { wait_vm<14>(); rd_flush(3); }
                if (gp == 49) rd_flush(4);
                if (gp == 50) rd_flush(1);
                if (gp == 51) rd_flush(2);
                if (gp == 54) check_step(nxt, 0);
                if (gp >= 56 && gp <= 62) check_step(nxt, gp - 55);
                if (gp == 58) { wrong = __ballot(ck0 != 0); asm volatile("" : "+s"(wrong)); }
                if (gp == 60) {
                    asm volatile("" : "+v"(pr), "+v"(fq[0]), "+v"(fq[1]));
                    nofl = __ballot(pr <= f) | (f < 4 ? 0ull : __ballot(imp_wrong() != 0));
                    asm volatile("" : "+s"(nofl));
                }
                if (gp == 62) {
                    so_pub = nofl == 0 ? (unsigned)f * 8192u : OOB;
                    so_out = nofl == 0 && f >= 4 ? obase + (unsigned)f * (16 * ND * 4) : OOB;
                    asm volatile("s_cmp_eq_u64 %3, 0\n\ts_addc_u32 %0, %0, 0" : "+s"(f), "+s"(so_pub), "+s"(so_out) : "s"(nofl) : "scc");
                }
                SB();
            }
            pacc = c;
        }
        paddr = aad ^ (7 << 4);
        aad += 8192;
        ++p;
        return wrong == 0;
    };

    XB A, B;
    fa[0] = fa[1] = fq[0] = fq[1] = u32x4{0, 0, 0, 0};
    while (true) {
        asm volatile("s_nop 7" : "+v"(pacc));
        lds_add64(paddr + 7 * 1024, pacc.x, pacc.y);
        lds_add64(paddr + 7 * 1024 + 512, pacc.z, pacc.w);
        CBAR();
        *(volatile LDS_AS int*)(size_t)pgw = p + dead;
        pacc = i32x4{0, 0, 0, 0};
        STAMP_AT(15 + p);
        bool go = false, mem = false;
        float sc = scale;
#pragma unroll
        for (int i = 0; i < 8; ++i) asm volatile("" : "=v"(A.f[i]));
        asm volatile("" : "=v"(A.e[0]), "=v"(A.e[1]));
        while (true) {
            if ((so_pub & so_out) != OOB) {
#pragma unroll
                for (int i = 0; i < 2; ++i) {
#pragma unroll
                    for (int st = 0; st < 8; ++st) out_step(i, st, sc);
                    asm volatile("s_nop 1" :: "v"(fo));
                    pub_step(i, 0); pub_step(i, 1);
                }
                asm volatile("s_nop 1" :: "v"(fa[0]), "v"(fa[1]));
                so_pub = so_out = OOB;
                STAMP_AT(31 + f);
            } else {
                if (mem) {
                    wait_vm<0>();
#pragma unroll
                    for (int st = 0; st < 8; ++st) check_step(A, st);
                    go = (__all(ck0 == 0) || dead) && p < 8;
                    if (go) break;
                }
                if (__builtin_amdgcn_s_memrealtime() - tstart > 10000) {
                    if (__builtin_amdgcn_s_memrealtime() - tstart > DEADLINE_TICKS ||
                        __builtin_amdgcn_readfirstlane(__hip_atomic_load(tmo, __ATOMIC_RELAXED, __HIP_MEMORY_SCOPE_AGENT)) == call) {
                        __hip_atomic_store(tmo, call, __ATOMIC_RELAXED, __HIP_MEMORY_SCOPE_AGENT);
                        dead = -65536;
                    }
                }
                mem = false;
            }
            if (f == 8) break;
#pragma unroll
            for (int st = 0; st < 5; ++st) rd_flush(st);
            const bool impbad = f >= 4 && __any(imp_wrong() != 0) && !dead;
            if (!mem && (p < 8 || impbad)) {
                mem = true;
                if (impbad) ld_imp(f);
#pragma unroll
                for (int i = 0; i < 10; ++i) ld_x(A, i, (unsigned)p * (2 * UNIT));
            }
            if (__all((pr & 0xffff) > f) && !impbad) {
                const bool bad = __any(pr < 0) || dead;
                if (bad) sc = __builtin_nanf("");
                so_pub = bad ? 0x60000000u : (unsigned)f * 8192u;
                so_out = f >= 4 ? obase + (unsigned)f * (16 * ND * 4) : 0x60000000u;
                ++f;
            }
        }
        if (!go) break;
        asm volatile("s_nop 1");
        while (block(A, B) && block(B, A)) {}
    }
    if (lane == 0) __hip_atomic_store(cntp, call, __ATOMIC_RELAXED, __HIP_MEMORY_SCOPE_AGENT);
}
}

extern "C" void kernel_launch(int rank, int nranks, ncclComm_t comm,
                              hipStream_t stream,
                              void* const* d_in, const int* in_sizes, int n_in,
                              void* d_out, int out_size,
                              const int* in_dtypes, int out_dtype,
                              void* const* d_out_all,
                              void* const* d_ws_all, size_t ws_bytes) {
    if (nranks != NR || (unsigned)rank >= (unsigned)NR || ws_bytes < WS_NEED || n_in < 4 ||
        out_size != MLOC * ND) {
        fprintf(stderr, "unexpected configuration\n");
        return;
    }
    static bool init = false;
    if (!init) {
        HIP_CHECK(hipFuncSetAttribute((const void*)a2a_gemm_kernel,
                                      hipFuncAttributeMaxDynamicSharedMemorySize, LDS_BYTES + LDS_EXTRA));
        init = true;
    }
    Params P;
    P.x = (const signed char*)d_in[0];
    P.w = (const signed char*)d_in[1];
    P.sx = (const float*)d_in[2];
    P.sw = (const float*)d_in[3];
    P.out = (float*)d_out;
    for (int p = 0; p < NR; ++p) P.ws[p] = (char*)d_ws_all[p];
    P.myws = P.ws[rank];
    P.rank = rank;
    P.pad = 0;
    hipLaunchKernelGGL(a2a_gemm_kernel, dim3(NWG), dim3(256), LDS_BYTES + LDS_EXTRA, stream, P);
    HIP_CHECK(hipGetLastError());
}
```

```cpp
#include <hip/hip_runtime.h>
#include <rccl/rccl.h>
#include <cstdio>

#define HIP_CHECK(expr) do { hipError_t _e = (expr); if (_e != hipSuccess) { \
    fprintf(stderr, "HIP error %s at %s:%d\n", hipGetErrorString(_e), __FILE__, __LINE__); \
    return; } } while (0)

namespace {
constexpr int NR = 8;
constexpr unsigned WS_X    = 1u << 20;
constexpr unsigned WS_NEED = (10u << 20) + 256u * 65536u;
constexpr unsigned LDS_BYTES = 158720;

struct Params {
    const void* x;
    const void* w;
    char* myws;
    void* out;
    const void* sx;
    const void* sw;
    int rank;
    int slow;
    int pad[2];
    unsigned long long list[16][2];
    struct { unsigned long long base; unsigned chunks; int info; } snd[64];
};
const unsigned long long LISTS[16][2] = {
    {0x0fff1c61440c2040ull, 0x0008040007060504ull},
    {0x0fff0c20401c6144ull, 0x0008040003020100ull},
    {0x0fd03ce34c2ca248ull, 0x000904100f0e0d0cull},
    {0x0fcb2892103ce34cull, 0x000905000b0a0908ull},
    {0x0fd9617595513491ull, 0x0009041918171615ull},
    {0x0fd44d2459617595ull, 0x0009050014131211ull},
    {0x0ffffff7de75c6daull, 0x00060300001f1e1dull},
    {0x0ffffff71b69f79dull, 0x00060300001c1b1aull},
    {0x0fff64f356713488ull, 0x00080400190f0d16ull},
    {0x0fff71348864f356ull, 0x000804001c131208ull},
    {0x0fff1c614e31b691ull, 0x000804000706050eull},
    {0x0fff31b6911c614eull, 0x000804000c1b1a11ull},
    {0x0fff4185c40c2040ull, 0x0008040010181704ull},
    {0x0fff0c20404185c4ull, 0x0008040003020100ull},
    {0x0fff7de75550b289ull, 0x000804001f1e1d15ull},
    {0x0fff50b2897de755ull, 0x00080400140b0a09ull}
};
const unsigned ORDER[2][8] = {
    {0x00091108u, 0x0b130a12u, 0x1b021401u, 0x1c150c1au, 0x1d160d03u, 0x1e170e04u, 0x1f180f05u, 0x07191006u},
    {0x040d150cu, 0x0f170e16u, 0x1e061805u, 0x1f19101du, 0x1a110807u, 0x1b120900u, 0x1c130a01u, 0x03140b02u}
};

__global__ __launch_bounds__(256, 1) void a2a_gemm_kernel(Params P) {
    asm volatile(
        "LSTART%=:\n\t"
        "s_memrealtime s[68:69]\n\t"
        "s_lshr_b32 s23, s2, 5\n\t"
        "s_and_b32 s26, s2, 7\n\t"
        "s_lshl3_add_u32 s26, s23, s26\n\t"
        "s_lshl_b32 s26, s26, 4\n\t"
        "s_load_dwordx8 s[8:15], s[0:1], 0x0\n\t"
        "s_load_dwordx4 s[56:59], s[0:1], s26 offset:0x140\n\t"
        "s_load_dwordx4 s[16:19], s[0:1], 0x20\n\t"
        "s_load_dwordx2 s[20:21], s[0:1], 0x30\n\t"
        "v_readfirstlane_b32 s3, v0\n\t"
        "v_and_b32 v1, 63, v0\n\t"
        "s_lshr_b32 s3, s3, 6\n\t"
        "s_bfe_u32 s4, s2, 0x20003\n\t"
        "s_mul_i32 s64, s3, 20480\n\t"
        "s_add_u32 s64, s64, 76800\n\t"
        "s_lshl2_add_u32 s35, s2, s3\n\t"
        "s_lshl_b32 s35, s35, 2\n\t"
        "s_cmp_eq_u32 s4, 3\n\t"
        "s_cbranch_scc0 LMAIN%=\n\t"
        "v_lshrrev_b32 v5, 5, v1\n\t"
        "v_xor_b32 v6, v5, v1\n\t"
        "v_and_b32 v6, 31, v6\n\t"
        "v_lshlrev_b32 v5, 9, v5\n\t"
        "v_lshl_or_b32 v32, v6, 4, v5\n\t"
        "v_xor_b32 v33, 32, v32\n\t"
        "v_xor_b32 v34, 64, v32\n\t"
        "v_xor_b32 v35, 96, v32\n\t"
        "v_xor_b32 v36, 128, v32\n\t"
        "v_xor_b32 v37, 160, v32\n\t"
        "v_xor_b32 v38, 192, v32\n\t"
        "v_xor_b32 v39, 224, v32\n\t"
        "s_mov_b32 s42, 2097152\n\t"
        "s_mov_b32 s43, 0x20000\n\t"
        "s_add_u32 m0, s64, 8192\n\t"
        "s_lshl3_add_u32 s26, s3, 0x80000\n\t"
        "s_waitcnt lgkmcnt(0)\n\t"
        "s_bfe_u32 s58, s58, s26\n\t"
        "s_lshl_b32 s83, s58, 13\n\t"
        "s_mov_b32 s40, s8\n\t"
        "s_and_b32 s41, s9, 0xffff\n\t"
        "s_cmp_eq_u32 s23, s21\n\t"
        "s_cbranch_scc1 LMAIN%=\n\t"
        "s_bitcmp1_b32 s59, 12\n\t"
        "s_cbranch_scc0 LXP%=\n\t"
        "s_cmp_gt_u32 s3, 0\n\t"
        "s_cbranch_scc0 LXP%=\n\t"
        "s_sleep 2\n\t"
        "s_cmp_gt_u32 s3, 1\n\t"
        "s_cbranch_scc0 LXP%=\n\t"
        "s_sleep 2\n\t"
        "s_cmp_gt_u32 s3, 2\n\t"
        "s_cbranch_scc0 LXP%=\n\t"
        "s_sleep 2\n\t"
        "LXP%=:\n\t"
        "buffer_load_dwordx4 v32, s[40:43], s83 offen offset:0 lds\n\t"
        "s_load_dword s65, s[12:13], s35\n\t"
        "buffer_load_dwordx4 v33, s[40:43], s83 offen offset:1024 lds\n\t"
        "buffer_load_dwordx4 v34, s[40:43], s83 offen offset:2048 lds\n\t"
        "buffer_load_dwordx4 v35, s[40:43], s83 offen offset:3072 lds\n\t"
        "s_add_u32 m0, s64, 12288\n\t"
        "s_add_u32 s83, s83, 4096\n\t"
        "buffer_load_dwordx4 v36, s[40:43], s83 offen offset:0 lds\n\t"
        "buffer_load_dwordx4 v37, s[40:43], s83 offen offset:1024 lds\n\t"
        "buffer_load_dwordx4 v38, s[40:43], s83 offen offset:2048 lds\n\t"
        "buffer_load_dwordx4 v39, s[40:43], s83 offen offset:3072 lds\n\t"
        "s_and_b32 s22, s2, 1\n\t"
        "s_lshl_b32 s22, s22, 4\n\t"
        "s_bitcmp1_b32 s59, 12\n\t"
        "s_cbranch_scc1 LNSL%=\n\t"
        "s_load_dword s94, s[16:17], 0x0\n\t"
        "s_load_dword s95, s[18:19], 0x0\n\t"
        "s_load_dwordx4 s[28:31], s[0:1], s22 offset:0xa0\n\t"
        "LNSL%=:\n\t"
        "s_mov_b32 s52, s10\n\t"
        "s_and_b32 s53, s11, 0xffff\n\t"
        "s_mov_b32 s81, -1\n\t"
        "s_mov_b32 s87, 0\n\t"
        "LNOSELF%=:\n\t"
        "v_bfe_u32 v2, v0, 4, 2\n\t"
        "v_and_b32 v3, 15, v0\n\t"
        "s_and_b32 s5, s2, 1\n\t"
        "s_and_b32 s26, s2, 6\n\t"
        "s_lshl2_add_u32 s6, s26, s23\n\t"
        "s_lshl2_add_u32 s7, s5, s3\n\t"
        "s_lshl_b32 s67, s7, 21\n\t"
        "v_and_b32 v5, 7, v0\n\t"
        "v_xor_b32 v5, v5, v2\n\t"
        "v_lshlrev_b32 v5, 4, v5\n\t"
        "v_bfe_u32 v6, v0, 3, 3\n\t"
        "v_lshl_or_b32 v5, v6, 12, v5\n\t"
        "s_lshl_b32 s26, s6, 7\n\t"
        "v_add_u32 v26, s26, v5\n\t"
        "v_xor_b32 v5, 64, v26\n\t"
        "v_add_u32 v27, 261120, v5\n\t"
        "v_add_u32 v28, 129024, v26\n\t"
        "v_add_u32 v29, 390144, v5\n\t"
        "s_mov_b32 s54, 16777216\n\t"
        "s_mov_b32 s55, 0x20000\n\t"
        "s_and_b32 s58, s58, 31\n\t"
        "s_lshl_b32 s26, s20, 5\n\t"
        "s_add_u32 s26, s26, s58\n\t"
        "s_mul_i32 s72, s26, 8448\n\t"
        "s_lshr_b32 s26, s59, 16\n\t"
        "s_add_u32 s74, s68, s26\n\t"
        "s_lshl3_add_u32 s26, s3, 0x80000\n\t"
        "s_bfe_u32 s26, 0x4c2d1e00, s26\n\t"
        "s_add_u32 s73, s74, s26\n\t"
        "s_add_u32 s75, s74, 84\n\t"
        "s_and_b32 s66, s59, 0xffff\n\t"
        "s_and_b32 s57, s57, 0xffff\n\t"
        "s_mov_b32 s58, 4325376\n\t"
        "s_mov_b32 s59, 0x20000\n\t"
        "v_lshl_or_b32 v5, v2, 2, v2\n\t"
        "v_and_b32 v5, 6, v5\n\t"
        "v_xor_b32 v5, v3, v5\n\t"
        "v_lshlrev_b32 v5, 4, v5\n\t"
        "v_lshl_or_b32 v5, v3, 9, v5\n\t"
        "v_add_u32 v40, s64, v5\n\t"
        "v_xor_b32 v41, 16, v40\n\t"
        "v_xor_b32 v42, 128, v40\n\t"
        "v_xor_b32 v43, 144, v40\n\t"
        "v_mov_b32 v15, 0xe0000000\n\t"
        "s_getpc_b64 s[98:99]\n\t"
        "LPC%=:\n\t"
        "s_add_u32 s90, s98, LSETUP%=-LPC%=\n\t"
        "s_addc_u32 s91, s99, 0\n\t"
        "s_add_u32 s98, s98, LSEND%=-LPC%=\n\t"
        "s_addc_u32 s99, s99, 0\n\t"
        "s_mov_b32 s23, 0x0\n\t"
        "s_mov_b32 s69, 0x0\n\t"
        "s_mov_b32 s82, 0x2\n\t"
        "s_swappc_b64 s[96:97], s[90:91]\n\t"
        "s_cmp_lt_u32 s66, 4096\n\t"
        "s_cbranch_scc1 LNOTFIRST%=\n\t"
        "s_waitcnt lgkmcnt(0)\n\t"
        "s_add_u32 s65, s65, 1\n\t"
        "s_bitcmp1_b32 s65, 0\n\t"
        "s_cselect_b32 s27, 2162688, 0\n\t"
        "s_bfe_u32 s26, s65, 0x20001\n\t"
        "s_or_b32 s61, s26, 0x03000000\n\t"
        "s_or_b32 s26, s26, 4\n\t"
        "s_lshl_b32 s60, s26, 29\n\t"
        "s_add_u32 s72, s72, s27\n\t"
        "v_mov_b32 v5, s60\n\t"
        "LFIRST%=:\n\t"
        "s_mov_b32 s69, 1\n\t"
        "s_waitcnt vmcnt(0)\n\t"
        "LDRY%=:\n\t"
        "ds_read_b128 v[192:195], v40 offset:8192\n\t"
        "ds_read_b128 v[196:199], v41 offset:8192\n\t"
        "ds_read_b128 v[200:203], v42 offset:8192\n\t"
        "ds_read_b128 v[204:207], v43 offset:8192\n\t"
        "ds_read_b128 v[208:211], v40 offset:8448\n\t"
        "ds_read_b128 v[212:215], v41 offset:8448\n\t"
        "ds_read_b128 v[216:219], v42 offset:8448\n\t"
        "ds_read_b128 v[220:223], v43 offset:8448\n\t"
        "v_mov_b32 v48, s61\n\t"
        "s_waitcnt lgkmcnt(7)\n\t"
        "v_bitop3_b32 v8, v195, v5, v15 bitop3:0x28\n\t"
        "v_lshrrev_b32 v8, 24, v8\n\t"
        "s_waitcnt lgkmcnt(6)\n\t"
        "v_bitop3_b32 v9, v199, v5, v15 bitop3:0x28\n\t"
        "v_lshrrev_b32 v9, 16, v9\n\t"
        "v_or3_b32 v48, v8, v9, v48\n\t"
        "s_waitcnt lgkmcnt(5)\n\t"
        "v_bitop3_b32 v8, v203, v5, v15 bitop3:0x28\n\t"
        "v_lshrrev_b32 v8, 8, v8\n\t"
        "s_waitcnt lgkmcnt(4)\n\t"
        "v_bitop3_b32 v9, v207, v5, v15 bitop3:0x28\n\t"
        "v_or3_b32 v48, v8, v9, v48\n\t"
        "s_waitcnt lgkmcnt(3)\n\t"
        "v_bitop3_b32 v8, v211, v5, v15 bitop3:0x28\n\t"
        "v_lshrrev_b32 v8, 27, v8\n\t"
        "s_waitcnt lgkmcnt(2)\n\t"
        "v_bitop3_b32 v9, v215, v5, v15 bitop3:0x28\n\t"
        "v_lshrrev_b32 v9, 19, v9\n\t"
        "v_or3_b32 v48, v8, v9, v48\n\t"
        "s_waitcnt lgkmcnt(1)\n\t"
        "v_bitop3_b32 v8, v219, v5, v15 bitop3:0x28\n\t"
        "v_lshrrev_b32 v8, 11, v8\n\t"
        "s_waitcnt lgkmcnt(0)\n\t"
        "v_bitop3_b32 v9, v223, v5, v15 bitop3:0x28\n\t"
        "v_lshrrev_b32 v9, 3, v9\n\t"
        "v_or3_b32 v48, v8, v9, v48\n\t"
        "v_lshlrev_b32 v8, 3, v48\n\t"
        "v_xor_b32_sdwa v195, v195, v48 dst_sel:BYTE_3 dst_unused:UNUSED_PRESERVE src0_sel:BYTE_3 src1_sel:BYTE_0\n\t"
        "v_xor_b32_sdwa v199, v199, v48 dst_sel:BYTE_3 dst_unused:UNUSED_PRESERVE src0_sel:BYTE_3 src1_sel:BYTE_1\n\t"
        "buffer_store_dwordx4 v[192:195], v16, s[56:59], s72 offen offset:0 sc0 sc1\n\t"
        "v_xor_b32_sdwa v203, v203, v48 dst_sel:BYTE_3 dst_unused:UNUSED_PRESERVE src0_sel:BYTE_3 src1_sel:BYTE_2\n\t"
        "buffer_store_dwordx4 v[196:199], v16, s[56:59], s72 offen offset:1024 sc0 sc1\n\t"
        "v_xor_b32_sdwa v207, v207, v48 dst_sel:BYTE_3 dst_unused:UNUSED_PRESERVE src0_sel:BYTE_3 src1_sel:BYTE_3\n\t"
        "buffer_store_dwordx4 v[200:203], v16, s[56:59], s72 offen offset:2048 sc0 sc1\n\t"
        "v_xor_b32_sdwa v211, v211, v8 dst_sel:BYTE_3 dst_unused:UNUSED_PRESERVE src0_sel:BYTE_3 src1_sel:BYTE_0\n\t"
        "buffer_store_dwordx4 v[204:207], v16, s[56:59], s72 offen offset:3072 sc0 sc1\n\t"
        "v_xor_b32_sdwa v215, v215, v8 dst_sel:BYTE_3 dst_unused:UNUSED_PRESERVE src0_sel:BYTE_3 src1_sel:BYTE_1\n\t"
        "buffer_store_dwordx4 v[208:211], v30, s[56:59], s72 offen offset:0 sc0 sc1\n\t"
        "v_xor_b32_sdwa v219, v219, v8 dst_sel:BYTE_3 dst_unused:UNUSED_PRESERVE src0_sel:BYTE_3 src1_sel:BYTE_2\n\t"
        "buffer_store_dwordx4 v[212:215], v30, s[56:59], s72 offen offset:1024 sc0 sc1\n\t"
        "v_xor_b32_sdwa v223, v223, v8 dst_sel:BYTE_3 dst_unused:UNUSED_PRESERVE src0_sel:BYTE_3 src1_sel:BYTE_3\n\t"
        "buffer_store_dwordx4 v[216:219], v30, s[56:59], s72 offen offset:2048 sc0 sc1\n\t"
        "s_nop 0\n\t"
        "buffer_store_dwordx4 v[220:223], v30, s[56:59], s72 offen offset:3072 sc0 sc1\n\t"
        "buffer_store_dword v48, v17, s[56:59], s72 offen sc0 sc1\n\t"
        "v_mov_b32 v4, 0\n\t"
        "v_mov_b32 v6, s65\n\t"
        "s_and_b64 exec, exec, 1\n\t"
        "global_store_dword v4, v6, s[24:25] sc1\n\t"
        "s_mov_b64 exec, -1\n\t"
        "s_add_u32 s74, s74, 0x70000000\n\t"
        "s_load_dword s94, s[16:17], 0x0\n\t"
        "s_load_dword s95, s[18:19], 0x0\n\t"
        "s_load_dwordx4 s[28:31], s[0:1], s22 offset:0xa0\n\t"
        "s_branch LZERO%=\n\t"
        "LNOTFIRST%=:\n\t"
        "s_cmp_gt_u32 s66, 255\n\t"
        "s_cbranch_scc1 LNOZ%=\n\t"
        "ds_write_b128 v7, v[144:147] offset:0\n\t"
        "ds_write_b128 v7, v[144:147] offset:4096\n\t"
        "ds_write_b128 v7, v[144:147] offset:8192\n\t"
        "ds_write_b128 v7, v[144:147] offset:12288\n\t"
        "ds_write_b128 v7, v[144:147] offset:16384\n\t"
        "ds_write_b128 v7, v[144:147] offset:20480\n\t"
        "ds_write_b128 v7, v[144:147] offset:24576\n\t"
        "ds_write_b128 v7, v[144:147] offset:28672\n\t"
        "ds_write_b128 v7, v[144:147] offset:32768\n\t"
        "ds_write_b128 v7, v[144:147] offset:36864\n\t"
        "ds_write_b128 v7, v[144:147] offset:40960\n\t"
        "ds_write_b128 v7, v[144:147] offset:45056\n\t"
        "ds_write_b128 v7, v[144:147] offset:49152\n\t"
        "LNOZ%=:\n\t"
        "s_waitcnt lgkmcnt(0)\n\t"
        "s_add_u32 s65, s65, 1\n\t"
        "s_bitcmp1_b32 s65, 0\n\t"
        "s_cselect_b32 s27, 2162688, 0\n\t"
        "s_bfe_u32 s26, s65, 0x20001\n\t"
        "s_or_b32 s61, s26, 0x03000000\n\t"
        "s_or_b32 s26, s26, 4\n\t"
        "s_lshl_b32 s60, s26, 29\n\t"
        "s_add_u32 s72, s72, s27\n\t"
        "v_mov_b32 v5, s60\n\t"
        "s_cmp_eq_u32 s87, 1\n\t"
        "s_waitcnt vmcnt(8)\n\t"
        "s_cbranch_scc1 LSF3%=\n\t"
        "s_waitcnt vmcnt(0)\n\t"
        "LSF3%=:\n\t"
        "ds_read_b128 v[192:195], v40 offset:8192\n\t"
        "ds_read_b128 v[196:199], v41 offset:8192\n\t"
        "ds_read_b128 v[200:203], v42 offset:8192\n\t"
        "ds_read_b128 v[204:207], v43 offset:8192\n\t"
        "ds_read_b128 v[208:211], v40 offset:8448\n\t"
        "ds_read_b128 v[212:215], v41 offset:8448\n\t"
        "ds_read_b128 v[216:219], v42 offset:8448\n\t"
        "ds_read_b128 v[220:223], v43 offset:8448\n\t"
        "v_mov_b32 v48, s61\n\t"
        "s_waitcnt lgkmcnt(7)\n\t"
        "v_bitop3_b32 v8, v195, v5, v15 bitop3:0x28\n\t"
        "v_lshrrev_b32 v8, 24, v8\n\t"
        "s_waitcnt lgkmcnt(6)\n\t"
        "v_bitop3_b32 v9, v199, v5, v15 bitop3:0x28\n\t"
        "v_lshrrev_b32 v9, 16, v9\n\t"
        "v_or3_b32 v48, v8, v9, v48\n\t"
        "s_waitcnt lgkmcnt(5)\n\t"
        "v_bitop3_b32 v8, v203, v5, v15 bitop3:0x28\n\t"
        "v_lshrrev_b32 v8, 8, v8\n\t"
        "s_waitcnt lgkmcnt(4)\n\t"
        "v_bitop3_b32 v9, v207, v5, v15 bitop3:0x28\n\t"
        "v_or3_b32 v48, v8, v9, v48\n\t"
        "s_waitcnt lgkmcnt(3)\n\t"
        "v_bitop3_b32 v8, v211, v5, v15 bitop3:0x28\n\t"
        "v_lshrrev_b32 v8, 27, v8\n\t"
        "s_waitcnt lgkmcnt(2)\n\t"
        "v_bitop3_b32 v9, v215, v5, v15 bitop3:0x28\n\t"
        "v_lshrrev_b32 v9, 19, v9\n\t"
        "v_or3_b32 v48, v8, v9, v48\n\t"
        "s_waitcnt lgkmcnt(1)\n\t"
        "v_bitop3_b32 v8, v219, v5, v15 bitop3:0x28\n\t"
        "v_lshrrev_b32 v8, 11, v8\n\t"
        "s_waitcnt lgkmcnt(0)\n\t"
        "v_bitop3_b32 v9, v223, v5, v15 bitop3:0x28\n\t"
        "v_lshrrev_b32 v9, 3, v9\n\t"
        "v_or3_b32 v48, v8, v9, v48\n\t"
        "v_lshlrev_b32 v8, 3, v48\n\t"
        "v_xor_b32_sdwa v195, v195, v48 dst_sel:BYTE_3 dst_unused:UNUSED_PRESERVE src0_sel:BYTE_3 src1_sel:BYTE_0\n\t"
        "v_xor_b32_sdwa v199, v199, v48 dst_sel:BYTE_3 dst_unused:UNUSED_PRESERVE src0_sel:BYTE_3 src1_sel:BYTE_1\n\t"
        "v_xor_b32_sdwa v203, v203, v48 dst_sel:BYTE_3 dst_unused:UNUSED_PRESERVE src0_sel:BYTE_3 src1_sel:BYTE_2\n\t"
        "v_xor_b32_sdwa v207, v207, v48 dst_sel:BYTE_3 dst_unused:UNUSED_PRESERVE src0_sel:BYTE_3 src1_sel:BYTE_3\n\t"
        "v_xor_b32_sdwa v211, v211, v8 dst_sel:BYTE_3 dst_unused:UNUSED_PRESERVE src0_sel:BYTE_3 src1_sel:BYTE_0\n\t"
        "v_xor_b32_sdwa v215, v215, v8 dst_sel:BYTE_3 dst_unused:UNUSED_PRESERVE src0_sel:BYTE_3 src1_sel:BYTE_1\n\t"
        "v_xor_b32_sdwa v219, v219, v8 dst_sel:BYTE_3 dst_unused:UNUSED_PRESERVE src0_sel:BYTE_3 src1_sel:BYTE_2\n\t"
        "v_xor_b32_sdwa v223, v223, v8 dst_sel:BYTE_3 dst_unused:UNUSED_PRESERVE src0_sel:BYTE_3 src1_sel:BYTE_3\n\t"
        "s_nop 0\n\t"
        "LNOSELF2%=:\n\t"
        "s_cmp_gt_u32 s66, 255\n\t"
        "s_cbranch_scc0 LHPRE%=\n\t"
        "s_swappc_b64 s[96:97], s[98:99]\n\t"
        "LHPRE%=:\n\t"
        "s_add_u32 m0, s64, 0\n\t"
        "s_add_u32 s83, s67, 0x0\n\t"
        "buffer_load_dwordx4 v26, s[52:55], s83 offen offset:0 lds\n\t"
        "buffer_load_dwordx4 v27, s[52:55], s83 offen offset:1024 lds\n\t"
        "buffer_load_dwordx4 v28, s[52:55], s83 offen offset:2048 lds\n\t"
        "buffer_load_dwordx4 v29, s[52:55], s83 offen offset:3072 lds\n\t"
        "s_add_u32 m0, s64, 4096\n\t"
        "s_add_u32 s83, s67, 0x108000\n\t"
        "buffer_load_dwordx4 v26, s[52:55], s83 offen offset:0 lds\n\t"
        "buffer_load_dwordx4 v27, s[52:55], s83 offen offset:1024 lds\n\t"
        "buffer_load_dwordx4 v28, s[52:55], s83 offen offset:2048 lds\n\t"
        "buffer_load_dwordx4 v29, s[52:55], s83 offen offset:3072 lds\n\t"
        "s_cmp_gt_u32 s66, 255\n\t"
        "s_cbranch_scc0 LNOPREP%=\n\t"
        "LZERO%=:\n\t"
        "ds_write_b128 v7, v[144:147] offset:0\n\t"
        "ds_write_b128 v7, v[144:147] offset:4096\n\t"
        "ds_write_b128 v7, v[144:147] offset:8192\n\t"
        "ds_write_b128 v7, v[144:147] offset:12288\n\t"
        "ds_write_b128 v7, v[144:147] offset:16384\n\t"
        "ds_write_b128 v7, v[144:147] offset:20480\n\t"
        "ds_write_b128 v7, v[144:147] offset:24576\n\t"
        "ds_write_b128 v7, v[144:147] offset:28672\n\t"
        "ds_write_b128 v7, v[144:147] offset:32768\n\t"
        "ds_write_b128 v7, v[144:147] offset:36864\n\t"
        "ds_write_b128 v7, v[144:147] offset:40960\n\t"
        "ds_write_b128 v7, v[144:147] offset:45056\n\t"
        "ds_write_b128 v7, v[144:147] offset:49152\n\t"
        "s_waitcnt lgkmcnt(0)\n\t"
        "s_branch LNOPREP%=\n\t"
        "LSEND%=:\n\t"
        "s_memrealtime s[92:93]\n\t"
        "s_waitcnt lgkmcnt(0)\n\t"
        "s_sub_i32 s84, s74, s92\n\t"
        "s_cmp_gt_i32 s84, s66\n\t"
        "s_cbranch_scc1 LSRET%=\n\t"
        "s_sub_i32 s84, s73, s92\n\t"
        "s_cmp_gt_i32 s84, 0\n\t"
        "s_cbranch_scc1 LSEND%=\n\t"
        "buffer_store_dwordx4 v[192:195], v16, s[56:59], s72 offen offset:0 sc0 sc1\n\t"
        "buffer_store_dwordx4 v[196:199], v16, s[56:59], s72 offen offset:1024 sc0 sc1\n\t"
        "buffer_store_dwordx4 v[200:203], v16, s[56:59], s72 offen offset:2048 sc0 sc1\n\t"
        "buffer_store_dwordx4 v[204:207], v16, s[56:59], s72 offen offset:3072 sc0 sc1\n\t"
        "buffer_store_dwordx4 v[208:211], v30, s[56:59], s72 offen offset:0 sc0 sc1\n\t"
        "buffer_store_dwordx4 v[212:215], v30, s[56:59], s72 offen offset:1024 sc0 sc1\n\t"
        "buffer_store_dwordx4 v[216:219], v30, s[56:59], s72 offen offset:2048 sc0 sc1\n\t"
        "buffer_store_dwordx4 v[220:223], v30, s[56:59], s72 offen offset:3072 sc0 sc1\n\t"
        "buffer_store_dword v48, v17, s[56:59], s72 offen sc0 sc1\n\t"
        "v_mov_b32 v4, 0\n\t"
        "v_mov_b32 v6, s65\n\t"
        "s_mov_b64 exec, 1\n\t"
        "global_store_dword v4, v6, s[24:25] sc1\n\t"
        "s_mov_b64 exec, -1\n\t"
        "s_mov_b32 s69, 1\n\t"
        "s_mov_b32 s35, s23\n\t"
        "s_add_u32 s74, s74, 0x70000000\n\t"
        "LSRES%=:\n\t"
        "s_memrealtime s[92:93]\n\t"
        "s_waitcnt lgkmcnt(0)\n\t"
        "s_sub_i32 s84, s75, s92\n\t"
        "s_cmp_gt_i32 s84, 0\n\t"
        "s_cbranch_scc1 LSRES%=\n\t"
        "LSRET%=:\n\t"
        "s_setpc_b64 s[96:97]\n\t"
        "LMAIN%=:\n\t"
        "s_cmp_lt_u32 s2, 8\n\t"
        "s_cbranch_scc0 LNOPF%=\n\t"
        "s_getpc_b64 s[26:27]\n\t"
        "LPF%=:\n\t"
        "s_sub_u32 s26, s26, LPF%=-LSTART%=\n\t"
        "s_subb_u32 s27, s27, 0\n\t"
        "v_lshlrev_b32 v5, 7, v0\n\t"
        "v_min_u32 v5, LEXIT%=-LSTART%=, v5\n\t"
        "global_load_dword v44, v5, s[26:27]\n\t"
        "LNOPF%=:\n\t"
        "v_bfe_u32 v2, v0, 4, 2\n\t"
        "v_and_b32 v3, 15, v0\n\t"
        "s_and_b32 s5, s2, 1\n\t"
        "s_and_b32 s26, s2, 6\n\t"
        "s_lshl2_add_u32 s6, s26, s23\n\t"
        "s_lshl2_add_u32 s7, s5, s3\n\t"
        "s_lshl_b32 s67, s7, 21\n\t"
        "v_and_b32 v5, 7, v0\n\t"
        "v_xor_b32 v5, v5, v2\n\t"
        "v_lshlrev_b32 v5, 4, v5\n\t"
        "v_bfe_u32 v6, v0, 3, 3\n\t"
        "v_lshl_or_b32 v5, v6, 12, v5\n\t"
        "s_lshl_b32 s26, s6, 7\n\t"
        "v_add_u32 v26, s26, v5\n\t"
        "v_xor_b32 v5, 64, v26\n\t"
        "v_add_u32 v27, 261120, v5\n\t"
        "v_add_u32 v28, 129024, v26\n\t"
        "v_add_u32 v29, 390144, v5\n\t"
        "s_mov_b32 s54, 16777216\n\t"
        "s_mov_b32 s55, 0x20000\n\t"
        "s_lshl1_add_u32 s27, s4, s5\n\t"
        "s_lshl_b32 s27, s27, 4\n\t"
        "s_cmp_eq_u32 s4, 3\n\t"
        "s_addc_u32 s4, s4, 0\n\t"
        "s_mov_b32 s83, 0x0\n\t"
        "s_cmp_eq_u32 s4, 1\n\t"
        "s_cselect_b32 s83, 0x108000, s83\n\t"
        "s_cmp_eq_u32 s4, 2\n\t"
        "s_cselect_b32 s83, 0x188000, s83\n\t"
        "s_cmp_eq_u32 s4, 4\n\t"
        "s_cselect_b32 s83, 0x88000, s83\n\t"
        "s_add_u32 s83, s83, s67\n\t"
        "s_mov_b32 s84, 0x8000\n\t"
        "s_cmp_eq_u32 s4, 1\n\t"
        "s_cselect_b32 s84, 0x110000, s84\n\t"
        "s_cmp_eq_u32 s4, 2\n\t"
        "s_cselect_b32 s84, 0x190000, s84\n\t"
        "s_cmp_eq_u32 s4, 4\n\t"
        "s_cselect_b32 s84, 0x90000, s84\n\t"
        "s_add_u32 s84, s84, s67\n\t"
        "s_mov_b32 m0, s64\n\t"
        "s_waitcnt lgkmcnt(0)\n\t"
        "s_mov_b32 s52, s10\n\t"
        "s_and_b32 s53, s11, 0xffff\n\t"
        "s_load_dword s65, s[12:13], s35\n\t"
        "s_cmp_eq_u32 s23, s21\n\t"
        "s_cselect_b32 s26, 128, 0\n\t"
        "s_cselect_b32 s82, 1, 0\n\t"
        "s_add_u32 s27, s27, s26\n\t"
        "s_load_dword s94, s[16:17], 0x0\n\t"
        "s_load_dword s95, s[18:19], 0x0\n\t"
        "s_load_dwordx4 s[28:31], s[0:1], s27 offset:0x40\n\t"
        "s_cmp_eq_u32 s4, 0\n\t"
        "s_cselect_b32 s82, 1, s82\n\t"
        "s_sleep 2\n\t"
        "s_sub_u32 s26, s23, s21\n\t"
        "s_and_b32 s26, s26, 7\n\t"
        "s_lshl2_add_u32 s26, s26, 0x40000\n\t"
        "s_bfe_u32 s26, 0x21077737, s26\n\t"
        "s_lshl2_add_u32 s27, s3, s26\n\t"
        "s_xor_b32 s81, s4, 2\n\t"
        "s_lshl2_add_u32 s27, s81, s27\n\t"
        "s_mov_b32 s81, -1\n\t"
        "s_cmp_lt_u32 s27, 4\n\t"
        "s_cbranch_scc1 LSM%=\n\t"
        "LSM1%=:\n\t"
        "s_cmp_eq_u32 s82, 0\n\t"
        "s_cbranch_scc1 LSETUP%=\n\t"
        "buffer_load_dwordx4 v26, s[52:55], s83 offen offset:0 lds\n\t"
        "buffer_load_dwordx4 v27, s[52:55], s83 offen offset:1024 lds\n\t"
        "buffer_load_dwordx4 v28, s[52:55], s83 offen offset:2048 lds\n\t"
        "buffer_load_dwordx4 v29, s[52:55], s83 offen offset:3072 lds\n\t"
        "s_add_u32 m0, s64, 4096\n\t"
        "s_mov_b32 s83, s84\n\t"
        "buffer_load_dwordx4 v26, s[52:55], s83 offen offset:0 lds\n\t"
        "buffer_load_dwordx4 v27, s[52:55], s83 offen offset:1024 lds\n\t"
        "buffer_load_dwordx4 v28, s[52:55], s83 offen offset:2048 lds\n\t"
        "buffer_load_dwordx4 v29, s[52:55], s83 offen offset:3072 lds\n\t"
        "LSETUP%=:\n\t"
        "s_add_u32 s24, s12, s35\n\t"
        "s_addc_u32 s25, s13, 0\n\t"
        "v_lshlrev_b32 v16, 4, v1\n\t"
        "v_add_u32 v30, 4096, v16\n\t"
        "s_lshl_b32 s26, s3, 11\n\t"
        "v_add_u32 v18, s26, v16\n\t"
        "v_lshrrev_b32 v5, 5, v1\n\t"
        "v_and_b32 v6, 31, v1\n\t"
        "s_lshl_b32 s26, s3, 2\n\t"
        "v_add_u32 v12, s26, v5\n\t"
        "v_lshlrev_b32 v13, 4, v6\n\t"
        "v_lshl_or_b32 v19, v12, 14, v13\n\t"
        "s_lshl_b32 s27, s6, 9\n\t"
        "v_add_u32 v19, s27, v19\n\t"
        "v_add_u32 v20, 0x8000, v19\n\t"
        "v_lshlrev_b32 v14, 2, v1\n\t"
        "v_or_b32 v17, 0x2000, v14\n\t"
        "v_lshrrev_b32 v6, 2, v6\n\t"
        "v_and_b32 v13, 3, v1\n\t"
        "v_lshrrev_b32 v14, 1, v13\n\t"
        "v_xor_b32 v14, v14, v12\n\t"
        "v_lshlrev_b32 v14, 3, v14\n\t"
        "v_lshl_or_b32 v14, v13, 7, v14\n\t"
        "v_mul_u32_u24 v6, 1040, v6\n\t"
        "v_add_u32 v14, 1040, v14\n\t"
        "v_add_u32 v21, v6, v14\n\t"
        "v_add_u32 v22, s64, v16\n\t"
        "v_lshlrev_b32 v23, 2, v13\n\t"
        "v_mov_b32 v24, s26\n\t"
        "v_xor_b32 v5, v5, v3\n\t"
        "v_lshlrev_b32 v5, 3, v5\n\t"
        "v_lshl_or_b32 v25, v2, 7, v5\n\t"
        "v_lshrrev_b32 v5, 1, v1\n\t"
        "v_bfe_u32 v6, v1, 2, 3\n\t"
        "v_lshlrev_b32 v6, 4, v6\n\t"
        "v_lshl_or_b32 v5, v5, 7, v6\n\t"
        "v_and_b32 v6, 1, v1\n\t"
        "v_lshl_or_b32 v5, v6, 3, v5\n\t"
        "v_add_u32 v180, s64, v5\n\t"
        "v_xor_b32 v181, 16, v180\n\t"
        "v_xor_b32 v182, 32, v180\n\t"
        "v_xor_b32 v183, 48, v180\n\t"
        "v_xor_b32 v184, 64, v180\n\t"
        "v_xor_b32 v185, 80, v180\n\t"
        "v_xor_b32 v186, 96, v180\n\t"
        "v_xor_b32 v187, 112, v180\n\t"
        "v_mov_b32 v144, 0\n\t"
        "v_mov_b32 v145, 0\n\t"
        "v_mov_b32 v146, 0\n\t"
        "v_mov_b32 v147, 0\n\t"
        "s_lshl_b32 s26, s2, 16\n\t"
        "s_add_u32 s26, s26, 10485760\n\t"
        "s_add_u32 s44, s12, s26\n\t"
        "s_addc_u32 s45, s13, 0\n\t"
        "s_and_b32 s45, s45, 0xffff\n\t"
        "s_mov_b32 s46, 40960\n\t"
        "s_mov_b32 s47, 0x20000\n\t"
        "s_xor_b32 s26, s26, 0x10000\n\t"
        "s_add_u32 s40, s12, s26\n\t"
        "s_addc_u32 s41, s13, 0\n\t"
        "s_and_b32 s41, s41, 0xffff\n\t"
        "s_mov_b32 s42, 40960\n\t"
        "s_mov_b32 s43, 0x20000\n\t"
        "s_mov_b32 s48, s14\n\t"
        "s_and_b32 s49, s15, 0xffff\n\t"
        "s_mov_b32 s50, 8388608\n\t"
        "s_mov_b32 s51, 0x20000\n\t"
        "s_add_u32 s70, s12, 16384\n\t"
        "s_addc_u32 s71, s13, 0\n\t"
        "s_mul_i32 s14, s7, 270336\n\t"
        "s_add_u32 s21, s14, 1048576\n\t"
        "s_mov_b32 s38, 270336\n\t"
        "s_mov_b32 s39, 0x20000\n\t"
        "v_lshlrev_b32 v7, 4, v0\n\t"
        "s_cmp_eq_u32 s82, 2\n\t"
        "s_cbranch_scc0 LSETM%=\n\t"
        "s_setpc_b64 s[96:97]\n\t"
        "LSETM%=:\n\t"
        "ds_write_b128 v7, v[144:147] offset:0\n\t"
        "ds_write_b128 v7, v[144:147] offset:4096\n\t"
        "ds_write_b128 v7, v[144:147] offset:8192\n\t"
        "ds_write_b128 v7, v[144:147] offset:12288\n\t"
        "ds_write_b128 v7, v[144:147] offset:16384\n\t"
        "ds_write_b128 v7, v[144:147] offset:20480\n\t"
        "ds_write_b128 v7, v[144:147] offset:24576\n\t"
        "ds_write_b128 v7, v[144:147] offset:28672\n\t"
        "ds_write_b128 v7, v[144:147] offset:32768\n\t"
        "ds_write_b128 v7, v[144:147] offset:36864\n\t"
        "ds_write_b128 v7, v[144:147] offset:40960\n\t"
        "ds_write_b128 v7, v[144:147] offset:45056\n\t"
        "ds_write_b128 v7, v[144:147] offset:49152\n\t"
        "ds_write_b128 v7, v[144:147] offset:53248\n\t"
        "ds_write_b128 v7, v[144:147] offset:57344\n\t"
        "ds_write_b128 v7, v[144:147] offset:61440\n\t"
        "v_add_u32 v7, 64512, v7\n\t"
        "ds_write_b128 v7, v[144:147] offset:0\n\t"
        "s_cmp_lg_u32 s82, 0\n\t"
        "s_cbranch_scc1 LSF1%=\n\t"
        "buffer_load_dwordx4 v26, s[52:55], s83 offen offset:0 lds\n\t"
        "buffer_load_dwordx4 v27, s[52:55], s83 offen offset:1024 lds\n\t"
        "buffer_load_dwordx4 v28, s[52:55], s83 offen offset:2048 lds\n\t"
        "buffer_load_dwordx4 v29, s[52:55], s83 offen offset:3072 lds\n\t"
        "s_add_u32 m0, s64, 4096\n\t"
        "s_mov_b32 s83, s84\n\t"
        "buffer_load_dwordx4 v26, s[52:55], s83 offen offset:0 lds\n\t"
        "buffer_load_dwordx4 v27, s[52:55], s83 offen offset:1024 lds\n\t"
        "buffer_load_dwordx4 v28, s[52:55], s83 offen offset:2048 lds\n\t"
        "buffer_load_dwordx4 v29, s[52:55], s83 offen offset:3072 lds\n\t"
        "LSF1%=:\n\t"
        "ds_write_b128 v7, v[144:147] offset:4096\n\t"
        "ds_write_b128 v7, v[144:147] offset:8192\n\t"
        "s_waitcnt lgkmcnt(0)\n\t"
        "s_add_u32 s65, s65, 1\n\t"
        "s_bitcmp1_b32 s65, 0\n\t"
        "s_cselect_b32 s27, 2162688, 0\n\t"
        "s_bfe_u32 s26, s65, 0x20001\n\t"
        "s_or_b32 s61, s26, 0x03000000\n\t"
        "s_or_b32 s26, s26, 4\n\t"
        "s_lshl_b32 s60, s26, 29\n\t"
        "s_cmp_lt_i32 s81, 0\n\t"
        "s_cbranch_scc0 LSM2%=\n\t"
        "LSM3%=:\n\t"
        "v_mov_b32 v4, 0\n\t"
        "v_mov_b32 v6, s65\n\t"
        "s_mov_b64 exec, 1\n\t"
        "global_store_dword v4, v6, s[24:25] sc1\n\t"
        "s_mov_b64 exec, -1\n\t"
        "LNOPREP%=:\n\t"
        "s_barrier\n\t"
        "v_mov_b32 v5, s94\n\t"
        "v_mul_f32 v52, s95, v5\n\t"
        "s_add_u32 s26, s21, s27\n\t"
        "s_add_u32 s36, s12, s26\n\t"
        "s_addc_u32 s37, s13, 0\n\t"
        "s_and_b32 s37, s37, 0xffff\n\t"
        "s_lshl_b32 s62, s65, 27\n\t"
        "s_bitset1_b32 s62, 31\n\t"
        "s_lshl_b32 s26, s62, 1\n\t"
        "s_sub_u32 s88, 0xf8000000, s26\n\t"
        "s_cmp_eq_u32 s3, 0\n\t"
        "s_cselect_b32 s26, s62, 0\n\t"
        "s_add_u32 s26, s26, 16777216\n\t"
        "v_mov_b32 v148, s26\n\t"
        "v_mov_b32 v150, s26\n\t"
        "v_mov_b32 v149, 0\n\t"
        "v_mov_b32 v151, 0\n\t"
        "s_mov_b32 s76, 0x0\n\t"
        "s_mov_b32 s77, 0x0\n\t"
        "s_mov_b32 s82, 0x0\n\t"
        "s_mov_b32 s86, 0x0\n\t"
        "s_mov_b32 s87, 0x8000000\n\t"
        "s_mov_b32 s91, 0x3030303\n\t"
        "s_mov_b32 s90, 0xe0000000\n\t"
        "s_mov_b32 s35, 0x0\n\t"
        "s_mov_b32 s89, 0x0\n\t"
        "s_mov_b32 s19, 0x0\n\t"
        "s_mov_b32 s16, 0x3f\n\t"
        "s_mov_b32 s11, 0x0\n\t"
        "s_bfe_u32 s34, s31, 0x80008\n\t"
        "s_bfe_u32 s79, s31, 0x80010\n\t"
        "s_lshl_b32 s26, s34, 3\n\t"
        "s_sub_u32 s63, 0x80000, s26\n\t"
        "s_lshl_b32 s26, s34, 11\n\t"
        "s_add_u32 s33, s64, s26\n\t"
        "s_mov_b32 s20, s34\n\t"
        "s_lshl_b32 s26, s3, 11\n\t"
        "s_add_u32 s26, s26, 1920\n\t"
        "v_mov_b32 v56, s26\n\t"
        "v_lshl_add_u32 v57, v1, 13, v56\n\t"
        "v_mov_b32 v55, 0\n\t"
        "s_and_b32 s21, s28, 63\n\t"
        "s_mul_i32 s78, s21, 8448\n\t"
        "s_lshr_b64 s[28:29], s[28:29], 6\n\t"
        "s_and_b32 s21, s28, 63\n\t"
        "v_mov_b32 v53, 0\n\t"
        "v_subrev_u32 v25, 8320, v25\n\t"
        "v_lshrrev_b32 v45, 3, v1\n\t"
        "v_and_b32 v46, 7, v1\n\t"
        "v_lshl_add_u32 v45, v45, 13, v56\n\t"
        "v_lshl_add_u32 v45, v46, 4, v45\n\t"
        "buffer_store_dwordx4 v[144:147], v45, s[44:47], 0 offen sc0 sc1\n\t"
        "s_cmp_eq_u32 s4, 1\n\t"
        "s_cbranch_scc1 LW1%=\n\t"
        "s_cmp_eq_u32 s4, 2\n\t"
        "s_cbranch_scc1 LW2%=\n\t"
        "s_cmp_eq_u32 s4, 3\n\t"
        "s_cbranch_scc1 LW3%=\n\t"
        "s_cmp_eq_u32 s4, 4\n\t"
        "s_cbranch_scc1 LW4%=\n\t"
        "LW0%=:\n\t"
        "s_add_u32 s83, s67, 0x10000\n\t"
        "s_add_u32 m0, s64, 8192\n\t"
        "s_nop 0\n\t"
        "buffer_load_dwordx4 v26, s[52:55], s83 offen offset:0 lds\n\t"
        "buffer_load_dwordx4 v27, s[52:55], s83 offen offset:1024 lds\n\t"
        "buffer_load_dwordx4 v28, s[52:55], s83 offen offset:2048 lds\n\t"
        "buffer_load_dwordx4 v29, s[52:55], s83 offen offset:3072 lds\n\t"
        "s_add_u32 s83, s67, 0x18000\n\t"
        "s_add_u32 m0, s64, 12288\n\t"
        "s_nop 0\n\t"
        "buffer_load_dwordx4 v26, s[52:55], s83 offen offset:0 lds\n\t"
        "buffer_load_dwordx4 v27, s[52:55], s83 offen offset:1024 lds\n\t"
        "buffer_load_dwordx4 v28, s[52:55], s83 offen offset:2048 lds\n\t"
        "buffer_load_dwordx4 v29, s[52:55], s83 offen offset:3072 lds\n\t"
        "s_waitcnt vmcnt(14)\n\t"
        "ds_read_b64_tr_b8 a[0:1], v180 offset:0\n\t"
        "ds_read_b64_tr_b8 a[4:5], v181 offset:0\n\t"
        "ds_read_b64_tr_b8 a[8:9], v182 offset:0\n\t"
        "ds_read_b64_tr_b8 a[12:13], v183 offset:0\n\t"
        "ds_read_b64_tr_b8 a[16:17], v184 offset:0\n\t"
        "ds_read_b64_tr_b8 a[20:21], v185 offset:0\n\t"
        "ds_read_b64_tr_b8 a[24:25], v186 offset:0\n\t"
        "ds_read_b64_tr_b8 a[28:29], v187 offset:0\n\t"
        "s_waitcnt lgkmcnt(0)\n\t"
        "s_add_u32 s83, s67, 0x80000\n\t"
        "s_add_u32 m0, s64, 0\n\t"
        "s_nop 0\n\t"
        "buffer_load_dwordx4 v26, s[52:55], s83 offen offset:0 lds\n\t"
        "buffer_load_dwordx4 v27, s[52:55], s83 offen offset:1024 lds\n\t"
        "buffer_load_dwordx4 v28, s[52:55], s83 offen offset:2048 lds\n\t"
        "buffer_load_dwordx4 v29, s[52:55], s83 offen offset:3072 lds\n\t"
        "s_waitcnt vmcnt(14)\n\t"
        "ds_read_b64_tr_b8 a[2:3], v180 offset:4096\n\t"
        "ds_read_b64_tr_b8 a[6:7], v181 offset:4096\n\t"
        "ds_read_b64_tr_b8 a[10:11], v182 offset:4096\n\t"
        "ds_read_b64_tr_b8 a[14:15], v183 offset:4096\n\t"
        "ds_read_b64_tr_b8 a[18:19], v184 offset:4096\n\t"
        "ds_read_b64_tr_b8 a[22:23], v185 offset:4096\n\t"
        "ds_read_b64_tr_b8 a[26:27], v186 offset:4096\n\t"
        "ds_read_b64_tr_b8 a[30:31], v187 offset:4096\n\t"
        "s_waitcnt lgkmcnt(0)\n\t"
        "s_add_u32 s83, s67, 0x88000\n\t"
        "s_add_u32 m0, s64, 4096\n\t"
        "s_nop 0\n\t"
        "buffer_load_dwordx4 v26, s[52:55], s83 offen offset:0 lds\n\t"
        "buffer_load_dwordx4 v27, s[52:55], s83 offen offset:1024 lds\n\t"
        "buffer_load_dwordx4 v28, s[52:55], s83 offen offset:2048 lds\n\t"
        "buffer_load_dwordx4 v29, s[52:55], s83 offen offset:3072 lds\n\t"
        "s_waitcnt vmcnt(12)\n\t"
        "ds_read_b64_tr_b8 a[32:33], v180 offset:8192\n\t"
        "ds_read_b64_tr_b8 a[36:37], v181 offset:8192\n\t"
        "ds_read_b64_tr_b8 a[40:41], v182 offset:8192\n\t"
        "ds_read_b64_tr_b8 a[44:45], v183 offset:8192\n\t"
        "ds_read_b64_tr_b8 a[48:49], v184 offset:8192\n\t"
        "ds_read_b64_tr_b8 a[52:53], v185 offset:8192\n\t"
        "ds_read_b64_tr_b8 a[56:57], v186 offset:8192\n\t"
        "ds_read_b64_tr_b8 a[60:61], v187 offset:8192\n\t"
        "s_waitcnt lgkmcnt(0)\n\t"
        "s_add_u32 s83, s67, 0x90000\n\t"
        "s_add_u32 m0, s64, 8192\n\t"
        "s_nop 0\n\t"
        "buffer_load_dwordx4 v26, s[52:55], s83 offen offset:0 lds\n\t"
        "buffer_load_dwordx4 v27, s[52:55], s83 offen offset:1024 lds\n\t"
        "buffer_load_dwordx4 v28, s[52:55], s83 offen offset:2048 lds\n\t"
        "buffer_load_dwordx4 v29, s[52:55], s83 offen offset:3072 lds\n\t"
        "s_waitcnt vmcnt(12)\n\t"
        "ds_read_b64_tr_b8 a[34:35], v180 offset:12288\n\t"
        "ds_read_b64_tr_b8 a[38:39], v181 offset:12288\n\t"
        "ds_read_b64_tr_b8 a[42:43], v182 offset:12288\n\t"
        "ds_read_b64_tr_b8 a[46:47], v183 offset:12288\n\t"
        "ds_read_b64_tr_b8 a[50:51], v184 offset:12288\n\t"
        "ds_read_b64_tr_b8 a[54:55], v185 offset:12288\n\t"
        "ds_read_b64_tr_b8 a[58:59], v186 offset:12288\n\t"
        "ds_read_b64_tr_b8 a[62:63], v187 offset:12288\n\t"
        "s_waitcnt lgkmcnt(0)\n\t"
        "s_add_u32 s83, s67, 0x98000\n\t"
        "s_add_u32 m0, s64, 12288\n\t"
        "s_nop 0\n\t"
        "buffer_load_dwordx4 v26, s[52:55], s83 offen offset:0 lds\n\t"
        "buffer_load_dwordx4 v27, s[52:55], s83 offen offset:1024 lds\n\t"
        "buffer_load_dwordx4 v28, s[52:55], s83 offen offset:2048 lds\n\t"
        "buffer_load_dwordx4 v29, s[52:55], s83 offen offset:3072 lds\n\t"
        "s_waitcnt vmcnt(12)\n\t"
        "ds_read_b64_tr_b8 a[64:65], v180 offset:0\n\t"
        "ds_read_b64_tr_b8 a[68:69], v181 offset:0\n\t"
        "ds_read_b64_tr_b8 a[72:73], v182 offset:0\n\t"
        "ds_read_b64_tr_b8 a[76:77], v183 offset:0\n\t"
        "ds_read_b64_tr_b8 a[80:81], v184 offset:0\n\t"
        "ds_read_b64_tr_b8 a[84:85], v185 offset:0\n\t"
        "ds_read_b64_tr_b8 a[88:89], v186 offset:0\n\t"
        "ds_read_b64_tr_b8 a[92:93], v187 offset:0\n\t"
        "s_waitcnt lgkmcnt(0)\n\t"
        "s_add_u32 s83, s67, 0x100000\n\t"
        "s_add_u32 m0, s64, 0\n\t"
        "s_nop 0\n\t"
        "buffer_load_dwordx4 v26, s[52:55], s83 offen offset:0 lds\n\t"
        "buffer_load_dwordx4 v27, s[52:55], s83 offen offset:1024 lds\n\t"
        "buffer_load_dwordx4 v28, s[52:55], s83 offen offset:2048 lds\n\t"
        "buffer_load_dwordx4 v29, s[52:55], s83 offen offset:3072 lds\n\t"
        "s_waitcnt vmcnt(12)\n\t"
        "ds_read_b64_tr_b8 a[66:67], v180 offset:4096\n\t"
        "ds_read_b64_tr_b8 a[70:71], v181 offset:4096\n\t"
        "ds_read_b64_tr_b8 a[74:75], v182 offset:4096\n\t"
        "ds_read_b64_tr_b8 a[78:79], v183 offset:4096\n\t"
        "ds_read_b64_tr_b8 a[82:83], v184 offset:4096\n\t"
        "ds_read_b64_tr_b8 a[86:87], v185 offset:4096\n\t"
        "ds_read_b64_tr_b8 a[90:91], v186 offset:4096\n\t"
        "ds_read_b64_tr_b8 a[94:95], v187 offset:4096\n\t"
        "s_waitcnt lgkmcnt(0)\n\t"
        "s_add_u32 s83, s67, 0x108000\n\t"
        "s_add_u32 m0, s64, 4096\n\t"
        "s_nop 0\n\t"
        "buffer_load_dwordx4 v26, s[52:55], s83 offen offset:0 lds\n\t"
        "buffer_load_dwordx4 v27, s[52:55], s83 offen offset:1024 lds\n\t"
        "buffer_load_dwordx4 v28, s[52:55], s83 offen offset:2048 lds\n\t"
        "buffer_load_dwordx4 v29, s[52:55], s83 offen offset:3072 lds\n\t"
        "s_waitcnt vmcnt(12)\n\t"
        "ds_read_b64_tr_b8 a[96:97], v180 offset:8192\n\t"
        "ds_read_b64_tr_b8 a[100:101], v181 offset:8192\n\t"
        "ds_read_b64_tr_b8 a[104:105], v182 offset:8192\n\t"
        "ds_read_b64_tr_b8 a[108:109], v183 offset:8192\n\t"
        "ds_read_b64_tr_b8 a[112:113], v184 offset:8192\n\t"
        "ds_read_b64_tr_b8 a[116:117], v185 offset:8192\n\t"
        "ds_read_b64_tr_b8 a[120:121], v186 offset:8192\n\t"
        "ds_read_b64_tr_b8 a[124:125], v187 offset:8192\n\t"
        "s_waitcnt lgkmcnt(0)\n\t"
        "s_add_u32 s83, s67, 0x188000\n\t"
        "s_add_u32 m0, s64, 8192\n\t"
        "s_nop 0\n\t"
        "buffer_load_dwordx4 v26, s[52:55], s83 offen offset:0 lds\n\t"
        "buffer_load_dwordx4 v27, s[52:55], s83 offen offset:1024 lds\n\t"
        "buffer_load_dwordx4 v28, s[52:55], s83 offen offset:2048 lds\n\t"
        "buffer_load_dwordx4 v29, s[52:55], s83 offen offset:3072 lds\n\t"
        "s_waitcnt vmcnt(12)\n\t"
        "ds_read_b64_tr_b8 a[98:99], v180 offset:12288\n\t"
        "ds_read_b64_tr_b8 a[102:103], v181 offset:12288\n\t"
        "ds_read_b64_tr_b8 a[106:107], v182 offset:12288\n\t"
        "ds_read_b64_tr_b8 a[110:111], v183 offset:12288\n\t"
        "ds_read_b64_tr_b8 a[114:115], v184 offset:12288\n\t"
        "ds_read_b64_tr_b8 a[118:119], v185 offset:12288\n\t"
        "ds_read_b64_tr_b8 a[122:123], v186 offset:12288\n\t"
        "ds_read_b64_tr_b8 a[126:127], v187 offset:12288\n\t"
        "s_waitcnt lgkmcnt(0)\n\t"
        "s_add_u32 s83, s67, 0x110000\n\t"
        "s_add_u32 m0, s64, 12288\n\t"
        "s_nop 0\n\t"
        "buffer_load_dwordx4 v26, s[52:55], s83 offen offset:0 lds\n\t"
        "buffer_load_dwordx4 v27, s[52:55], s83 offen offset:1024 lds\n\t"
        "buffer_load_dwordx4 v28, s[52:55], s83 offen offset:2048 lds\n\t"
        "buffer_load_dwordx4 v29, s[52:55], s83 offen offset:3072 lds\n\t"
        "s_waitcnt vmcnt(12)\n\t"
        "ds_read_b64_tr_b8 a[128:129], v180 offset:0\n\t"
        "ds_read_b64_tr_b8 a[132:133], v181 offset:0\n\t"
        "ds_read_b64_tr_b8 a[136:137], v182 offset:0\n\t"
        "ds_read_b64_tr_b8 a[140:141], v183 offset:0\n\t"
        "ds_read_b64_tr_b8 a[144:145], v184 offset:0\n\t"
        "ds_read_b64_tr_b8 a[148:149], v185 offset:0\n\t"
        "ds_read_b64_tr_b8 a[152:153], v186 offset:0\n\t"
        "ds_read_b64_tr_b8 a[156:157], v187 offset:0\n\t"
        "s_waitcnt lgkmcnt(0)\n\t"
        "s_add_u32 s83, s67, 0x190000\n\t"
        "s_add_u32 m0, s64, 0\n\t"
        "s_nop 0\n\t"
        "buffer_load_dwordx4 v26, s[52:55], s83 offen offset:0 lds\n\t"
        "buffer_load_dwordx4 v27, s[52:55], s83 offen offset:1024 lds\n\t"
        "buffer_load_dwordx4 v28, s[52:55], s83 offen offset:2048 lds\n\t"
        "buffer_load_dwordx4 v29, s[52:55], s83 offen offset:3072 lds\n\t"
        "s_waitcnt vmcnt(12)\n\t"
        "ds_read_b64_tr_b8 a[130:131], v180 offset:4096\n\t"
        "ds_read_b64_tr_b8 a[134:135], v181 offset:4096\n\t"
        "ds_read_b64_tr_b8 a[138:139], v182 offset:4096\n\t"
        "ds_read_b64_tr_b8 a[142:143], v183 offset:4096\n\t"
        "ds_read_b64_tr_b8 a[146:147], v184 offset:4096\n\t"
        "ds_read_b64_tr_b8 a[150:151], v185 offset:4096\n\t"
        "ds_read_b64_tr_b8 a[154:155], v186 offset:4096\n\t"
        "ds_read_b64_tr_b8 a[158:159], v187 offset:4096\n\t"
        "s_waitcnt lgkmcnt(0)\n\t"
        "s_add_u32 s83, s67, 0x118000\n\t"
        "s_add_u32 m0, s64, 4096\n\t"
        "s_nop 0\n\t"
        "buffer_load_dwordx4 v26, s[52:55], s83 offen offset:0 lds\n\t"
        "buffer_load_dwordx4 v27, s[52:55], s83 offen offset:1024 lds\n\t"
        "buffer_load_dwordx4 v28, s[52:55], s83 offen offset:2048 lds\n\t"
        "buffer_load_dwordx4 v29, s[52:55], s83 offen offset:3072 lds\n\t"
        "s_waitcnt vmcnt(12)\n\t"
        "ds_read_b64_tr_b8 a[194:195], v180 offset:8192\n\t"
        "ds_read_b64_tr_b8 a[198:199], v181 offset:8192\n\t"
        "ds_read_b64_tr_b8 a[202:203], v182 offset:8192\n\t"
        "ds_read_b64_tr_b8 a[206:207], v183 offset:8192\n\t"
        "ds_read_b64_tr_b8 a[210:211], v184 offset:8192\n\t"
        "ds_read_b64_tr_b8 a[214:215], v185 offset:8192\n\t"
        "ds_read_b64_tr_b8 a[218:219], v186 offset:8192\n\t"
        "ds_read_b64_tr_b8 a[222:223], v187 offset:8192\n\t"
        "s_waitcnt lgkmcnt(0)\n\t"
        "s_add_u32 s83, s67, 0x198000\n\t"
        "s_add_u32 m0, s64, 8192\n\t"
        "s_nop 0\n\t"
        "buffer_load_dwordx4 v26, s[52:55], s83 offen offset:0 lds\n\t"
        "buffer_load_dwordx4 v27, s[52:55], s83 offen offset:1024 lds\n\t"
        "buffer_load_dwordx4 v28, s[52:55], s83 offen offset:2048 lds\n\t"
        "buffer_load_dwordx4 v29, s[52:55], s83 offen offset:3072 lds\n\t"
        "s_waitcnt vmcnt(12)\n\t"
        "ds_read_b64_tr_b8 a[160:161], v180 offset:12288\n\t"
        "ds_read_b64_tr_b8 a[164:165], v181 offset:12288\n\t"
        "ds_read_b64_tr_b8 a[168:169], v182 offset:12288\n\t"
        "ds_read_b64_tr_b8 a[172:173], v183 offset:12288\n\t"
        "ds_read_b64_tr_b8 a[176:177], v184 offset:12288\n\t"
        "ds_read_b64_tr_b8 a[180:181], v185 offset:12288\n\t"
        "ds_read_b64_tr_b8 a[184:185], v186 offset:12288\n\t"
        "ds_read_b64_tr_b8 a[188:189], v187 offset:12288\n\t"
        "s_waitcnt lgkmcnt(0)\n\t"
        "s_add_u32 s83, s67, 0x180000\n\t"
        "s_add_u32 m0, s64, 12288\n\t"
        "s_nop 0\n\t"
        "buffer_load_dwordx4 v26, s[52:55], s83 offen offset:0 lds\n\t"
        "buffer_load_dwordx4 v27, s[52:55], s83 offen offset:1024 lds\n\t"
        "buffer_load_dwordx4 v28, s[52:55], s83 offen offset:2048 lds\n\t"
        "buffer_load_dwordx4 v29, s[52:55], s83 offen offset:3072 lds\n\t"
        "buffer_load_dwordx4 v[64:67], v16, s[36:39], s78 offen offset:0 sc0 sc1\n\t"
        "buffer_load_dwordx4 v[68:71], v16, s[36:39], s78 offen offset:1024 sc0 sc1\n\t"
        "buffer_load_dwordx4 v[72:75], v16, s[36:39], s78 offen offset:2048 sc0 sc1\n\t"
        "buffer_load_dwordx4 v[76:79], v16, s[36:39], s78 offen offset:3072 sc0 sc1\n\t"
        "buffer_load_dwordx4 v[80:83], v30, s[36:39], s78 offen offset:0 sc0 sc1\n\t"
        "buffer_load_dwordx4 v[84:87], v30, s[36:39], s78 offen offset:1024 sc0 sc1\n\t"
        "buffer_load_dwordx4 v[88:91], v30, s[36:39], s78 offen offset:2048 sc0 sc1\n\t"
        "buffer_load_dwordx4 v[92:95], v30, s[36:39], s78 offen offset:3072 sc0 sc1\n\t"
        "buffer_load_dword v128, v17, s[36:39], s78 offen sc0 sc1\n\t"
        "s_waitcnt vmcnt(21)\n\t"
        "ds_read_b64_tr_b8 a[224:225], v180 offset:0\n\t"
        "ds_read_b64_tr_b8 a[228:229], v181 offset:0\n\t"
        "ds_read_b64_tr_b8 a[232:233], v182 offset:0\n\t"
        "ds_read_b64_tr_b8 a[236:237], v183 offset:0\n\t"
        "ds_read_b64_tr_b8 a[240:241], v184 offset:0\n\t"
        "ds_read_b64_tr_b8 a[244:245], v185 offset:0\n\t"
        "ds_read_b64_tr_b8 a[248:249], v186 offset:0\n\t"
        "ds_read_b64_tr_b8 a[252:253], v187 offset:0\n\t"
        "s_waitcnt lgkmcnt(0)\n\t"
        "s_waitcnt vmcnt(17)\n\t"
        "ds_read_b64_tr_b8 a[162:163], v180 offset:4096\n\t"
        "ds_read_b64_tr_b8 a[166:167], v181 offset:4096\n\t"
        "ds_read_b64_tr_b8 a[170:171], v182 offset:4096\n\t"
        "ds_read_b64_tr_b8 a[174:175], v183 offset:4096\n\t"
        "ds_read_b64_tr_b8 a[178:179], v184 offset:4096\n\t"
        "ds_read_b64_tr_b8 a[182:183], v185 offset:4096\n\t"
        "ds_read_b64_tr_b8 a[186:187], v186 offset:4096\n\t"
        "ds_read_b64_tr_b8 a[190:191], v187 offset:4096\n\t"
        "s_waitcnt lgkmcnt(0)\n\t"
        "s_waitcnt vmcnt(13)\n\t"
        "ds_read_b64_tr_b8 a[226:227], v180 offset:8192\n\t"
        "ds_read_b64_tr_b8 a[230:231], v181 offset:8192\n\t"
        "ds_read_b64_tr_b8 a[234:235], v182 offset:8192\n\t"
        "ds_read_b64_tr_b8 a[238:239], v183 offset:8192\n\t"
        "ds_read_b64_tr_b8 a[242:243], v184 offset:8192\n\t"
        "ds_read_b64_tr_b8 a[246:247], v185 offset:8192\n\t"
        "ds_read_b64_tr_b8 a[250:251], v186 offset:8192\n\t"
        "ds_read_b64_tr_b8 a[254:255], v187 offset:8192\n\t"
        "s_waitcnt lgkmcnt(0)\n\t"
        "s_waitcnt vmcnt(9)\n\t"
        "ds_read_b64_tr_b8 a[192:193], v180 offset:12288\n\t"
        "ds_read_b64_tr_b8 a[196:197], v181 offset:12288\n\t"
        "ds_read_b64_tr_b8 a[200:201], v182 offset:12288\n\t"
        "ds_read_b64_tr_b8 a[204:205], v183 offset:12288\n\t"
        "ds_read_b64_tr_b8 a[208:209], v184 offset:12288\n\t"
        "ds_read_b64_tr_b8 a[212:213], v185 offset:12288\n\t"
        "ds_read_b64_tr_b8 a[216:217], v186 offset:12288\n\t"
        "ds_read_b64_tr_b8 a[220:221], v187 offset:12288\n\t"
        "s_waitcnt lgkmcnt(0)\n\t"
        "s_branch LWAIT%=\n\t"
        "LW1%=:\n\t"
        "s_add_u32 s83, s67, 0x118000\n\t"
        "s_add_u32 m0, s64, 8192\n\t"
        "s_nop 0\n\t"
        "buffer_load_dwordx4 v26, s[52:55], s83 offen offset:0 lds\n\t"
        "buffer_load_dwordx4 v27, s[52:55], s83 offen offset:1024 lds\n\t"
        "buffer_load_dwordx4 v28, s[52:55], s83 offen offset:2048 lds\n\t"
        "buffer_load_dwordx4 v29, s[52:55], s83 offen offset:3072 lds\n\t"
        "s_add_u32 s83, s67, 0x180000\n\t"
        "s_add_u32 m0, s64, 12288\n\t"
        "s_nop 0\n\t"
        "buffer_load_dwordx4 v26, s[52:55], s83 offen offset:0 lds\n\t"
        "buffer_load_dwordx4 v27, s[52:55], s83 offen offset:1024 lds\n\t"
        "buffer_load_dwordx4 v28, s[52:55], s83 offen offset:2048 lds\n\t"
        "buffer_load_dwordx4 v29, s[52:55], s83 offen offset:3072 lds\n\t"
        "s_waitcnt vmcnt(14)\n\t"
        "ds_read_b64_tr_b8 a[130:131], v180 offset:0\n\t"
        "ds_read_b64_tr_b8 a[134:135], v181 offset:0\n\t"
        "ds_read_b64_tr_b8 a[138:139], v182 offset:0\n\t"
        "ds_read_b64_tr_b8 a[142:143], v183 offset:0\n\t"
        "ds_read_b64_tr_b8 a[146:147], v184 offset:0\n\t"
        "ds_read_b64_tr_b8 a[150:151], v185 offset:0\n\t"
        "ds_read_b64_tr_b8 a[154:155], v186 offset:0\n\t"
        "ds_read_b64_tr_b8 a[158:159], v187 offset:0\n\t"
        "s_waitcnt lgkmcnt(0)\n\t"
        "s_add_u32 s83, s67, 0x188000\n\t"
        "s_add_u32 m0, s64, 0\n\t"
        "s_nop 0\n\t"
        "buffer_load_dwordx4 v26, s[52:55], s83 offen offset:0 lds\n\t"
        "buffer_load_dwordx4 v27, s[52:55], s83 offen offset:1024 lds\n\t"
        "buffer_load_dwordx4 v28, s[52:55], s83 offen offset:2048 lds\n\t"
        "buffer_load_dwordx4 v29, s[52:55], s83 offen offset:3072 lds\n\t"
        "s_waitcnt vmcnt(14)\n\t"
        "ds_read_b64_tr_b8 a[160:161], v180 offset:4096\n\t"
        "ds_read_b64_tr_b8 a[164:165], v181 offset:4096\n\t"
        "ds_read_b64_tr_b8 a[168:169], v182 offset:4096\n\t"
        "ds_read_b64_tr_b8 a[172:173], v183 offset:4096\n\t"
        "ds_read_b64_tr_b8 a[176:177], v184 offset:4096\n\t"
        "ds_read_b64_tr_b8 a[180:181], v185 offset:4096\n\t"
        "ds_read_b64_tr_b8 a[184:185], v186 offset:4096\n\t"
        "ds_read_b64_tr_b8 a[188:189], v187 offset:4096\n\t"
        "s_waitcnt lgkmcnt(0)\n\t"
        "s_add_u32 s83, s67, 0x0\n\t"
        "s_add_u32 m0, s64, 4096\n\t"
        "s_nop 0\n\t"
        "buffer_load_dwordx4 v26, s[52:55], s83 offen offset:0 lds\n\t"
        "buffer_load_dwordx4 v27, s[52:55], s83 offen offset:1024 lds\n\t"
        "buffer_load_dwordx4 v28, s[52:55], s83 offen offset:2048 lds\n\t"
        "buffer_load_dwordx4 v29, s[52:55], s83 offen offset:3072 lds\n\t"
        "s_waitcnt vmcnt(12)\n\t"
        "ds_read_b64_tr_b8 a[162:163], v180 offset:8192\n\t"
        "ds_read_b64_tr_b8 a[166:167], v181 offset:8192\n\t"
        "ds_read_b64_tr_b8 a[170:171], v182 offset:8192\n\t"
        "ds_read_b64_tr_b8 a[174:175], v183 offset:8192\n\t"
        "ds_read_b64_tr_b8 a[178:179], v184 offset:8192\n\t"
        "ds_read_b64_tr_b8 a[182:183], v185 offset:8192\n\t"
        "ds_read_b64_tr_b8 a[186:187], v186 offset:8192\n\t"
        "ds_read_b64_tr_b8 a[190:191], v187 offset:8192\n\t"
        "s_waitcnt lgkmcnt(0)\n\t"
        "s_add_u32 s83, s67, 0x190000\n\t"
        "s_add_u32 m0, s64, 8192\n\t"
        "s_nop 0\n\t"
        "buffer_load_dwordx4 v26, s[52:55], s83 offen offset:0 lds\n\t"
        "buffer_load_dwordx4 v27, s[52:55], s83 offen offset:1024 lds\n\t"
        "buffer_load_dwordx4 v28, s[52:55], s83 offen offset:2048 lds\n\t"
        "buffer_load_dwordx4 v29, s[52:55], s83 offen offset:3072 lds\n\t"
        "s_waitcnt vmcnt(12)\n\t"
        "ds_read_b64_tr_b8 a[192:193], v180 offset:12288\n\t"
        "ds_read_b64_tr_b8 a[196:197], v181 offset:12288\n\t"
        "ds_read_b64_tr_b8 a[200:201], v182 offset:12288\n\t"
        "ds_read_b64_tr_b8 a[204:205], v183 offset:12288\n\t"
        "ds_read_b64_tr_b8 a[208:209], v184 offset:12288\n\t"
        "ds_read_b64_tr_b8 a[212:213], v185 offset:12288\n\t"
        "ds_read_b64_tr_b8 a[216:217], v186 offset:12288\n\t"
        "ds_read_b64_tr_b8 a[220:221], v187 offset:12288\n\t"
        "s_waitcnt lgkmcnt(0)\n\t"
        "s_add_u32 s83, s67, 0x8000\n\t"
        "s_add_u32 m0, s64, 12288\n\t"
        "s_nop 0\n\t"
        "buffer_load_dwordx4 v26, s[52:55], s83 offen offset:0 lds\n\t"
        "buffer_load_dwordx4 v27, s[52:55], s83 offen offset:1024 lds\n\t"
        "buffer_load_dwordx4 v28, s[52:55], s83 offen offset:2048 lds\n\t"
        "buffer_load_dwordx4 v29, s[52:55], s83 offen offset:3072 lds\n\t"
        "s_waitcnt vmcnt(12)\n\t"
        "ds_read_b64_tr_b8 a[194:195], v180 offset:0\n\t"
        "ds_read_b64_tr_b8 a[198:199], v181 offset:0\n\t"
        "ds_read_b64_tr_b8 a[202:203], v182 offset:0\n\t"
        "ds_read_b64_tr_b8 a[206:207], v183 offset:0\n\t"
        "ds_read_b64_tr_b8 a[210:211], v184 offset:0\n\t"
        "ds_read_b64_tr_b8 a[214:215], v185 offset:0\n\t"
        "ds_read_b64_tr_b8 a[218:219], v186 offset:0\n\t"
        "ds_read_b64_tr_b8 a[222:223], v187 offset:0\n\t"
        "s_waitcnt lgkmcnt(0)\n\t"
        "s_add_u32 s83, s67, 0x198000\n\t"
        "s_add_u32 m0, s64, 0\n\t"
        "s_nop 0\n\t"
        "buffer_load_dwordx4 v26, s[52:55], s83 offen offset:0 lds\n\t"
        "buffer_load_dwordx4 v27, s[52:55], s83 offen offset:1024 lds\n\t"
        "buffer_load_dwordx4 v28, s[52:55], s83 offen offset:2048 lds\n\t"
        "buffer_load_dwordx4 v29, s[52:55], s83 offen offset:3072 lds\n\t"
        "s_waitcnt vmcnt(12)\n\t"
        "ds_read_b64_tr_b8 a[0:1], v180 offset:4096\n\t"
        "ds_read_b64_tr_b8 a[4:5], v181 offset:4096\n\t"
        "ds_read_b64_tr_b8 a[8:9], v182 offset:4096\n\t"
        "ds_read_b64_tr_b8 a[12:13], v183 offset:4096\n\t"
        "ds_read_b64_tr_b8 a[16:17], v184 offset:4096\n\t"
        "ds_read_b64_tr_b8 a[20:21], v185 offset:4096\n\t"
        "ds_read_b64_tr_b8 a[24:25], v186 offset:4096\n\t"
        "ds_read_b64_tr_b8 a[28:29], v187 offset:4096\n\t"
        "s_waitcnt lgkmcnt(0)\n\t"
        "s_add_u32 s83, s67, 0x10000\n\t"
        "s_add_u32 m0, s64, 4096\n\t"
        "s_nop 0\n\t"
        "buffer_load_dwordx4 v26, s[52:55], s83 offen offset:0 lds\n\t"
        "buffer_load_dwordx4 v27, s[52:55], s83 offen offset:1024 lds\n\t"
        "buffer_load_dwordx4 v28, s[52:55], s83 offen offset:2048 lds\n\t"
        "buffer_load_dwordx4 v29, s[52:55], s83 offen offset:3072 lds\n\t"
        "s_waitcnt vmcnt(12)\n\t"
        "ds_read_b64_tr_b8 a[224:225], v180 offset:8192\n\t"
        "ds_read_b64_tr_b8 a[228:229], v181 offset:8192\n\t"
        "ds_read_b64_tr_b8 a[232:233], v182 offset:8192\n\t"
        "ds_read_b64_tr_b8 a[236:237], v183 offset:8192\n\t"
        "ds_read_b64_tr_b8 a[240:241], v184 offset:8192\n\t"
        "ds_read_b64_tr_b8 a[244:245], v185 offset:8192\n\t"
        "ds_read_b64_tr_b8 a[248:249], v186 offset:8192\n\t"
        "ds_read_b64_tr_b8 a[252:253], v187 offset:8192\n\t"
        "s_waitcnt lgkmcnt(0)\n\t"
        "s_add_u32 s83, s67, 0x18000\n\t"
        "s_add_u32 m0, s64, 8192\n\t"
        "s_nop 0\n\t"
        "buffer_load_dwordx4 v26, s[52:55], s83 offen offset:0 lds\n\t"
        "buffer_load_dwordx4 v27, s[52:55], s83 offen offset:1024 lds\n\t"
        "buffer_load_dwordx4 v28, s[52:55], s83 offen offset:2048 lds\n\t"
        "buffer_load_dwordx4 v29, s[52:55], s83 offen offset:3072 lds\n\t"
        "s_waitcnt vmcnt(12)\n\t"
        "ds_read_b64_tr_b8 a[2:3], v180 offset:12288\n\t"
        "ds_read_b64_tr_b8 a[6:7], v181 offset:12288\n\t"
        "ds_read_b64_tr_b8 a[10:11], v182 offset:12288\n\t"
        "ds_read_b64_tr_b8 a[14:15], v183 offset:12288\n\t"
        "ds_read_b64_tr_b8 a[18:19], v184 offset:12288\n\t"
        "ds_read_b64_tr_b8 a[22:23], v185 offset:12288\n\t"
        "ds_read_b64_tr_b8 a[26:27], v186 offset:12288\n\t"
        "ds_read_b64_tr_b8 a[30:31], v187 offset:12288\n\t"
        "s_waitcnt lgkmcnt(0)\n\t"
        "s_add_u32 s83, s67, 0x80000\n\t"
        "s_add_u32 m0, s64, 12288\n\t"
        "s_nop 0\n\t"
        "buffer_load_dwordx4 v26, s[52:55], s83 offen offset:0 lds\n\t"
        "buffer_load_dwordx4 v27, s[52:55], s83 offen offset:1024 lds\n\t"
        "buffer_load_dwordx4 v28, s[52:55], s83 offen offset:2048 lds\n\t"
        "buffer_load_dwordx4 v29, s[52:55], s83 offen offset:3072 lds\n\t"
        "s_waitcnt vmcnt(12)\n\t"
        "ds_read_b64_tr_b8 a[226:227], v180 offset:0\n\t"
        "ds_read_b64_tr_b8 a[230:231], v181 offset:0\n\t"
        "ds_read_b64_tr_b8 a[234:235], v182 offset:0\n\t"
        "ds_read_b64_tr_b8 a[238:239], v183 offset:0\n\t"
        "ds_read_b64_tr_b8 a[242:243], v184 offset:0\n\t"
        "ds_read_b64_tr_b8 a[246:247], v185 offset:0\n\t"
        "ds_read_b64_tr_b8 a[250:251], v186 offset:0\n\t"
        "ds_read_b64_tr_b8 a[254:255], v187 offset:0\n\t"
        "s_waitcnt lgkmcnt(0)\n\t"
        "s_add_u32 s83, s67, 0x88000\n\t"
        "s_add_u32 m0, s64, 0\n\t"
        "s_nop 0\n\t"
        "buffer_load_dwordx4 v26, s[52:55], s83 offen offset:0 lds\n\t"
        "buffer_load_dwordx4 v27, s[52:55], s83 offen offset:1024 lds\n\t"
        "buffer_load_dwordx4 v28, s[52:55], s83 offen offset:2048 lds\n\t"
        "buffer_load_dwordx4 v29, s[52:55], s83 offen offset:3072 lds\n\t"
        "s_waitcnt vmcnt(12)\n\t"
        "ds_read_b64_tr_b8 a[32:33], v180 offset:4096\n\t"
        "ds_read_b64_tr_b8 a[36:37], v181 offset:4096\n\t"
        "ds_read_b64_tr_b8 a[40:41], v182 offset:4096\n\t"
        "ds_read_b64_tr_b8 a[44:45], v183 offset:4096\n\t"
        "ds_read_b64_tr_b8 a[48:49], v184 offset:4096\n\t"
        "ds_read_b64_tr_b8 a[52:53], v185 offset:4096\n\t"
        "ds_read_b64_tr_b8 a[56:57], v186 offset:4096\n\t"
        "ds_read_b64_tr_b8 a[60:61], v187 offset:4096\n\t"
        "s_waitcnt lgkmcnt(0)\n\t"
        "s_add_u32 s83, s67, 0x90000\n\t"
        "s_add_u32 m0, s64, 4096\n\t"
        "s_nop 0\n\t"
        "buffer_load_dwordx4 v26, s[52:55], s83 offen offset:0 lds\n\t"
        "buffer_load_dwordx4 v27, s[52:55], s83 offen offset:1024 lds\n\t"
        "buffer_load_dwordx4 v28, s[52:55], s83 offen offset:2048 lds\n\t"
        "buffer_load_dwordx4 v29, s[52:55], s83 offen offset:3072 lds\n\t"
        "s_waitcnt vmcnt(12)\n\t"
        "ds_read_b64_tr_b8 a[34:35], v180 offset:8192\n\t"
        "ds_read_b64_tr_b8 a[38:39], v181 offset:8192\n\t"
        "ds_read_b64_tr_b8 a[42:43], v182 offset:8192\n\t"
        "ds_read_b64_tr_b8 a[46:47], v183 offset:8192\n\t"
        "ds_read_b64_tr_b8 a[50:51], v184 offset:8192\n\t"
        "ds_read_b64_tr_b8 a[54:55], v185 offset:8192\n\t"
        "ds_read_b64_tr_b8 a[58:59], v186 offset:8192\n\t"
        "ds_read_b64_tr_b8 a[62:63], v187 offset:8192\n\t"
        "s_waitcnt lgkmcnt(0)\n\t"
        "s_add_u32 s83, s67, 0x98000\n\t"
        "s_add_u32 m0, s64, 8192\n\t"
        "s_nop 0\n\t"
        "buffer_load_dwordx4 v26, s[52:55], s83 offen offset:0 lds\n\t"
        "buffer_load_dwordx4 v27, s[52:55], s83 offen offset:1024 lds\n\t"
        "buffer_load_dwordx4 v28, s[52:55], s83 offen offset:2048 lds\n\t"
        "buffer_load_dwordx4 v29, s[52:55], s83 offen offset:3072 lds\n\t"
        "s_waitcnt vmcnt(12)\n\t"
        "ds_read_b64_tr_b8 a[64:65], v180 offset:12288\n\t"
        "ds_read_b64_tr_b8 a[68:69], v181 offset:12288\n\t"
        "ds_read_b64_tr_b8 a[72:73], v182 offset:12288\n\t"
        "ds_read_b64_tr_b8 a[76:77], v183 offset:12288\n\t"
        "ds_read_b64_tr_b8 a[80:81], v184 offset:12288\n\t"
        "ds_read_b64_tr_b8 a[84:85], v185 offset:12288\n\t"
        "ds_read_b64_tr_b8 a[88:89], v186 offset:12288\n\t"
        "ds_read_b64_tr_b8 a[92:93], v187 offset:12288\n\t"
        "s_waitcnt lgkmcnt(0)\n\t"
        "s_add_u32 s83, s67, 0x100000\n\t"
        "s_add_u32 m0, s64, 12288\n\t"
        "s_nop 0\n\t"
        "buffer_load_dwordx4 v26, s[52:55], s83 offen offset:0 lds\n\t"
        "buffer_load_dwordx4 v27, s[52:55], s83 offen offset:1024 lds\n\t"
        "buffer_load_dwordx4 v28, s[52:55], s83 offen offset:2048 lds\n\t"
        "buffer_load_dwordx4 v29, s[52:55], s83 offen offset:3072 lds\n\t"
        "buffer_load_dwordx4 v[64:67], v16, s[36:39], s78 offen offset:0 sc0 sc1\n\t"
        "buffer_load_dwordx4 v[68:71], v16, s[36:39], s78 offen offset:1024 sc0 sc1\n\t"
        "buffer_load_dwordx4 v[72:75], v16, s[36:39], s78 offen offset:2048 sc0 sc1\n\t"
        "buffer_load_dwordx4 v[76:79], v16, s[36:39], s78 offen offset:3072 sc0 sc1\n\t"
        "buffer_load_dwordx4 v[80:83], v30, s[36:39], s78 offen offset:0 sc0 sc1\n\t"
        "buffer_load_dwordx4 v[84:87], v30, s[36:39], s78 offen offset:1024 sc0 sc1\n\t"
        "buffer_load_dwordx4 v[88:91], v30, s[36:39], s78 offen offset:2048 sc0 sc1\n\t"
        "buffer_load_dwordx4 v[92:95], v30, s[36:39], s78 offen offset:3072 sc0 sc1\n\t"
        "buffer_load_dword v128, v17, s[36:39], s78 offen sc0 sc1\n\t"
        "s_waitcnt vmcnt(21)\n\t"
        "ds_read_b64_tr_b8 a[66:67], v180 offset:0\n\t"
        "ds_read_b64_tr_b8 a[70:71], v181 offset:0\n\t"
        "ds_read_b64_tr_b8 a[74:75], v182 offset:0\n\t"
        "ds_read_b64_tr_b8 a[78:79], v183 offset:0\n\t"
        "ds_read_b64_tr_b8 a[82:83], v184 offset:0\n\t"
        "ds_read_b64_tr_b8 a[86:87], v185 offset:0\n\t"
        "ds_read_b64_tr_b8 a[90:91], v186 offset:0\n\t"
        "ds_read_b64_tr_b8 a[94:95], v187 offset:0\n\t"
        "s_waitcnt lgkmcnt(0)\n\t"
        "s_waitcnt vmcnt(17)\n\t"
        "ds_read_b64_tr_b8 a[96:97], v180 offset:4096\n\t"
        "ds_read_b64_tr_b8 a[100:101], v181 offset:4096\n\t"
        "ds_read_b64_tr_b8 a[104:105], v182 offset:4096\n\t"
        "ds_read_b64_tr_b8 a[108:109], v183 offset:4096\n\t"
        "ds_read_b64_tr_b8 a[112:113], v184 offset:4096\n\t"
        "ds_read_b64_tr_b8 a[116:117], v185 offset:4096\n\t"
        "ds_read_b64_tr_b8 a[120:121], v186 offset:4096\n\t"
        "ds_read_b64_tr_b8 a[124:125], v187 offset:4096\n\t"
        "s_waitcnt lgkmcnt(0)\n\t"
        "s_waitcnt vmcnt(13)\n\t"
        "ds_read_b64_tr_b8 a[98:99], v180 offset:8192\n\t"
        "ds_read_b64_tr_b8 a[102:103], v181 offset:8192\n\t"
        "ds_read_b64_tr_b8 a[106:107], v182 offset:8192\n\t"
        "ds_read_b64_tr_b8 a[110:111], v183 offset:8192\n\t"
        "ds_read_b64_tr_b8 a[114:115], v184 offset:8192\n\t"
        "ds_read_b64_tr_b8 a[118:119], v185 offset:8192\n\t"
        "ds_read_b64_tr_b8 a[122:123], v186 offset:8192\n\t"
        "ds_read_b64_tr_b8 a[126:127], v187 offset:8192\n\t"
        "s_waitcnt lgkmcnt(0)\n\t"
        "s_waitcnt vmcnt(9)\n\t"
        "ds_read_b64_tr_b8 a[128:129], v180 offset:12288\n\t"
        "ds_read_b64_tr_b8 a[132:133], v181 offset:12288\n\t"
        "ds_read_b64_tr_b8 a[136:137], v182 offset:12288\n\t"
        "ds_read_b64_tr_b8 a[140:141], v183 offset:12288\n\t"
        "ds_read_b64_tr_b8 a[144:145], v184 offset:12288\n\t"
        "ds_read_b64_tr_b8 a[148:149], v185 offset:12288\n\t"
        "ds_read_b64_tr_b8 a[152:153], v186 offset:12288\n\t"
        "ds_read_b64_tr_b8 a[156:157], v187 offset:12288\n\t"
        "s_waitcnt lgkmcnt(0)\n\t"
        "s_branch LWAIT%=\n\t"
        "LW2%=:\n\t"
        "s_add_u32 s83, s67, 0x198000\n\t"
        "s_add_u32 m0, s64, 8192\n\t"
        "s_nop 0\n\t"
        "buffer_load_dwordx4 v26, s[52:55], s83 offen offset:0 lds\n\t"
        "buffer_load_dwordx4 v27, s[52:55], s83 offen offset:1024 lds\n\t"
        "buffer_load_dwordx4 v28, s[52:55], s83 offen offset:2048 lds\n\t"
        "buffer_load_dwordx4 v29, s[52:55], s83 offen offset:3072 lds\n\t"
        "s_add_u32 s83, s67, 0x0\n\t"
        "s_add_u32 m0, s64, 12288\n\t"
        "s_nop 0\n\t"
        "buffer_load_dwordx4 v26, s[52:55], s83 offen offset:0 lds\n\t"
        "buffer_load_dwordx4 v27, s[52:55], s83 offen offset:1024 lds\n\t"
        "buffer_load_dwordx4 v28, s[52:55], s83 offen offset:2048 lds\n\t"
        "buffer_load_dwordx4 v29, s[52:55], s83 offen offset:3072 lds\n\t"
        "s_waitcnt vmcnt(14)\n\t"
        "ds_read_b64_tr_b8 a[194:195], v180 offset:0\n\t"
        "ds_read_b64_tr_b8 a[198:199], v181 offset:0\n\t"
        "ds_read_b64_tr_b8 a[202:203], v182 offset:0\n\t"
        "ds_read_b64_tr_b8 a[206:207], v183 offset:0\n\t"
        "ds_read_b64_tr_b8 a[210:211], v184 offset:0\n\t"
        "ds_read_b64_tr_b8 a[214:215], v185 offset:0\n\t"
        "ds_read_b64_tr_b8 a[218:219], v186 offset:0\n\t"
        "ds_read_b64_tr_b8 a[222:223], v187 offset:0\n\t"
        "s_waitcnt lgkmcnt(0)\n\t"
        "s_add_u32 s83, s67, 0x108000\n\t"
        "s_add_u32 m0, s64, 0\n\t"
        "s_nop 0\n\t"
        "buffer_load_dwordx4 v26, s[52:55], s83 offen offset:0 lds\n\t"
        "buffer_load_dwordx4 v27, s[52:55], s83 offen offset:1024 lds\n\t"
        "buffer_load_dwordx4 v28, s[52:55], s83 offen offset:2048 lds\n\t"
        "buffer_load_dwordx4 v29, s[52:55], s83 offen offset:3072 lds\n\t"
        "s_waitcnt vmcnt(14)\n\t"
        "ds_read_b64_tr_b8 a[224:225], v180 offset:4096\n\t"
        "ds_read_b64_tr_b8 a[228:229], v181 offset:4096\n\t"
        "ds_read_b64_tr_b8 a[232:233], v182 offset:4096\n\t"
        "ds_read_b64_tr_b8 a[236:237], v183 offset:4096\n\t"
        "ds_read_b64_tr_b8 a[240:241], v184 offset:4096\n\t"
        "ds_read_b64_tr_b8 a[244:245], v185 offset:4096\n\t"
        "ds_read_b64_tr_b8 a[248:249], v186 offset:4096\n\t"
        "ds_read_b64_tr_b8 a[252:253], v187 offset:4096\n\t"
        "s_waitcnt lgkmcnt(0)\n\t"
        "s_add_u32 s83, s67, 0x8000\n\t"
        "s_add_u32 m0, s64, 4096\n\t"
        "s_nop 0\n\t"
        "buffer_load_dwordx4 v26, s[52:55], s83 offen offset:0 lds\n\t"
        "buffer_load_dwordx4 v27, s[52:55], s83 offen offset:1024 lds\n\t"
        "buffer_load_dwordx4 v28, s[52:55], s83 offen offset:2048 lds\n\t"
        "buffer_load_dwordx4 v29, s[52:55], s83 offen offset:3072 lds\n\t"
        "s_waitcnt vmcnt(12)\n\t"
        "ds_read_b64_tr_b8 a[226:227], v180 offset:8192\n\t"
        "ds_read_b64_tr_b8 a[230:231], v181 offset:8192\n\t"
        "ds_read_b64_tr_b8 a[234:235], v182 offset:8192\n\t"
        "ds_read_b64_tr_b8 a[238:239], v183 offset:8192\n\t"
        "ds_read_b64_tr_b8 a[242:243], v184 offset:8192\n\t"
        "ds_read_b64_tr_b8 a[246:247], v185 offset:8192\n\t"
        "ds_read_b64_tr_b8 a[250:251], v186 offset:8192\n\t"
        "ds_read_b64_tr_b8 a[254:255], v187 offset:8192\n\t"
        "s_waitcnt lgkmcnt(0)\n\t"
        "s_add_u32 s83, s67, 0x110000\n\t"
        "s_add_u32 m0, s64, 8192\n\t"
        "s_nop 0\n\t"
        "buffer_load_dwordx4 v26, s[52:55], s83 offen offset:0 lds\n\t"
        "buffer_load_dwordx4 v27, s[52:55], s83 offen offset:1024 lds\n\t"
        "buffer_load_dwordx4 v28, s[52:55], s83 offen offset:2048 lds\n\t"
        "buffer_load_dwordx4 v29, s[52:55], s83 offen offset:3072 lds\n\t"
        "s_waitcnt vmcnt(12)\n\t"
        "ds_read_b64_tr_b8 a[0:1], v180 offset:12288\n\t"
        "ds_read_b64_tr_b8 a[4:5], v181 offset:12288\n\t"
        "ds_read_b64_tr_b8 a[8:9], v182 offset:12288\n\t"
        "ds_read_b64_tr_b8 a[12:13], v183 offset:12288\n\t"
        "ds_read_b64_tr_b8 a[16:17], v184 offset:12288\n\t"
        "ds_read_b64_tr_b8 a[20:21], v185 offset:12288\n\t"
        "ds_read_b64_tr_b8 a[24:25], v186 offset:12288\n\t"
        "ds_read_b64_tr_b8 a[28:29], v187 offset:12288\n\t"
        "s_waitcnt lgkmcnt(0)\n\t"
        "s_add_u32 s83, s67, 0x10000\n\t"
        "s_add_u32 m0, s64, 12288\n\t"
        "s_nop 0\n\t"
        "buffer_load_dwordx4 v26, s[52:55], s83 offen offset:0 lds\n\t"
        "buffer_load_dwordx4 v27, s[52:55], s83 offen offset:1024 lds\n\t"
        "buffer_load_dwordx4 v28, s[52:55], s83 offen offset:2048 lds\n\t"
        "buffer_load_dwordx4 v29, s[52:55], s83 offen offset:3072 lds\n\t"
        "s_waitcnt vmcnt(12)\n\t"
        "ds_read_b64_tr_b8 a[130:131], v180 offset:0\n\t"
        "ds_read_b64_tr_b8 a[134:135], v181 offset:0\n\t"
        "ds_read_b64_tr_b8 a[138:139], v182 offset:0\n\t"
        "ds_read_b64_tr_b8 a[142:143], v183 offset:0\n\t"
        "ds_read_b64_tr_b8 a[146:147], v184 offset:0\n\t"
        "ds_read_b64_tr_b8 a[150:151], v185 offset:0\n\t"
        "ds_read_b64_tr_b8 a[154:155], v186 offset:0\n\t"
        "ds_read_b64_tr_b8 a[158:159], v187 offset:0\n\t"
        "s_waitcnt lgkmcnt(0)\n\t"
        "s_add_u32 s83, s67, 0x118000\n\t"
        "s_add_u32 m0, s64, 0\n\t"
        "s_nop 0\n\t"
        "buffer_load_dwordx4 v26, s[52:55], s83 offen offset:0 lds\n\t"
        "buffer_load_dwordx4 v27, s[52:55], s83 offen offset:1024 lds\n\t"
        "buffer_load_dwordx4 v28, s[52:55], s83 offen offset:2048 lds\n\t"
        "buffer_load_dwordx4 v29, s[52:55], s83 offen offset:3072 lds\n\t"
        "s_waitcnt vmcnt(12)\n\t"
        "ds_read_b64_tr_b8 a[2:3], v180 offset:4096\n\t"
        "ds_read_b64_tr_b8 a[6:7], v181 offset:4096\n\t"
        "ds_read_b64_tr_b8 a[10:11], v182 offset:4096\n\t"
        "ds_read_b64_tr_b8 a[14:15], v183 offset:4096\n\t"
        "ds_read_b64_tr_b8 a[18:19], v184 offset:4096\n\t"
        "ds_read_b64_tr_b8 a[22:23], v185 offset:4096\n\t"
        "ds_read_b64_tr_b8 a[26:27], v186 offset:4096\n\t"
        "ds_read_b64_tr_b8 a[30:31], v187 offset:4096\n\t"
        "s_waitcnt lgkmcnt(0)\n\t"
        "s_add_u32 s83, s67, 0x18000\n\t"
        "s_add_u32 m0, s64, 4096\n\t"
        "s_nop 0\n\t"
        "buffer_load_dwordx4 v26, s[52:55], s83 offen offset:0 lds\n\t"
        "buffer_load_dwordx4 v27, s[52:55], s83 offen offset:1024 lds\n\t"
        "buffer_load_dwordx4 v28, s[52:55], s83 offen offset:2048 lds\n\t"
        "buffer_load_dwordx4 v29, s[52:55], s83 offen offset:3072 lds\n\t"
        "s_waitcnt vmcnt(12)\n\t"
        "ds_read_b64_tr_b8 a[160:161], v180 offset:8192\n\t"
        "ds_read_b64_tr_b8 a[164:165], v181 offset:8192\n\t"
        "ds_read_b64_tr_b8 a[168:169], v182 offset:8192\n\t"
        "ds_read_b64_tr_b8 a[172:173], v183 offset:8192\n\t"
        "ds_read_b64_tr_b8 a[176:177], v184 offset:8192\n\t"
        "ds_read_b64_tr_b8 a[180:181], v185 offset:8192\n\t"
        "ds_read_b64_tr_b8 a[184:185], v186 offset:8192\n\t"
        "ds_read_b64_tr_b8 a[188:189], v187 offset:8192\n\t"
        "s_waitcnt lgkmcnt(0)\n\t"
        "s_add_u32 s83, s67, 0x180000\n\t"
        "s_add_u32 m0, s64, 8192\n\t"
        "s_nop 0\n\t"
        "buffer_load_dwordx4 v26, s[52:55], s83 offen offset:0 lds\n\t"
        "buffer_load_dwordx4 v27, s[52:55], s83 offen offset:1024 lds\n\t"
        "buffer_load_dwordx4 v28, s[52:55], s83 offen offset:2048 lds\n\t"
        "buffer_load_dwordx4 v29, s[52:55], s83 offen offset:3072 lds\n\t"
        "s_waitcnt vmcnt(12)\n\t"
        "ds_read_b64_tr_b8 a[32:33], v180 offset:12288\n\t"
        "ds_read_b64_tr_b8 a[36:37], v181 offset:12288\n\t"
        "ds_read_b64_tr_b8 a[40:41], v182 offset:12288\n\t"
        "ds_read_b64_tr_b8 a[44:45], v183 offset:12288\n\t"
        "ds_read_b64_tr_b8 a[48:49], v184 offset:12288\n\t"
        "ds_read_b64_tr_b8 a[52:53], v185 offset:12288\n\t"
        "ds_read_b64_tr_b8 a[56:57], v186 offset:12288\n\t"
        "ds_read_b64_tr_b8 a[60:61], v187 offset:12288\n\t"
        "s_waitcnt lgkmcnt(0)\n\t"
        "s_add_u32 s83, s67, 0x80000\n\t"
        "s_add_u32 m0, s64, 12288\n\t"
        "s_nop 0\n\t"
        "buffer_load_dwordx4 v26, s[52:55], s83 offen offset:0 lds\n\t"
        "buffer_load_dwordx4 v27, s[52:55], s83 offen offset:1024 lds\n\t"
        "buffer_load_dwordx4 v28, s[52:55], s83 offen offset:2048 lds\n\t"
        "buffer_load_dwordx4 v29, s[52:55], s83 offen offset:3072 lds\n\t"
        "s_waitcnt vmcnt(12)\n\t"
        "ds_read_b64_tr_b8 a[162:163], v180 offset:0\n\t"
        "ds_read_b64_tr_b8 a[166:167], v181 offset:0\n\t"
        "ds_read_b64_tr_b8 a[170:171], v182 offset:0\n\t"
        "ds_read_b64_tr_b8 a[174:175], v183 offset:0\n\t"
        "ds_read_b64_tr_b8 a[178:179], v184 offset:0\n\t"
        "ds_read_b64_tr_b8 a[182:183], v185 offset:0\n\t"
        "ds_read_b64_tr_b8 a[186:187], v186 offset:0\n\t"
        "ds_read_b64_tr_b8 a[190:191], v187 offset:0\n\t"
        "s_waitcnt lgkmcnt(0)\n\t"
        "s_add_u32 s83, s67, 0x88000\n\t"
        "s_add_u32 m0, s64, 0\n\t"
        "s_nop 0\n\t"
        "buffer_load_dwordx4 v26, s[52:55], s83 offen offset:0 lds\n\t"
        "buffer_load_dwordx4 v27, s[52:55], s83 offen offset:1024 lds\n\t"
        "buffer_load_dwordx4 v28, s[52:55], s83 offen offset:2048 lds\n\t"
        "buffer_load_dwordx4 v29, s[52:55], s83 offen offset:3072 lds\n\t"
        "s_waitcnt vmcnt(12)\n\t"
        "ds_read_b64_tr_b8 a[34:35], v180 offset:4096\n\t"
        "ds_read_b64_tr_b8 a[38:39], v181 offset:4096\n\t"
        "ds_read_b64_tr_b8 a[42:43], v182 offset:4096\n\t"
        "ds_read_b64_tr_b8 a[46:47], v183 offset:4096\n\t"
        "ds_read_b64_tr_b8 a[50:51], v184 offset:4096\n\t"
        "ds_read_b64_tr_b8 a[54:55], v185 offset:4096\n\t"
        "ds_read_b64_tr_b8 a[58:59], v186 offset:4096\n\t"
        "ds_read_b64_tr_b8 a[62:63], v187 offset:4096\n\t"
        "s_waitcnt lgkmcnt(0)\n\t"
        "s_add_u32 s83, s67, 0x90000\n\t"
        "s_add_u32 m0, s64, 4096\n\t"
        "s_nop 0\n\t"
        "buffer_load_dwordx4 v26, s[52:55], s83 offen offset:0 lds\n\t"
        "buffer_load_dwordx4 v27, s[52:55], s83 offen offset:1024 lds\n\t"
        "buffer_load_dwordx4 v28, s[52:55], s83 offen offset:2048 lds\n\t"
        "buffer_load_dwordx4 v29, s[52:55], s83 offen offset:3072 lds\n\t"
        "s_waitcnt vmcnt(12)\n\t"
        "ds_read_b64_tr_b8 a[192:193], v180 offset:8192\n\t"
        "ds_read_b64_tr_b8 a[196:197], v181 offset:8192\n\t"
        "ds_read_b64_tr_b8 a[200:201], v182 offset:8192\n\t"
        "ds_read_b64_tr_b8 a[204:205], v183 offset:8192\n\t"
        "ds_read_b64_tr_b8 a[208:209], v184 offset:8192\n\t"
        "ds_read_b64_tr_b8 a[212:213], v185 offset:8192\n\t"
        "ds_read_b64_tr_b8 a[216:217], v186 offset:8192\n\t"
        "ds_read_b64_tr_b8 a[220:221], v187 offset:8192\n\t"
        "s_waitcnt lgkmcnt(0)\n\t"
        "s_add_u32 s83, s67, 0x98000\n\t"
        "s_add_u32 m0, s64, 8192\n\t"
        "s_nop 0\n\t"
        "buffer_load_dwordx4 v26, s[52:55], s83 offen offset:0 lds\n\t"
        "buffer_load_dwordx4 v27, s[52:55], s83 offen offset:1024 lds\n\t"
        "buffer_load_dwordx4 v28, s[52:55], s83 offen offset:2048 lds\n\t"
        "buffer_load_dwordx4 v29, s[52:55], s83 offen offset:3072 lds\n\t"
        "s_waitcnt vmcnt(12)\n\t"
        "ds_read_b64_tr_b8 a[64:65], v180 offset:12288\n\t"
        "ds_read_b64_tr_b8 a[68:69], v181 offset:12288\n\t"
        "ds_read_b64_tr_b8 a[72:73], v182 offset:12288\n\t"
        "ds_read_b64_tr_b8 a[76:77], v183 offset:12288\n\t"
        "ds_read_b64_tr_b8 a[80:81], v184 offset:12288\n\t"
        "ds_read_b64_tr_b8 a[84:85], v185 offset:12288\n\t"
        "ds_read_b64_tr_b8 a[88:89], v186 offset:12288\n\t"
        "ds_read_b64_tr_b8 a[92:93], v187 offset:12288\n\t"
        "s_waitcnt lgkmcnt(0)\n\t"
        "s_add_u32 s83, s67, 0x100000\n\t"
        "s_add_u32 m0, s64, 12288\n\t"
        "s_nop 0\n\t"
        "buffer_load_dwordx4 v26, s[52:55], s83 offen offset:0 lds\n\t"
        "buffer_load_dwordx4 v27, s[52:55], s83 offen offset:1024 lds\n\t"
        "buffer_load_dwordx4 v28, s[52:55], s83 offen offset:2048 lds\n\t"
        "buffer_load_dwordx4 v29, s[52:55], s83 offen offset:3072 lds\n\t"
        "buffer_load_dwordx4 v[64:67], v16, s[36:39], s78 offen offset:0 sc0 sc1\n\t"
        "buffer_load_dwordx4 v[68:71], v16, s[36:39], s78 offen offset:1024 sc0 sc1\n\t"
        "buffer_load_dwordx4 v[72:75], v16, s[36:39], s78 offen offset:2048 sc0 sc1\n\t"
        "buffer_load_dwordx4 v[76:79], v16, s[36:39], s78 offen offset:3072 sc0 sc1\n\t"
        "buffer_load_dwordx4 v[80:83], v30, s[36:39], s78 offen offset:0 sc0 sc1\n\t"
        "buffer_load_dwordx4 v[84:87], v30, s[36:39], s78 offen offset:1024 sc0 sc1\n\t"
        "buffer_load_dwordx4 v[88:91], v30, s[36:39], s78 offen offset:2048 sc0 sc1\n\t"
        "buffer_load_dwordx4 v[92:95], v30, s[36:39], s78 offen offset:3072 sc0 sc1\n\t"
        "buffer_load_dword v128, v17, s[36:39], s78 offen sc0 sc1\n\t"
        "s_waitcnt vmcnt(21)\n\t"
        "ds_read_b64_tr_b8 a[66:67], v180 offset:0\n\t"
        "ds_read_b64_tr_b8 a[70:71], v181 offset:0\n\t"
        "ds_read_b64_tr_b8 a[74:75], v182 offset:0\n\t"
        "ds_read_b64_tr_b8 a[78:79], v183 offset:0\n\t"
        "ds_read_b64_tr_b8 a[82:83], v184 offset:0\n\t"
        "ds_read_b64_tr_b8 a[86:87], v185 offset:0\n\t"
        "ds_read_b64_tr_b8 a[90:91], v186 offset:0\n\t"
        "ds_read_b64_tr_b8 a[94:95], v187 offset:0\n\t"
        "s_waitcnt lgkmcnt(0)\n\t"
        "s_waitcnt vmcnt(17)\n\t"
        "ds_read_b64_tr_b8 a[96:97], v180 offset:4096\n\t"
        "ds_read_b64_tr_b8 a[100:101], v181 offset:4096\n\t"
        "ds_read_b64_tr_b8 a[104:105], v182 offset:4096\n\t"
        "ds_read_b64_tr_b8 a[108:109], v183 offset:4096\n\t"
        "ds_read_b64_tr_b8 a[112:113], v184 offset:4096\n\t"
        "ds_read_b64_tr_b8 a[116:117], v185 offset:4096\n\t"
        "ds_read_b64_tr_b8 a[120:121], v186 offset:4096\n\t"
        "ds_read_b64_tr_b8 a[124:125], v187 offset:4096\n\t"
        "s_waitcnt lgkmcnt(0)\n\t"
        "s_waitcnt vmcnt(13)\n\t"
        "ds_read_b64_tr_b8 a[98:99], v180 offset:8192\n\t"
        "ds_read_b64_tr_b8 a[102:103], v181 offset:8192\n\t"
        "ds_read_b64_tr_b8 a[106:107], v182 offset:8192\n\t"
        "ds_read_b64_tr_b8 a[110:111], v183 offset:8192\n\t"
        "ds_read_b64_tr_b8 a[114:115], v184 offset:8192\n\t"
        "ds_read_b64_tr_b8 a[118:119], v185 offset:8192\n\t"
        "ds_read_b64_tr_b8 a[122:123], v186 offset:8192\n\t"
        "ds_read_b64_tr_b8 a[126:127], v187 offset:8192\n\t"
        "s_waitcnt lgkmcnt(0)\n\t"
        "s_waitcnt vmcnt(9)\n\t"
        "ds_read_b64_tr_b8 a[128:129], v180 offset:12288\n\t"
        "ds_read_b64_tr_b8 a[132:133], v181 offset:12288\n\t"
        "ds_read_b64_tr_b8 a[136:137], v182 offset:12288\n\t"
        "ds_read_b64_tr_b8 a[140:141], v183 offset:12288\n\t"
        "ds_read_b64_tr_b8 a[144:145], v184 offset:12288\n\t"
        "ds_read_b64_tr_b8 a[148:149], v185 offset:12288\n\t"
        "ds_read_b64_tr_b8 a[152:153], v186 offset:12288\n\t"
        "ds_read_b64_tr_b8 a[156:157], v187 offset:12288\n\t"
        "s_waitcnt lgkmcnt(0)\n\t"
        "s_branch LWAIT%=\n\t"
        "LW4%=:\n\t"
        "s_add_u32 s83, s67, 0x98000\n\t"
        "s_add_u32 m0, s64, 8192\n\t"
        "s_nop 0\n\t"
        "buffer_load_dwordx4 v26, s[52:55], s83 offen offset:0 lds\n\t"
        "buffer_load_dwordx4 v27, s[52:55], s83 offen offset:1024 lds\n\t"
        "buffer_load_dwordx4 v28, s[52:55], s83 offen offset:2048 lds\n\t"
        "buffer_load_dwordx4 v29, s[52:55], s83 offen offset:3072 lds\n\t"
        "s_add_u32 s83, s67, 0x100000\n\t"
        "s_add_u32 m0, s64, 12288\n\t"
        "s_nop 0\n\t"
        "buffer_load_dwordx4 v26, s[52:55], s83 offen offset:0 lds\n\t"
        "buffer_load_dwordx4 v27, s[52:55], s83 offen offset:1024 lds\n\t"
        "buffer_load_dwordx4 v28, s[52:55], s83 offen offset:2048 lds\n\t"
        "buffer_load_dwordx4 v29, s[52:55], s83 offen offset:3072 lds\n\t"
        "s_waitcnt vmcnt(14)\n\t"
        "ds_read_b64_tr_b8 a[66:67], v180 offset:0\n\t"
        "ds_read_b64_tr_b8 a[70:71], v181 offset:0\n\t"
        "ds_read_b64_tr_b8 a[74:75], v182 offset:0\n\t"
        "ds_read_b64_tr_b8 a[78:79], v183 offset:0\n\t"
        "ds_read_b64_tr_b8 a[82:83], v184 offset:0\n\t"
        "ds_read_b64_tr_b8 a[86:87], v185 offset:0\n\t"
        "ds_read_b64_tr_b8 a[90:91], v186 offset:0\n\t"
        "ds_read_b64_tr_b8 a[94:95], v187 offset:0\n\t"
        "s_waitcnt lgkmcnt(0)\n\t"
        "s_add_u32 s83, s67, 0x0\n\t"
        "s_add_u32 m0, s64, 0\n\t"
        "s_nop 0\n\t"
        "buffer_load_dwordx4 v26, s[52:55], s83 offen offset:0 lds\n\t"
        "buffer_load_dwordx4 v27, s[52:55], s83 offen offset:1024 lds\n\t"
        "buffer_load_dwordx4 v28, s[52:55], s83 offen offset:2048 lds\n\t"
        "buffer_load_dwordx4 v29, s[52:55], s83 offen offset:3072 lds\n\t"
        "s_waitcnt vmcnt(14)\n\t"
        "ds_read_b64_tr_b8 a[96:97], v180 offset:4096\n\t"
        "ds_read_b64_tr_b8 a[100:101], v181 offset:4096\n\t"
        "ds_read_b64_tr_b8 a[104:105], v182 offset:4096\n\t"
        "ds_read_b64_tr_b8 a[108:109], v183 offset:4096\n\t"
        "ds_read_b64_tr_b8 a[112:113], v184 offset:4096\n\t"
        "ds_read_b64_tr_b8 a[116:117], v185 offset:4096\n\t"
        "ds_read_b64_tr_b8 a[120:121], v186 offset:4096\n\t"
        "ds_read_b64_tr_b8 a[124:125], v187 offset:4096\n\t"
        "s_waitcnt lgkmcnt(0)\n\t"
        "s_add_u32 s83, s67, 0x108000\n\t"
        "s_add_u32 m0, s64, 4096\n\t"
        "s_nop 0\n\t"
        "buffer_load_dwordx4 v26, s[52:55], s83 offen offset:0 lds\n\t"
        "buffer_load_dwordx4 v27, s[52:55], s83 offen offset:1024 lds\n\t"
        "buffer_load_dwordx4 v28, s[52:55], s83 offen offset:2048 lds\n\t"
        "buffer_load_dwordx4 v29, s[52:55], s83 offen offset:3072 lds\n\t"
        "s_waitcnt vmcnt(12)\n\t"
        "ds_read_b64_tr_b8 a[98:99], v180 offset:8192\n\t"
        "ds_read_b64_tr_b8 a[102:103], v181 offset:8192\n\t"
        "ds_read_b64_tr_b8 a[106:107], v182 offset:8192\n\t"
        "ds_read_b64_tr_b8 a[110:111], v183 offset:8192\n\t"
        "ds_read_b64_tr_b8 a[114:115], v184 offset:8192\n\t"
        "ds_read_b64_tr_b8 a[118:119], v185 offset:8192\n\t"
        "ds_read_b64_tr_b8 a[122:123], v186 offset:8192\n\t"
        "ds_read_b64_tr_b8 a[126:127], v187 offset:8192\n\t"
        "s_waitcnt lgkmcnt(0)\n\t"
        "s_add_u32 s83, s67, 0x188000\n\t"
        "s_add_u32 m0, s64, 8192\n\t"
        "s_nop 0\n\t"
        "buffer_load_dwordx4 v26, s[52:55], s83 offen offset:0 lds\n\t"
        "buffer_load_dwordx4 v27, s[52:55], s83 offen offset:1024 lds\n\t"
        "buffer_load_dwordx4 v28, s[52:55], s83 offen offset:2048 lds\n\t"
        "buffer_load_dwordx4 v29, s[52:55], s83 offen offset:3072 lds\n\t"
        "s_waitcnt vmcnt(12)\n\t"
        "ds_read_b64_tr_b8 a[128:129], v180 offset:12288\n\t"
        "ds_read_b64_tr_b8 a[132:133], v181 offset:12288\n\t"
        "ds_read_b64_tr_b8 a[136:137], v182 offset:12288\n\t"
        "ds_read_b64_tr_b8 a[140:141], v183 offset:12288\n\t"
        "ds_read_b64_tr_b8 a[144:145], v184 offset:12288\n\t"
        "ds_read_b64_tr_b8 a[148:149], v185 offset:12288\n\t"
        "ds_read_b64_tr_b8 a[152:153], v186 offset:12288\n\t"
        "ds_read_b64_tr_b8 a[156:157], v187 offset:12288\n\t"
        "s_waitcnt lgkmcnt(0)\n\t"
        "s_add_u32 s83, s67, 0x8000\n\t"
        "s_add_u32 m0, s64, 12288\n\t"
        "s_nop 0\n\t"
        "buffer_load_dwordx4 v26, s[52:55], s83 offen offset:0 lds\n\t"
        "buffer_load_dwordx4 v27, s[52:55], s83 offen offset:1024 lds\n\t"
        "buffer_load_dwordx4 v28, s[52:55], s83 offen offset:2048 lds\n\t"
        "buffer_load_dwordx4 v29, s[52:55], s83 offen offset:3072 lds\n\t"
        "s_waitcnt vmcnt(12)\n\t"
        "ds_read_b64_tr_b8 a[0:1], v180 offset:0\n\t"
        "ds_read_b64_tr_b8 a[4:5], v181 offset:0\n\t"
        "ds_read_b64_tr_b8 a[8:9], v182 offset:0\n\t"
        "ds_read_b64_tr_b8 a[12:13], v183 offset:0\n\t"
        "ds_read_b64_tr_b8 a[16:17], v184 offset:0\n\t"
        "ds_read_b64_tr_b8 a[20:21], v185 offset:0\n\t"
        "ds_read_b64_tr_b8 a[24:25], v186 offset:0\n\t"
        "ds_read_b64_tr_b8 a[28:29], v187 offset:0\n\t"
        "s_waitcnt lgkmcnt(0)\n\t"
        "s_add_u32 s83, s67, 0x110000\n\t"
        "s_add_u32 m0, s64, 0\n\t"
        "s_nop 0\n\t"
        "buffer_load_dwordx4 v26, s[52:55], s83 offen offset:0 lds\n\t"
        "buffer_load_dwordx4 v27, s[52:55], s83 offen offset:1024 lds\n\t"
        "buffer_load_dwordx4 v28, s[52:55], s83 offen offset:2048 lds\n\t"
        "buffer_load_dwordx4 v29, s[52:55], s83 offen offset:3072 lds\n\t"
        "s_waitcnt vmcnt(12)\n\t"
        "ds_read_b64_tr_b8 a[130:131], v180 offset:4096\n\t"
        "ds_read_b64_tr_b8 a[134:135], v181 offset:4096\n\t"
        "ds_read_b64_tr_b8 a[138:139], v182 offset:4096\n\t"
        "ds_read_b64_tr_b8 a[142:143], v183 offset:4096\n\t"
        "ds_read_b64_tr_b8 a[146:147], v184 offset:4096\n\t"
        "ds_read_b64_tr_b8 a[150:151], v185 offset:4096\n\t"
        "ds_read_b64_tr_b8 a[154:155], v186 offset:4096\n\t"
        "ds_read_b64_tr_b8 a[158:159], v187 offset:4096\n\t"
        "s_waitcnt lgkmcnt(0)\n\t"
        "s_add_u32 s83, s67, 0x190000\n\t"
        "s_add_u32 m0, s64, 4096\n\t"
        "s_nop 0\n\t"
        "buffer_load_dwordx4 v26, s[52:55], s83 offen offset:0 lds\n\t"
        "buffer_load_dwordx4 v27, s[52:55], s83 offen offset:1024 lds\n\t"
        "buffer_load_dwordx4 v28, s[52:55], s83 offen offset:2048 lds\n\t"
        "buffer_load_dwordx4 v29, s[52:55], s83 offen offset:3072 lds\n\t"
        "s_waitcnt vmcnt(12)\n\t"
        "ds_read_b64_tr_b8 a[194:195], v180 offset:8192\n\t"
        "ds_read_b64_tr_b8 a[198:199], v181 offset:8192\n\t"
        "ds_read_b64_tr_b8 a[202:203], v182 offset:8192\n\t"
        "ds_read_b64_tr_b8 a[206:207], v183 offset:8192\n\t"
        "ds_read_b64_tr_b8 a[210:211], v184 offset:8192\n\t"
        "ds_read_b64_tr_b8 a[214:215], v185 offset:8192\n\t"
        "ds_read_b64_tr_b8 a[218:219], v186 offset:8192\n\t"
        "ds_read_b64_tr_b8 a[222:223], v187 offset:8192\n\t"
        "s_waitcnt lgkmcnt(0)\n\t"
        "s_add_u32 s83, s67, 0x10000\n\t"
        "s_add_u32 m0, s64, 8192\n\t"
        "s_nop 0\n\t"
        "buffer_load_dwordx4 v26, s[52:55], s83 offen offset:0 lds\n\t"
        "buffer_load_dwordx4 v27, s[52:55], s83 offen offset:1024 lds\n\t"
        "buffer_load_dwordx4 v28, s[52:55], s83 offen offset:2048 lds\n\t"
        "buffer_load_dwordx4 v29, s[52:55], s83 offen offset:3072 lds\n\t"
        "s_waitcnt vmcnt(12)\n\t"
        "ds_read_b64_tr_b8 a[2:3], v180 offset:12288\n\t"
        "ds_read_b64_tr_b8 a[6:7], v181 offset:12288\n\t"
        "ds_read_b64_tr_b8 a[10:11], v182 offset:12288\n\t"
        "ds_read_b64_tr_b8 a[14:15], v183 offset:12288\n\t"
        "ds_read_b64_tr_b8 a[18:19], v184 offset:12288\n\t"
        "ds_read_b64_tr_b8 a[22:23], v185 offset:12288\n\t"
        "ds_read_b64_tr_b8 a[26:27], v186 offset:12288\n\t"
        "ds_read_b64_tr_b8 a[30:31], v187 offset:12288\n\t"
        "s_waitcnt lgkmcnt(0)\n\t"
        "s_add_u32 s83, s67, 0x118000\n\t"
        "s_add_u32 m0, s64, 12288\n\t"
        "s_nop 0\n\t"
        "buffer_load_dwordx4 v26, s[52:55], s83 offen offset:0 lds\n\t"
        "buffer_load_dwordx4 v27, s[52:55], s83 offen offset:1024 lds\n\t"
        "buffer_load_dwordx4 v28, s[52:55], s83 offen offset:2048 lds\n\t"
        "buffer_load_dwordx4 v29, s[52:55], s83 offen offset:3072 lds\n\t"
        "s_waitcnt vmcnt(12)\n\t"
        "ds_read_b64_tr_b8 a[160:161], v180 offset:0\n\t"
        "ds_read_b64_tr_b8 a[164:165], v181 offset:0\n\t"
        "ds_read_b64_tr_b8 a[168:169], v182 offset:0\n\t"
        "ds_read_b64_tr_b8 a[172:173], v183 offset:0\n\t"
        "ds_read_b64_tr_b8 a[176:177], v184 offset:0\n\t"
        "ds_read_b64_tr_b8 a[180:181], v185 offset:0\n\t"
        "ds_read_b64_tr_b8 a[184:185], v186 offset:0\n\t"
        "ds_read_b64_tr_b8 a[188:189], v187 offset:0\n\t"
        "s_waitcnt lgkmcnt(0)\n\t"
        "s_add_u32 s83, s67, 0x198000\n\t"
        "s_add_u32 m0, s64, 0\n\t"
        "s_nop 0\n\t"
        "buffer_load_dwordx4 v26, s[52:55], s83 offen offset:0 lds\n\t"
        "buffer_load_dwordx4 v27, s[52:55], s83 offen offset:1024 lds\n\t"
        "buffer_load_dwordx4 v28, s[52:55], s83 offen offset:2048 lds\n\t"
        "buffer_load_dwordx4 v29, s[52:55], s83 offen offset:3072 lds\n\t"
        "s_waitcnt vmcnt(12)\n\t"
        "ds_read_b64_tr_b8 a[224:225], v180 offset:4096\n\t"
        "ds_read_b64_tr_b8 a[228:229], v181 offset:4096\n\t"
        "ds_read_b64_tr_b8 a[232:233], v182 offset:4096\n\t"
        "ds_read_b64_tr_b8 a[236:237], v183 offset:4096\n\t"
        "ds_read_b64_tr_b8 a[240:241], v184 offset:4096\n\t"
        "ds_read_b64_tr_b8 a[244:245], v185 offset:4096\n\t"
        "ds_read_b64_tr_b8 a[248:249], v186 offset:4096\n\t"
        "ds_read_b64_tr_b8 a[252:253], v187 offset:4096\n\t"
        "s_waitcnt lgkmcnt(0)\n\t"
        "s_add_u32 s83, s67, 0x18000\n\t"
        "s_add_u32 m0, s64, 4096\n\t"
        "s_nop 0\n\t"
        "buffer_load_dwordx4 v26, s[52:55], s83 offen offset:0 lds\n\t"
        "buffer_load_dwordx4 v27, s[52:55], s83 offen offset:1024 lds\n\t"
        "buffer_load_dwordx4 v28, s[52:55], s83 offen offset:2048 lds\n\t"
        "buffer_load_dwordx4 v29, s[52:55], s83 offen offset:3072 lds\n\t"
        "s_waitcnt vmcnt(12)\n\t"
        "ds_read_b64_tr_b8 a[32:33], v180 offset:8192\n\t"
        "ds_read_b64_tr_b8 a[36:37], v181 offset:8192\n\t"
        "ds_read_b64_tr_b8 a[40:41], v182 offset:8192\n\t"
        "ds_read_b64_tr_b8 a[44:45], v183 offset:8192\n\t"
        "ds_read_b64_tr_b8 a[48:49], v184 offset:8192\n\t"
        "ds_read_b64_tr_b8 a[52:53], v185 offset:8192\n\t"
        "ds_read_b64_tr_b8 a[56:57], v186 offset:8192\n\t"
        "ds_read_b64_tr_b8 a[60:61], v187 offset:8192\n\t"
        "s_waitcnt lgkmcnt(0)\n\t"
        "s_add_u32 s83, s67, 0x180000\n\t"
        "s_add_u32 m0, s64, 8192\n\t"
        "s_nop 0\n\t"
        "buffer_load_dwordx4 v26, s[52:55], s83 offen offset:0 lds\n\t"
        "buffer_load_dwordx4 v27, s[52:55], s83 offen offset:1024 lds\n\t"
        "buffer_load_dwordx4 v28, s[52:55], s83 offen offset:2048 lds\n\t"
        "buffer_load_dwordx4 v29, s[52:55], s83 offen offset:3072 lds\n\t"
        "s_waitcnt vmcnt(12)\n\t"
        "ds_read_b64_tr_b8 a[162:163], v180 offset:12288\n\t"
        "ds_read_b64_tr_b8 a[166:167], v181 offset:12288\n\t"
        "ds_read_b64_tr_b8 a[170:171], v182 offset:12288\n\t"
        "ds_read_b64_tr_b8 a[174:175], v183 offset:12288\n\t"
        "ds_read_b64_tr_b8 a[178:179], v184 offset:12288\n\t"
        "ds_read_b64_tr_b8 a[182:183], v185 offset:12288\n\t"
        "ds_read_b64_tr_b8 a[186:187], v186 offset:12288\n\t"
        "ds_read_b64_tr_b8 a[190:191], v187 offset:12288\n\t"
        "s_waitcnt lgkmcnt(0)\n\t"
        "s_add_u32 s83, s67, 0x80000\n\t"
        "s_add_u32 m0, s64, 12288\n\t"
        "s_nop 0\n\t"
        "buffer_load_dwordx4 v26, s[52:55], s83 offen offset:0 lds\n\t"
        "buffer_load_dwordx4 v27, s[52:55], s83 offen offset:1024 lds\n\t"
        "buffer_load_dwordx4 v28, s[52:55], s83 offen offset:2048 lds\n\t"
        "buffer_load_dwordx4 v29, s[52:55], s83 offen offset:3072 lds\n\t"
        "buffer_load_dwordx4 v[64:67], v16, s[36:39], s78 offen offset:0 sc0 sc1\n\t"
        "buffer_load_dwordx4 v[68:71], v16, s[36:39], s78 offen offset:1024 sc0 sc1\n\t"
        "buffer_load_dwordx4 v[72:75], v16, s[36:39], s78 offen offset:2048 sc0 sc1\n\t"
        "buffer_load_dwordx4 v[76:79], v16, s[36:39], s78 offen offset:3072 sc0 sc1\n\t"
        "buffer_load_dwordx4 v[80:83], v30, s[36:39], s78 offen offset:0 sc0 sc1\n\t"
        "buffer_load_dwordx4 v[84:87], v30, s[36:39], s78 offen offset:1024 sc0 sc1\n\t"
        "buffer_load_dwordx4 v[88:91], v30, s[36:39], s78 offen offset:2048 sc0 sc1\n\t"
        "buffer_load_dwordx4 v[92:95], v30, s[36:39], s78 offen offset:3072 sc0 sc1\n\t"
        "buffer_load_dword v128, v17, s[36:39], s78 offen sc0 sc1\n\t"
        "s_waitcnt vmcnt(21)\n\t"
        "ds_read_b64_tr_b8 a[226:227], v180 offset:0\n\t"
        "ds_read_b64_tr_b8 a[230:231], v181 offset:0\n\t"
        "ds_read_b64_tr_b8 a[234:235], v182 offset:0\n\t"
        "ds_read_b64_tr_b8 a[238:239], v183 offset:0\n\t"
        "ds_read_b64_tr_b8 a[242:243], v184 offset:0\n\t"
        "ds_read_b64_tr_b8 a[246:247], v185 offset:0\n\t"
        "ds_read_b64_tr_b8 a[250:251], v186 offset:0\n\t"
        "ds_read_b64_tr_b8 a[254:255], v187 offset:0\n\t"
        "s_waitcnt lgkmcnt(0)\n\t"
        "s_waitcnt vmcnt(17)\n\t"
        "ds_read_b64_tr_b8 a[34:35], v180 offset:4096\n\t"
        "ds_read_b64_tr_b8 a[38:39], v181 offset:4096\n\t"
        "ds_read_b64_tr_b8 a[42:43], v182 offset:4096\n\t"
        "ds_read_b64_tr_b8 a[46:47], v183 offset:4096\n\t"
        "ds_read_b64_tr_b8 a[50:51], v184 offset:4096\n\t"
        "ds_read_b64_tr_b8 a[54:55], v185 offset:4096\n\t"
        "ds_read_b64_tr_b8 a[58:59], v186 offset:4096\n\t"
        "ds_read_b64_tr_b8 a[62:63], v187 offset:4096\n\t"
        "s_waitcnt lgkmcnt(0)\n\t"
        "s_waitcnt vmcnt(13)\n\t"
        "ds_read_b64_tr_b8 a[192:193], v180 offset:8192\n\t"
        "ds_read_b64_tr_b8 a[196:197], v181 offset:8192\n\t"
        "ds_read_b64_tr_b8 a[200:201], v182 offset:8192\n\t"
        "ds_read_b64_tr_b8 a[204:205], v183 offset:8192\n\t"
        "ds_read_b64_tr_b8 a[208:209], v184 offset:8192\n\t"
        "ds_read_b64_tr_b8 a[212:213], v185 offset:8192\n\t"
        "ds_read_b64_tr_b8 a[216:217], v186 offset:8192\n\t"
        "ds_read_b64_tr_b8 a[220:221], v187 offset:8192\n\t"
        "s_waitcnt lgkmcnt(0)\n\t"
        "s_waitcnt vmcnt(9)\n\t"
        "ds_read_b64_tr_b8 a[64:65], v180 offset:12288\n\t"
        "ds_read_b64_tr_b8 a[68:69], v181 offset:12288\n\t"
        "ds_read_b64_tr_b8 a[72:73], v182 offset:12288\n\t"
        "ds_read_b64_tr_b8 a[76:77], v183 offset:12288\n\t"
        "ds_read_b64_tr_b8 a[80:81], v184 offset:12288\n\t"
        "ds_read_b64_tr_b8 a[84:85], v185 offset:12288\n\t"
        "ds_read_b64_tr_b8 a[88:89], v186 offset:12288\n\t"
        "ds_read_b64_tr_b8 a[92:93], v187 offset:12288\n\t"
        "s_waitcnt lgkmcnt(0)\n\t"
        "s_branch LWAIT%=\n\t"
        "LW3%=:\n\t"
        "s_cmp_lt_u32 s66, 4096\n\t"
        "s_cbranch_scc1 LHP2%=\n\t"
        "s_add_u32 m0, s64, 0\n\t"
        "s_add_u32 s83, s67, 0x0\n\t"
        "buffer_load_dwordx4 v26, s[52:55], s83 offen offset:0 lds\n\t"
        "buffer_load_dwordx4 v27, s[52:55], s83 offen offset:1024 lds\n\t"
        "buffer_load_dwordx4 v28, s[52:55], s83 offen offset:2048 lds\n\t"
        "buffer_load_dwordx4 v29, s[52:55], s83 offen offset:3072 lds\n\t"
        "s_add_u32 m0, s64, 4096\n\t"
        "s_add_u32 s83, s67, 0x108000\n\t"
        "buffer_load_dwordx4 v26, s[52:55], s83 offen offset:0 lds\n\t"
        "buffer_load_dwordx4 v27, s[52:55], s83 offen offset:1024 lds\n\t"
        "buffer_load_dwordx4 v28, s[52:55], s83 offen offset:2048 lds\n\t"
        "buffer_load_dwordx4 v29, s[52:55], s83 offen offset:3072 lds\n\t"
        "LHP2%=:\n\t"
        "s_memrealtime s[92:93]\n\t"
        "s_waitcnt lgkmcnt(0)\n\t"
        "s_mov_b32 s23, 2\n\t"
        "s_memrealtime s[92:93]\n\t"
        "s_waitcnt lgkmcnt(0)\n\t"
        "s_sub_i32 s84, s74, s92\n\t"
        "s_cmp_gt_i32 s84, s66\n\t"
        "s_cbranch_scc0 LXQ2_%=\n\t"
        "LYQ2_%=:\n\t"
        "s_add_u32 s83, s67, 0x188000\n\t"
        "s_add_u32 m0, s64, 8192\n\t"
        "s_nop 0\n\t"
        "buffer_load_dwordx4 v26, s[52:55], s83 offen offset:0 lds\n\t"
        "buffer_load_dwordx4 v27, s[52:55], s83 offen offset:1024 lds\n\t"
        "buffer_load_dwordx4 v28, s[52:55], s83 offen offset:2048 lds\n\t"
        "buffer_load_dwordx4 v29, s[52:55], s83 offen offset:3072 lds\n\t"
        "s_mov_b32 s23, 3\n\t"
        "s_memrealtime s[92:93]\n\t"
        "s_waitcnt lgkmcnt(0)\n\t"
        "s_sub_i32 s84, s74, s92\n\t"
        "s_cmp_gt_i32 s84, s66\n\t"
        "s_cbranch_scc0 LXQ3_%=\n\t"
        "LYQ3_%=:\n\t"
        "s_add_u32 s83, s67, 0x8000\n\t"
        "s_add_u32 m0, s64, 12288\n\t"
        "s_nop 0\n\t"
        "buffer_load_dwordx4 v26, s[52:55], s83 offen offset:0 lds\n\t"
        "buffer_load_dwordx4 v27, s[52:55], s83 offen offset:1024 lds\n\t"
        "buffer_load_dwordx4 v28, s[52:55], s83 offen offset:2048 lds\n\t"
        "buffer_load_dwordx4 v29, s[52:55], s83 offen offset:3072 lds\n\t"
        "s_waitcnt vmcnt(22)\n\t"
        "s_sub_u32 s35, s35, 1\n\t"
        "s_cbranch_scc0 LWS0_%=\n\t"
        "s_mov_b32 s35, 0\n\t"
        "s_waitcnt vmcnt(12)\n\t"
        "LWS0_%=:\n\t"
        "ds_read_b64_tr_b8 a[0:1], v180 offset:0\n\t"
        "ds_read_b64_tr_b8 a[4:5], v181 offset:0\n\t"
        "ds_read_b64_tr_b8 a[8:9], v182 offset:0\n\t"
        "ds_read_b64_tr_b8 a[12:13], v183 offset:0\n\t"
        "ds_read_b64_tr_b8 a[16:17], v184 offset:0\n\t"
        "ds_read_b64_tr_b8 a[20:21], v185 offset:0\n\t"
        "ds_read_b64_tr_b8 a[24:25], v186 offset:0\n\t"
        "ds_read_b64_tr_b8 a[28:29], v187 offset:0\n\t"
        "s_memrealtime s[92:93]\n\t"
        "s_waitcnt lgkmcnt(0)\n\t"
        "s_sub_i32 s84, s74, s92\n\t"
        "s_cmp_gt_i32 s84, s66\n\t"
        "s_cbranch_scc0 LXQ4_%=\n\t"
        "LYQ4_%=:\n\t"
        "s_add_u32 s83, s67, 0x110000\n\t"
        "s_add_u32 m0, s64, 0\n\t"
        "s_nop 0\n\t"
        "buffer_load_dwordx4 v26, s[52:55], s83 offen offset:0 lds\n\t"
        "buffer_load_dwordx4 v27, s[52:55], s83 offen offset:1024 lds\n\t"
        "buffer_load_dwordx4 v28, s[52:55], s83 offen offset:2048 lds\n\t"
        "buffer_load_dwordx4 v29, s[52:55], s83 offen offset:3072 lds\n\t"
        "s_waitcnt vmcnt(22)\n\t"
        "s_sub_u32 s35, s35, 1\n\t"
        "s_cbranch_scc0 LWS1_%=\n\t"
        "s_mov_b32 s35, 0\n\t"
        "s_waitcnt vmcnt(12)\n\t"
        "LWS1_%=:\n\t"
        "ds_read_b64_tr_b8 a[130:131], v180 offset:4096\n\t"
        "ds_read_b64_tr_b8 a[134:135], v181 offset:4096\n\t"
        "ds_read_b64_tr_b8 a[138:139], v182 offset:4096\n\t"
        "ds_read_b64_tr_b8 a[142:143], v183 offset:4096\n\t"
        "ds_read_b64_tr_b8 a[146:147], v184 offset:4096\n\t"
        "ds_read_b64_tr_b8 a[150:151], v185 offset:4096\n\t"
        "ds_read_b64_tr_b8 a[154:155], v186 offset:4096\n\t"
        "ds_read_b64_tr_b8 a[158:159], v187 offset:4096\n\t"
        "s_memrealtime s[92:93]\n\t"
        "s_waitcnt lgkmcnt(0)\n\t"
        "s_sub_i32 s84, s74, s92\n\t"
        "s_cmp_gt_i32 s84, s66\n\t"
        "s_cbranch_scc0 LXQ5_%=\n\t"
        "LYQ5_%=:\n\t"
        "s_add_u32 s83, s67, 0x190000\n\t"
        "s_add_u32 m0, s64, 4096\n\t"
        "s_nop 0\n\t"
        "buffer_load_dwordx4 v26, s[52:55], s83 offen offset:0 lds\n\t"
        "buffer_load_dwordx4 v27, s[52:55], s83 offen offset:1024 lds\n\t"
        "buffer_load_dwordx4 v28, s[52:55], s83 offen offset:2048 lds\n\t"
        "buffer_load_dwordx4 v29, s[52:55], s83 offen offset:3072 lds\n\t"
        "s_waitcnt vmcnt(22)\n\t"
        "s_sub_u32 s35, s35, 1\n\t"
        "s_cbranch_scc0 LWS2_%=\n\t"
        "s_mov_b32 s35, 0\n\t"
        "s_waitcnt vmcnt(12)\n\t"
        "LWS2_%=:\n\t"
        "ds_read_b64_tr_b8 a[194:195], v180 offset:8192\n\t"
        "ds_read_b64_tr_b8 a[198:199], v181 offset:8192\n\t"
        "ds_read_b64_tr_b8 a[202:203], v182 offset:8192\n\t"
        "ds_read_b64_tr_b8 a[206:207], v183 offset:8192\n\t"
        "ds_read_b64_tr_b8 a[210:211], v184 offset:8192\n\t"
        "ds_read_b64_tr_b8 a[214:215], v185 offset:8192\n\t"
        "ds_read_b64_tr_b8 a[218:219], v186 offset:8192\n\t"
        "ds_read_b64_tr_b8 a[222:223], v187 offset:8192\n\t"
        "s_memrealtime s[92:93]\n\t"
        "s_waitcnt lgkmcnt(0)\n\t"
        "s_sub_i32 s84, s74, s92\n\t"
        "s_cmp_gt_i32 s84, s66\n\t"
        "s_cbranch_scc0 LXQ6_%=\n\t"
        "LYQ6_%=:\n\t"
        "s_add_u32 s83, s67, 0x10000\n\t"
        "s_add_u32 m0, s64, 8192\n\t"
        "s_nop 0\n\t"
        "buffer_load_dwordx4 v26, s[52:55], s83 offen offset:0 lds\n\t"
        "buffer_load_dwordx4 v27, s[52:55], s83 offen offset:1024 lds\n\t"
        "buffer_load_dwordx4 v28, s[52:55], s83 offen offset:2048 lds\n\t"
        "buffer_load_dwordx4 v29, s[52:55], s83 offen offset:3072 lds\n\t"
        "s_waitcnt vmcnt(22)\n\t"
        "s_sub_u32 s35, s35, 1\n\t"
        "s_cbranch_scc0 LWS3_%=\n\t"
        "s_mov_b32 s35, 0\n\t"
        "s_waitcnt vmcnt(12)\n\t"
        "LWS3_%=:\n\t"
        "ds_read_b64_tr_b8 a[2:3], v180 offset:12288\n\t"
        "ds_read_b64_tr_b8 a[6:7], v181 offset:12288\n\t"
        "ds_read_b64_tr_b8 a[10:11], v182 offset:12288\n\t"
        "ds_read_b64_tr_b8 a[14:15], v183 offset:12288\n\t"
        "ds_read_b64_tr_b8 a[18:19], v184 offset:12288\n\t"
        "ds_read_b64_tr_b8 a[22:23], v185 offset:12288\n\t"
        "ds_read_b64_tr_b8 a[26:27], v186 offset:12288\n\t"
        "ds_read_b64_tr_b8 a[30:31], v187 offset:12288\n\t"
        "s_memrealtime s[92:93]\n\t"
        "s_waitcnt lgkmcnt(0)\n\t"
        "s_sub_i32 s84, s74, s92\n\t"
        "s_cmp_gt_i32 s84, s66\n\t"
        "s_cbranch_scc0 LXQ7_%=\n\t"
        "LYQ7_%=:\n\t"
        "s_add_u32 s83, s67, 0x118000\n\t"
        "s_add_u32 m0, s64, 12288\n\t"
        "s_nop 0\n\t"
        "buffer_load_dwordx4 v26, s[52:55], s83 offen offset:0 lds\n\t"
        "buffer_load_dwordx4 v27, s[52:55], s83 offen offset:1024 lds\n\t"
        "buffer_load_dwordx4 v28, s[52:55], s83 offen offset:2048 lds\n\t"
        "buffer_load_dwordx4 v29, s[52:55], s83 offen offset:3072 lds\n\t"
        "s_waitcnt vmcnt(22)\n\t"
        "s_sub_u32 s35, s35, 1\n\t"
        "s_cbranch_scc0 LWS4_%=\n\t"
        "s_mov_b32 s35, 0\n\t"
        "s_waitcnt vmcnt(12)\n\t"
        "LWS4_%=:\n\t"
        "ds_read_b64_tr_b8 a[160:161], v180 offset:0\n\t"
        "ds_read_b64_tr_b8 a[164:165], v181 offset:0\n\t"
        "ds_read_b64_tr_b8 a[168:169], v182 offset:0\n\t"
        "ds_read_b64_tr_b8 a[172:173], v183 offset:0\n\t"
        "ds_read_b64_tr_b8 a[176:177], v184 offset:0\n\t"
        "ds_read_b64_tr_b8 a[180:181], v185 offset:0\n\t"
        "ds_read_b64_tr_b8 a[184:185], v186 offset:0\n\t"
        "ds_read_b64_tr_b8 a[188:189], v187 offset:0\n\t"
        "s_memrealtime s[92:93]\n\t"
        "s_waitcnt lgkmcnt(0)\n\t"
        "s_sub_i32 s84, s74, s92\n\t"
        "s_cmp_gt_i32 s84, s66\n\t"
        "s_cbranch_scc0 LXQ8_%=\n\t"
        "LYQ8_%=:\n\t"
        "s_add_u32 s83, s67, 0x198000\n\t"
        "s_add_u32 m0, s64, 0\n\t"
        "s_nop 0\n\t"
        "buffer_load_dwordx4 v26, s[52:55], s83 offen offset:0 lds\n\t"
        "buffer_load_dwordx4 v27, s[52:55], s83 offen offset:1024 lds\n\t"
        "buffer_load_dwordx4 v28, s[52:55], s83 offen offset:2048 lds\n\t"
        "buffer_load_dwordx4 v29, s[52:55], s83 offen offset:3072 lds\n\t"
        "s_waitcnt vmcnt(22)\n\t"
        "s_sub_u32 s35, s35, 1\n\t"
        "s_cbranch_scc0 LWS5_%=\n\t"
        "s_mov_b32 s35, 0\n\t"
        "s_waitcnt vmcnt(12)\n\t"
        "LWS5_%=:\n\t"
        "ds_read_b64_tr_b8 a[224:225], v180 offset:4096\n\t"
        "ds_read_b64_tr_b8 a[228:229], v181 offset:4096\n\t"
        "ds_read_b64_tr_b8 a[232:233], v182 offset:4096\n\t"
        "ds_read_b64_tr_b8 a[236:237], v183 offset:4096\n\t"
        "ds_read_b64_tr_b8 a[240:241], v184 offset:4096\n\t"
        "ds_read_b64_tr_b8 a[244:245], v185 offset:4096\n\t"
        "ds_read_b64_tr_b8 a[248:249], v186 offset:4096\n\t"
        "ds_read_b64_tr_b8 a[252:253], v187 offset:4096\n\t"
        "s_memrealtime s[92:93]\n\t"
        "s_waitcnt lgkmcnt(0)\n\t"
        "s_sub_i32 s84, s74, s92\n\t"
        "s_cmp_gt_i32 s84, s66\n\t"
        "s_cbranch_scc0 LXQ9_%=\n\t"
        "LYQ9_%=:\n\t"
        "s_add_u32 s83, s67, 0x18000\n\t"
        "s_add_u32 m0, s64, 4096\n\t"
        "s_nop 0\n\t"
        "buffer_load_dwordx4 v26, s[52:55], s83 offen offset:0 lds\n\t"
        "buffer_load_dwordx4 v27, s[52:55], s83 offen offset:1024 lds\n\t"
        "buffer_load_dwordx4 v28, s[52:55], s83 offen offset:2048 lds\n\t"
        "buffer_load_dwordx4 v29, s[52:55], s83 offen offset:3072 lds\n\t"
        "s_waitcnt vmcnt(22)\n\t"
        "s_sub_u32 s35, s35, 1\n\t"
        "s_cbranch_scc0 LWS6_%=\n\t"
        "s_mov_b32 s35, 0\n\t"
        "s_waitcnt vmcnt(12)\n\t"
        "LWS6_%=:\n\t"
        "ds_read_b64_tr_b8 a[32:33], v180 offset:8192\n\t"
        "ds_read_b64_tr_b8 a[36:37], v181 offset:8192\n\t"
        "ds_read_b64_tr_b8 a[40:41], v182 offset:8192\n\t"
        "ds_read_b64_tr_b8 a[44:45], v183 offset:8192\n\t"
        "ds_read_b64_tr_b8 a[48:49], v184 offset:8192\n\t"
        "ds_read_b64_tr_b8 a[52:53], v185 offset:8192\n\t"
        "ds_read_b64_tr_b8 a[56:57], v186 offset:8192\n\t"
        "ds_read_b64_tr_b8 a[60:61], v187 offset:8192\n\t"
        "s_memrealtime s[92:93]\n\t"
        "s_waitcnt lgkmcnt(0)\n\t"
        "s_sub_i32 s84, s74, s92\n\t"
        "s_cmp_gt_i32 s84, s66\n\t"
        "s_cbranch_scc0 LXQ10_%=\n\t"
        "LYQ10_%=:\n\t"
        "s_add_u32 s83, s67, 0x180000\n\t"
        "s_add_u32 m0, s64, 8192\n\t"
        "s_nop 0\n\t"
        "buffer_load_dwordx4 v26, s[52:55], s83 offen offset:0 lds\n\t"
        "buffer_load_dwordx4 v27, s[52:55], s83 offen offset:1024 lds\n\t"
        "buffer_load_dwordx4 v28, s[52:55], s83 offen offset:2048 lds\n\t"
        "buffer_load_dwordx4 v29, s[52:55], s83 offen offset:3072 lds\n\t"
        "s_waitcnt vmcnt(22)\n\t"
        "s_sub_u32 s35, s35, 1\n\t"
        "s_cbranch_scc0 LWS7_%=\n\t"
        "s_mov_b32 s35, 0\n\t"
        "s_waitcnt vmcnt(12)\n\t"
        "LWS7_%=:\n\t"
        "ds_read_b64_tr_b8 a[162:163], v180 offset:12288\n\t"
        "ds_read_b64_tr_b8 a[166:167], v181 offset:12288\n\t"
        "ds_read_b64_tr_b8 a[170:171], v182 offset:12288\n\t"
        "ds_read_b64_tr_b8 a[174:175], v183 offset:12288\n\t"
        "ds_read_b64_tr_b8 a[178:179], v184 offset:12288\n\t"
        "ds_read_b64_tr_b8 a[182:183], v185 offset:12288\n\t"
        "ds_read_b64_tr_b8 a[186:187], v186 offset:12288\n\t"
        "ds_read_b64_tr_b8 a[190:191], v187 offset:12288\n\t"
        "s_memrealtime s[92:93]\n\t"
        "s_waitcnt lgkmcnt(0)\n\t"
        "s_sub_i32 s84, s74, s92\n\t"
        "s_cmp_gt_i32 s84, s66\n\t"
        "s_cbranch_scc0 LXQ11_%=\n\t"
        "LYQ11_%=:\n\t"
        "s_add_u32 s83, s67, 0x80000\n\t"
        "s_add_u32 m0, s64, 12288\n\t"
        "s_nop 0\n\t"
        "buffer_load_dwordx4 v26, s[52:55], s83 offen offset:0 lds\n\t"
        "buffer_load_dwordx4 v27, s[52:55], s83 offen offset:1024 lds\n\t"
        "buffer_load_dwordx4 v28, s[52:55], s83 offen offset:2048 lds\n\t"
        "buffer_load_dwordx4 v29, s[52:55], s83 offen offset:3072 lds\n\t"
        "s_waitcnt vmcnt(22)\n\t"
        "s_sub_u32 s35, s35, 1\n\t"
        "s_cbranch_scc0 LWS8_%=\n\t"
        "s_mov_b32 s35, 0\n\t"
        "s_waitcnt vmcnt(12)\n\t"
        "LWS8_%=:\n\t"
        "ds_read_b64_tr_b8 a[226:227], v180 offset:0\n\t"
        "ds_read_b64_tr_b8 a[230:231], v181 offset:0\n\t"
        "ds_read_b64_tr_b8 a[234:235], v182 offset:0\n\t"
        "ds_read_b64_tr_b8 a[238:239], v183 offset:0\n\t"
        "ds_read_b64_tr_b8 a[242:243], v184 offset:0\n\t"
        "ds_read_b64_tr_b8 a[246:247], v185 offset:0\n\t"
        "ds_read_b64_tr_b8 a[250:251], v186 offset:0\n\t"
        "ds_read_b64_tr_b8 a[254:255], v187 offset:0\n\t"
        "s_memrealtime s[92:93]\n\t"
        "s_waitcnt lgkmcnt(0)\n\t"
        "s_sub_i32 s84, s74, s92\n\t"
        "s_cmp_gt_i32 s84, s66\n\t"
        "s_cbranch_scc0 LXQ12_%=\n\t"
        "LYQ12_%=:\n\t"
        "s_add_u32 s83, s67, 0x88000\n\t"
        "s_add_u32 m0, s64, 0\n\t"
        "s_nop 0\n\t"
        "buffer_load_dwordx4 v26, s[52:55], s83 offen offset:0 lds\n\t"
        "buffer_load_dwordx4 v27, s[52:55], s83 offen offset:1024 lds\n\t"
        "buffer_load_dwordx4 v28, s[52:55], s83 offen offset:2048 lds\n\t"
        "buffer_load_dwordx4 v29, s[52:55], s83 offen offset:3072 lds\n\t"
        "s_waitcnt vmcnt(22)\n\t"
        "s_sub_u32 s35, s35, 1\n\t"
        "s_cbranch_scc0 LWS9_%=\n\t"
        "s_mov_b32 s35, 0\n\t"
        "s_waitcnt vmcnt(12)\n\t"
        "LWS9_%=:\n\t"
        "ds_read_b64_tr_b8 a[34:35], v180 offset:4096\n\t"
        "ds_read_b64_tr_b8 a[38:39], v181 offset:4096\n\t"
        "ds_read_b64_tr_b8 a[42:43], v182 offset:4096\n\t"
        "ds_read_b64_tr_b8 a[46:47], v183 offset:4096\n\t"
        "ds_read_b64_tr_b8 a[50:51], v184 offset:4096\n\t"
        "ds_read_b64_tr_b8 a[54:55], v185 offset:4096\n\t"
        "ds_read_b64_tr_b8 a[58:59], v186 offset:4096\n\t"
        "ds_read_b64_tr_b8 a[62:63], v187 offset:4096\n\t"
        "s_memrealtime s[92:93]\n\t"
        "s_waitcnt lgkmcnt(0)\n\t"
        "s_sub_i32 s84, s74, s92\n\t"
        "s_cmp_gt_i32 s84, s66\n\t"
        "s_cbranch_scc0 LXQ13_%=\n\t"
        "LYQ13_%=:\n\t"
        "s_add_u32 s83, s67, 0x90000\n\t"
        "s_add_u32 m0, s64, 4096\n\t"
        "s_nop 0\n\t"
        "buffer_load_dwordx4 v26, s[52:55], s83 offen offset:0 lds\n\t"
        "buffer_load_dwordx4 v27, s[52:55], s83 offen offset:1024 lds\n\t"
        "buffer_load_dwordx4 v28, s[52:55], s83 offen offset:2048 lds\n\t"
        "buffer_load_dwordx4 v29, s[52:55], s83 offen offset:3072 lds\n\t"
        "s_waitcnt vmcnt(22)\n\t"
        "s_sub_u32 s35, s35, 1\n\t"
        "s_cbranch_scc0 LWS10_%=\n\t"
        "s_mov_b32 s35, 0\n\t"
        "s_waitcnt vmcnt(12)\n\t"
        "LWS10_%=:\n\t"
        "ds_read_b64_tr_b8 a[192:193], v180 offset:8192\n\t"
        "ds_read_b64_tr_b8 a[196:197], v181 offset:8192\n\t"
        "ds_read_b64_tr_b8 a[200:201], v182 offset:8192\n\t"
        "ds_read_b64_tr_b8 a[204:205], v183 offset:8192\n\t"
        "ds_read_b64_tr_b8 a[208:209], v184 offset:8192\n\t"
        "ds_read_b64_tr_b8 a[212:213], v185 offset:8192\n\t"
        "ds_read_b64_tr_b8 a[216:217], v186 offset:8192\n\t"
        "ds_read_b64_tr_b8 a[220:221], v187 offset:8192\n\t"
        "s_memrealtime s[92:93]\n\t"
        "s_waitcnt lgkmcnt(0)\n\t"
        "s_sub_i32 s84, s74, s92\n\t"
        "s_cmp_gt_i32 s84, s66\n\t"
        "s_cbranch_scc0 LXQ14_%=\n\t"
        "LYQ14_%=:\n\t"
        "s_add_u32 s83, s67, 0x98000\n\t"
        "s_add_u32 m0, s64, 8192\n\t"
        "s_nop 0\n\t"
        "buffer_load_dwordx4 v26, s[52:55], s83 offen offset:0 lds\n\t"
        "buffer_load_dwordx4 v27, s[52:55], s83 offen offset:1024 lds\n\t"
        "buffer_load_dwordx4 v28, s[52:55], s83 offen offset:2048 lds\n\t"
        "buffer_load_dwordx4 v29, s[52:55], s83 offen offset:3072 lds\n\t"
        "s_waitcnt vmcnt(22)\n\t"
        "s_sub_u32 s35, s35, 1\n\t"
        "s_cbranch_scc0 LWS11_%=\n\t"
        "s_mov_b32 s35, 0\n\t"
        "s_waitcnt vmcnt(12)\n\t"
        "LWS11_%=:\n\t"
        "ds_read_b64_tr_b8 a[64:65], v180 offset:12288\n\t"
        "ds_read_b64_tr_b8 a[68:69], v181 offset:12288\n\t"
        "ds_read_b64_tr_b8 a[72:73], v182 offset:12288\n\t"
        "ds_read_b64_tr_b8 a[76:77], v183 offset:12288\n\t"
        "ds_read_b64_tr_b8 a[80:81], v184 offset:12288\n\t"
        "ds_read_b64_tr_b8 a[84:85], v185 offset:12288\n\t"
        "ds_read_b64_tr_b8 a[88:89], v186 offset:12288\n\t"
        "ds_read_b64_tr_b8 a[92:93], v187 offset:12288\n\t"
        "s_memrealtime s[92:93]\n\t"
        "s_waitcnt lgkmcnt(0)\n\t"
        "s_sub_i32 s84, s74, s92\n\t"
        "s_cmp_gt_i32 s84, s66\n\t"
        "s_cbranch_scc0 LXQ15_%=\n\t"
        "LYQ15_%=:\n\t"
        "s_add_u32 s83, s67, 0x100000\n\t"
        "s_add_u32 m0, s64, 12288\n\t"
        "s_nop 0\n\t"
        "buffer_load_dwordx4 v26, s[52:55], s83 offen offset:0 lds\n\t"
        "buffer_load_dwordx4 v27, s[52:55], s83 offen offset:1024 lds\n\t"
        "buffer_load_dwordx4 v28, s[52:55], s83 offen offset:2048 lds\n\t"
        "buffer_load_dwordx4 v29, s[52:55], s83 offen offset:3072 lds\n\t"
        "buffer_load_dwordx4 v[64:67], v16, s[36:39], s78 offen offset:0 sc0 sc1\n\t"
        "buffer_load_dwordx4 v[68:71], v16, s[36:39], s78 offen offset:1024 sc0 sc1\n\t"
        "buffer_load_dwordx4 v[72:75], v16, s[36:39], s78 offen offset:2048 sc0 sc1\n\t"
        "buffer_load_dwordx4 v[76:79], v16, s[36:39], s78 offen offset:3072 sc0 sc1\n\t"
        "buffer_load_dwordx4 v[80:83], v30, s[36:39], s78 offen offset:0 sc0 sc1\n\t"
        "buffer_load_dwordx4 v[84:87], v30, s[36:39], s78 offen offset:1024 sc0 sc1\n\t"
        "buffer_load_dwordx4 v[88:91], v30, s[36:39], s78 offen offset:2048 sc0 sc1\n\t"
        "buffer_load_dwordx4 v[92:95], v30, s[36:39], s78 offen offset:3072 sc0 sc1\n\t"
        "buffer_load_dword v128, v17, s[36:39], s78 offen sc0 sc1\n\t"
        "s_waitcnt vmcnt(31)\n\t"
        "s_sub_u32 s35, s35, 1\n\t"
        "s_cbranch_scc0 LWS12_%=\n\t"
        "s_mov_b32 s35, 0\n\t"
        "s_waitcnt vmcnt(21)\n\t"
        "LWS12_%=:\n\t"
        "ds_read_b64_tr_b8 a[66:67], v180 offset:0\n\t"
        "ds_read_b64_tr_b8 a[70:71], v181 offset:0\n\t"
        "ds_read_b64_tr_b8 a[74:75], v182 offset:0\n\t"
        "ds_read_b64_tr_b8 a[78:79], v183 offset:0\n\t"
        "ds_read_b64_tr_b8 a[82:83], v184 offset:0\n\t"
        "ds_read_b64_tr_b8 a[86:87], v185 offset:0\n\t"
        "ds_read_b64_tr_b8 a[90:91], v186 offset:0\n\t"
        "ds_read_b64_tr_b8 a[94:95], v187 offset:0\n\t"
        "s_memrealtime s[92:93]\n\t"
        "s_waitcnt lgkmcnt(0)\n\t"
        "s_mov_b32 s23, 99\n\t"
        "s_mov_b32 s66, 8\n\t"
        "s_sub_i32 s84, s74, s92\n\t"
        "s_cmp_gt_i32 s84, s66\n\t"
        "s_cbranch_scc0 LXR13_%=\n\t"
        "LYR13_%=:\n\t"
        "s_waitcnt vmcnt(27)\n\t"
        "s_sub_u32 s35, s35, 1\n\t"
        "s_cbranch_scc0 LWS13_%=\n\t"
        "s_mov_b32 s35, 0\n\t"
        "s_waitcnt vmcnt(17)\n\t"
        "LWS13_%=:\n\t"
        "ds_read_b64_tr_b8 a[96:97], v180 offset:4096\n\t"
        "ds_read_b64_tr_b8 a[100:101], v181 offset:4096\n\t"
        "ds_read_b64_tr_b8 a[104:105], v182 offset:4096\n\t"
        "ds_read_b64_tr_b8 a[108:109], v183 offset:4096\n\t"
        "ds_read_b64_tr_b8 a[112:113], v184 offset:4096\n\t"
        "ds_read_b64_tr_b8 a[116:117], v185 offset:4096\n\t"
        "ds_read_b64_tr_b8 a[120:121], v186 offset:4096\n\t"
        "ds_read_b64_tr_b8 a[124:125], v187 offset:4096\n\t"
        "s_memrealtime s[92:93]\n\t"
        "s_waitcnt lgkmcnt(0)\n\t"
        "s_sub_i32 s84, s74, s92\n\t"
        "s_cmp_gt_i32 s84, s66\n\t"
        "s_cbranch_scc0 LXR14_%=\n\t"
        "LYR14_%=:\n\t"
        "s_waitcnt vmcnt(23)\n\t"
        "s_sub_u32 s35, s35, 1\n\t"
        "s_cbranch_scc0 LWS14_%=\n\t"
        "s_mov_b32 s35, 0\n\t"
        "s_waitcnt vmcnt(13)\n\t"
        "LWS14_%=:\n\t"
        "ds_read_b64_tr_b8 a[98:99], v180 offset:8192\n\t"
        "ds_read_b64_tr_b8 a[102:103], v181 offset:8192\n\t"
        "ds_read_b64_tr_b8 a[106:107], v182 offset:8192\n\t"
        "ds_read_b64_tr_b8 a[110:111], v183 offset:8192\n\t"
        "ds_read_b64_tr_b8 a[114:115], v184 offset:8192\n\t"
        "ds_read_b64_tr_b8 a[118:119], v185 offset:8192\n\t"
        "ds_read_b64_tr_b8 a[122:123], v186 offset:8192\n\t"
        "ds_read_b64_tr_b8 a[126:127], v187 offset:8192\n\t"
        "s_memrealtime s[92:93]\n\t"
        "s_waitcnt lgkmcnt(0)\n\t"
        "s_sub_i32 s84, s74, s92\n\t"
        "s_cmp_gt_i32 s84, s66\n\t"
        "s_cbranch_scc0 LXR15_%=\n\t"
        "LYR15_%=:\n\t"
        "s_waitcnt vmcnt(19)\n\t"
        "s_sub_u32 s35, s35, 1\n\t"
        "s_cbranch_scc0 LWS15_%=\n\t"
        "s_mov_b32 s35, 0\n\t"
        "s_waitcnt vmcnt(9)\n\t"
        "LWS15_%=:\n\t"
        "ds_read_b64_tr_b8 a[128:129], v180 offset:12288\n\t"
        "ds_read_b64_tr_b8 a[132:133], v181 offset:12288\n\t"
        "ds_read_b64_tr_b8 a[136:137], v182 offset:12288\n\t"
        "ds_read_b64_tr_b8 a[140:141], v183 offset:12288\n\t"
        "ds_read_b64_tr_b8 a[144:145], v184 offset:12288\n\t"
        "ds_read_b64_tr_b8 a[148:149], v185 offset:12288\n\t"
        "ds_read_b64_tr_b8 a[152:153], v186 offset:12288\n\t"
        "ds_read_b64_tr_b8 a[156:157], v187 offset:12288\n\t"
        "s_waitcnt lgkmcnt(0)\n\t"
        "s_cmp_lg_u32 s69, 0\n\t"
        "s_cbranch_scc1 LHS%=\n\t"
        "s_mov_b32 s66, 0x7fffffff\n\t"
        "s_swappc_b64 s[96:97], s[98:99]\n\t"
        "LHS%=:\n\t"
        "s_cmp_lg_u32 s35, 0\n\t"
        "s_waitcnt vmcnt(10)\n\t"
        "s_cbranch_scc1 LHS2%=\n\t"
        "s_waitcnt vmcnt(0)\n\t"
        "LHS2%=:\n\t"
        "s_branch LCHK%=\n\t"
        "LSERV%=:\n\t"
        "s_waitcnt vmcnt(3)\n\t"
        "s_cmp_ge_u32 s76, s79\n\t"
        "s_cbranch_scc0 LSERV2%=\n\t"
        "s_mov_b32 s11, 1\n\t"
        "v_lshl_add_u32 v134, s77, 11, v22\n\t"
        "s_mul_i32 s83, s77, 8320\n\t"
        "v_add_u32 v135, s83, v21\n\t"
        "s_lshl3_add_u32 s84, s77, s63\n\t"
        "s_bfe_u64 s[84:85], s[30:31], s84\n\t"
        "s_lshl_b32 s81, s84, 18\n\t"
        "ds_read_b128 v[160:163], v134\n\t"
        "ds_read_b128 v[164:167], v134 offset:1024\n\t"
        "ds_add_u64 v25, v[144:145] offset:8320\n\t"
        "ds_add_u64 v25, v[146:147] offset:8832\n\t"
        "ds_write_b32 v24, v53\n\t"
        "ds_read_b32 v132, v23\n\t"
        "ds_read2_b64 v[152:155], v135 offset0:0 offset1:64\n\t"
        "ds_read2_b64 v[156:159], v135 offset0:2 offset1:66\n\t"
        "s_cmp_lt_u32 s77, s34\n\t"
        "s_cbranch_scc1 LFLP%=\n\t"
        "s_waitcnt lgkmcnt(6)\n\t"
        "s_branch LTAG%=\n\t"
        "LSERV2%=:\n\t"
        "buffer_load_dwordx4 v[64:67], v16, s[36:39], s78 offen offset:0 sc0 sc1\n\t"
        "buffer_load_dwordx4 v[68:71], v16, s[36:39], s78 offen offset:1024 sc0 sc1\n\t"
        "buffer_load_dwordx4 v[72:75], v16, s[36:39], s78 offen offset:2048 sc0 sc1\n\t"
        "buffer_load_dwordx4 v[76:79], v16, s[36:39], s78 offen offset:3072 sc0 sc1\n\t"
        "buffer_load_dwordx4 v[80:83], v30, s[36:39], s78 offen offset:0 sc0 sc1\n\t"
        "buffer_load_dwordx4 v[84:87], v30, s[36:39], s78 offen offset:1024 sc0 sc1\n\t"
        "buffer_load_dwordx4 v[88:91], v30, s[36:39], s78 offen offset:2048 sc0 sc1\n\t"
        "buffer_load_dwordx4 v[92:95], v30, s[36:39], s78 offen offset:3072 sc0 sc1\n\t"
        "buffer_load_dword v128, v17, s[36:39], s78 offen sc0 sc1\n\t"
        "ds_add_u64 v25, v[144:145] offset:8320\n\t"
        "ds_add_u64 v25, v[146:147] offset:8832\n\t"
        "ds_write_b32 v24, v53\n\t"
        "s_mov_b32 s19, -1\n\t"
        "v_mov_b64 v[144:145], 0\n\t"
        "v_mov_b64 v[146:147], 0\n\t"
        "s_branch LFL%=\n\t"
        "LPOLL%=:\n\t"
        "s_cmp_ge_u32 s76, s79\n\t"
        "s_cbranch_scc1 LFL%=\n\t"
        "buffer_load_dwordx4 v[64:67], v16, s[36:39], s78 offen offset:0 sc0 sc1\n\t"
        "buffer_load_dwordx4 v[68:71], v16, s[36:39], s78 offen offset:1024 sc0 sc1\n\t"
        "buffer_load_dwordx4 v[72:75], v16, s[36:39], s78 offen offset:2048 sc0 sc1\n\t"
        "buffer_load_dwordx4 v[76:79], v16, s[36:39], s78 offen offset:3072 sc0 sc1\n\t"
        "buffer_load_dwordx4 v[80:83], v30, s[36:39], s78 offen offset:0 sc0 sc1\n\t"
        "buffer_load_dwordx4 v[84:87], v30, s[36:39], s78 offen offset:1024 sc0 sc1\n\t"
        "buffer_load_dwordx4 v[88:91], v30, s[36:39], s78 offen offset:2048 sc0 sc1\n\t"
        "buffer_load_dwordx4 v[92:95], v30, s[36:39], s78 offen offset:3072 sc0 sc1\n\t"
        "buffer_load_dword v128, v17, s[36:39], s78 offen sc0 sc1\n\t"
        "LFL%=:\n\t"
        "s_mul_i32 s83, s77, 8320\n\t"
        "v_add_u32 v135, s83, v21\n\t"
        "v_lshl_add_u32 v134, s77, 11, v22\n\t"
        "ds_read_b32 v132, v23\n\t"
        "ds_read_b128 v[160:163], v134\n\t"
        "ds_read_b128 v[164:167], v134 offset:1024\n\t"
        "ds_read2_b64 v[152:155], v135 offset0:0 offset1:64\n\t"
        "ds_read2_b64 v[156:159], v135 offset0:2 offset1:66\n\t"
        "s_lshl3_add_u32 s84, s77, s63\n\t"
        "s_bfe_u64 s[84:85], s[30:31], s84\n\t"
        "s_lshl_b32 s81, s84, 18\n\t"
        "s_cmp_lt_u32 s77, s34\n\t"
        "s_cbranch_scc1 LFLP%=\n\t"
        "s_waitcnt lgkmcnt(2)\n\t"
        "LTAG%=:\n\t"
        "s_cmp_ge_u32 s77, s20\n\t"
        "s_cbranch_scc1 LIMP%=\n\t"
        "v_bitop3_b32 v133, v160, v162, s62 bitop3:0x7e\n\t"
        "v_bitop3_b32 v134, v164, v166, s62 bitop3:0x7e\n\t"
        "v_or_b32 v133, v133, v134\n\t"
        "v_cmp_le_u32 vcc, s87, v133\n\t"
        "s_cbranch_vccnz LIMP%=\n\t"
        "LFLP%=:\n\t"
        "s_waitcnt lgkmcnt(2)\n\t"
        "v_cmp_ge_i32 vcc, s77, v132\n\t"
        "s_cbranch_vccnz LSLOW%=\n\t"
        "LGO%=:\n\t"
        "s_cmp_lt_u32 s77, s34\n\t"
        "s_cbranch_scc1 LPUB%=\n\t"
        "s_waitcnt lgkmcnt(1)\n\t"
        "v_add3_u32 v160, v152, v160, s88\n\t"
        "v_add_u32 v161, v161, v153\n\t"
        "v_add3_u32 v162, v154, v162, s88\n\t"
        "v_add_u32 v163, v163, v155\n\t"
        "v_cvt_f32_i32 v168, v160\n\t"
        "v_cvt_f32_i32 v169, v161\n\t"
        "v_cvt_f32_i32 v170, v162\n\t"
        "v_cvt_f32_i32 v171, v163\n\t"
        "v_mul_f32 v168, v52, v168\n\t"
        "v_mul_f32 v169, v52, v169\n\t"
        "v_mul_f32 v170, v52, v170\n\t"
        "v_mul_f32 v171, v52, v171\n\t"
        "buffer_store_dwordx4 v[168:171], v19, s[48:51], s81 offen sc0 sc1\n\t"
        "s_waitcnt lgkmcnt(0)\n\t"
        "v_add3_u32 v164, v156, v164, s88\n\t"
        "v_add_u32 v165, v165, v157\n\t"
        "v_add3_u32 v166, v158, v166, s88\n\t"
        "v_add_u32 v167, v167, v159\n\t"
        "v_cvt_f32_i32 v172, v164\n\t"
        "v_cvt_f32_i32 v173, v165\n\t"
        "v_cvt_f32_i32 v174, v166\n\t"
        "v_cvt_f32_i32 v175, v167\n\t"
        "v_mul_f32 v172, v52, v172\n\t"
        "v_mul_f32 v173, v52, v173\n\t"
        "v_mul_f32 v174, v52, v174\n\t"
        "v_mul_f32 v175, v52, v175\n\t"
        "buffer_store_dwordx4 v[172:175], v20, s[48:51], s81 offen sc0 sc1\n\t"
        "LFLD%=:\n\t"
        "s_lshl_b32 s11, s11, 1\n\t"
        "s_add_u32 s77, s77, 1\n\t"
        "s_cmp_lt_u32 s76, s79\n\t"
        "s_cbranch_scc1 LWAIT2%=\n\t"
        "s_cmp_lt_u32 s77, s79\n\t"
        "s_cbranch_scc1 LFL%=\n\t"
        "s_branch LEND%=\n\t"
        "LPUB%=:\n\t"
        "s_lshl_b32 s80, s77, 13\n\t"
        "LPUB2%=:\n\t"
        "s_waitcnt lgkmcnt(0)\n\t"
        "buffer_store_dwordx4 v[152:155], v18, s[44:47], s80 offen offset:0 sc0 sc1\n\t"
        "buffer_store_dwordx4 v[156:159], v18, s[44:47], s80 offen offset:1024 sc0 sc1\n\t"
        "s_branch LFLD%=\n\t"
        "LSLOW%=:\n\t"
        "v_and_b32 v133, 0xffff, v132\n\t"
        "v_cmp_ge_i32 vcc, s77, v133\n\t"
        "s_cbranch_vccnz LLIGHT%=\n\t"
        "v_mov_b32 v52, 0x7fc00000\n\t"
        "s_mov_b32 s80, 0x40000000\n\t"
        "s_cmp_lt_u32 s77, s34\n\t"
        "s_cbranch_scc1 LPUB2%=\n\t"
        "s_branch LGO%=\n\t"
        "LLIGHT%=:\n\t"
        "s_cmp_lt_u32 s76, s79\n\t"
        "s_cbranch_scc1 LWAIT%=\n\t"
        "LLL%=:\n\t"
        "ds_read_b32 v132, v23\n\t"
        "ds_read2_b64 v[152:155], v135 offset0:0 offset1:64\n\t"
        "ds_read2_b64 v[156:159], v135 offset0:2 offset1:66\n\t"
        "s_add_u32 s85, s85, 1\n\t"
        "s_and_b32 s83, s85, 63\n\t"
        "s_cbranch_scc0 LTICK%=\n\t"
        "s_waitcnt lgkmcnt(2)\n\t"
        "v_cmp_ge_i32 vcc, s77, v132\n\t"
        "s_cbranch_vccnz LLL%=\n\t"
        "s_branch LGO%=\n\t"
        "LIMP%=:\n\t"
        "s_cmp_lg_u32 s82, 0\n\t"
        "s_cbranch_scc1 LFLP%=\n\t"
        "s_cmp_ge_u32 s77, s76\n\t"
        "s_cbranch_scc1 LWAIT%=\n\t"
        "s_cmp_eq_u32 s11, 0\n\t"
        "s_cbranch_scc1 LIMPV%=\n\t"
        "s_cmp_eq_u32 s11, 1\n\t"
        "s_mov_b32 s11, 0\n\t"
        "s_waitcnt vmcnt(2)\n\t"
        "s_cbranch_scc0 LIMPS%=\n\t"
        "s_waitcnt vmcnt(0)\n\t"
        "LIMPS%=:\n\t"
        "v_xor_b32 v55, s62, v55\n\t"
        "v_cmp_le_u32 vcc, s87, v55\n\t"
        "s_add_u32 s83, s20, 1\n\t"
        "s_cmp_lg_u32 s83, s79\n\t"
        "s_cbranch_scc1 LIMPV%=\n\t"
        "s_cmp_lg_u32 s77, s20\n\t"
        "s_cbranch_scc1 LIMPV%=\n\t"
        "s_cbranch_vccz LIMPL%=\n\t"
        "LIMPV%=:\n\t"
        "buffer_load_dword v55, v57, s[40:43], 0 offen sc0 sc1\n\t"
        "s_mov_b32 s20, s77\n\t"
        "s_lshl_b32 s84, s77, 11\n\t"
        "s_add_u32 s33, s84, s64\n\t"
        "s_sub_u32 s84, s77, s34\n\t"
        "s_lshl_b32 s89, s84, 13\n\t"
        "s_waitcnt vmcnt(0)\n\t"
        "v_xor_b32 v55, s62, v55\n\t"
        "v_cmp_le_u32 vcc, s87, v55\n\t"
        "s_ff1_i32_b64 s83, vcc\n\t"
        "s_add_u32 s83, s83, s34\n\t"
        "s_cmp_ge_u32 s20, s83\n\t"
        "s_cbranch_scc1 LWAIT%=\n\t"
        "LIMPL%=:\n\t"
        "s_mov_b32 m0, s33\n\t"
        "s_add_u32 s33, s33, 2048\n\t"
        "buffer_load_dwordx4 v18, s[40:43], s89 offen sc0 sc1 lds\n\t"
        "buffer_load_dwordx4 v18, s[40:43], s89 offen offset:1024 sc0 sc1 lds\n\t"
        "s_add_u32 s89, s89, 8192\n\t"
        "s_add_u32 s20, s20, 1\n\t"
        "s_cmp_lt_u32 s20, s83\n\t"
        "s_cbranch_scc1 LIMPL%=\n\t"
        "s_waitcnt vmcnt(0)\n\t"
        "LWAIT%=:\n\t"
        "s_cmp_ge_u32 s76, s79\n\t"
        "s_cbranch_scc1 LTICK%=\n\t"
        "s_waitcnt vmcnt(0)\n\t"
        "LWAIT2%=:\n\t"
        "s_waitcnt vmcnt(2)\n\t"
        "LCHK%=:\n\t"
        "v_bitop3_b32 v130, v67, v71, s60 bitop3:0x7e\n\t"
        "v_bitop3_b32 v131, v75, v79, s60 bitop3:0x7e\n\t"
        "v_bitop3_b32 v133, v83, v87, s60 bitop3:0x7e\n\t"
        "v_bitop3_b32 v134, v91, v95, s60 bitop3:0x7e\n\t"
        "v_xor_b32 v135, s61, v128\n\t"
        "v_or3_b32 v130, v130, v131, v133\n\t"
        "v_bitop3_b32 v130, v130, v134, s90 bitop3:0xa8\n\t"
        "v_and_or_b32 v130, v135, s91, v130\n\t"
        "v_cmp_ne_u32 s[94:95], 0, v130\n\t"
        "v_lshlrev_b32 v129, 3, v128\n\t"
        "v_xor_b32_sdwa v67, v67, v128 dst_sel:BYTE_3 dst_unused:UNUSED_PRESERVE src0_sel:BYTE_3 src1_sel:BYTE_0\n\t"
        "v_xor_b32_sdwa v71, v71, v128 dst_sel:BYTE_3 dst_unused:UNUSED_PRESERVE src0_sel:BYTE_3 src1_sel:BYTE_1\n\t"
        "v_xor_b32_sdwa v75, v75, v128 dst_sel:BYTE_3 dst_unused:UNUSED_PRESERVE src0_sel:BYTE_3 src1_sel:BYTE_2\n\t"
        "v_xor_b32_sdwa v79, v79, v128 dst_sel:BYTE_3 dst_unused:UNUSED_PRESERVE src0_sel:BYTE_3 src1_sel:BYTE_3\n\t"
        "v_xor_b32_sdwa v83, v83, v129 dst_sel:BYTE_3 dst_unused:UNUSED_PRESERVE src0_sel:BYTE_3 src1_sel:BYTE_0\n\t"
        "v_xor_b32_sdwa v87, v87, v129 dst_sel:BYTE_3 dst_unused:UNUSED_PRESERVE src0_sel:BYTE_3 src1_sel:BYTE_1\n\t"
        "v_xor_b32_sdwa v91, v91, v129 dst_sel:BYTE_3 dst_unused:UNUSED_PRESERVE src0_sel:BYTE_3 src1_sel:BYTE_2\n\t"
        "v_xor_b32_sdwa v95, v95, v129 dst_sel:BYTE_3 dst_unused:UNUSED_PRESERVE src0_sel:BYTE_3 src1_sel:BYTE_3\n\t"
        "s_waitcnt lgkmcnt(0)\n\t"
        "s_cmp_eq_u64 s[94:95], 0\n\t"
        "s_cbranch_scc1 LENTER%=\n\t"
        "s_cmp_lg_u32 s82, 0\n\t"
        "s_cbranch_scc1 LENTER%=\n\t"
        "LTICK%=:\n\t"
        "s_waitcnt lgkmcnt(0)\n\t"
        "s_add_u32 s86, s86, 1\n\t"
        "s_and_b32 s83, s86, 255\n\t"
        "s_cbranch_scc1 LPOLL%=\n\t"
        "s_memrealtime s[92:93]\n\t"
        "v_mov_b32 v133, 0\n\t"
        "global_load_dword v134, v133, s[70:71] sc1\n\t"
        "s_waitcnt vmcnt(0) lgkmcnt(0)\n\t"
        "v_readfirstlane_b32 s84, v134\n\t"
        "s_sub_u32 s92, s92, s68\n\t"
        "s_cmp_eq_u32 s84, s65\n\t"
        "s_cbranch_scc1 LDIE%=\n\t"
        "s_cmp_lt_u32 s92, 30000000\n\t"
        "s_cbranch_scc1 LPOLL%=\n\t"
        "LDIE%=:\n\t"
        "v_mov_b32 v134, s65\n\t"
        "global_store_dword v133, v134, s[70:71] sc1\n\t"
        "s_mov_b32 s82, 0xffff0000\n\t"
        "v_or_b32 v53, s82, v53\n\t"
        "v_mov_b32 v52, 0x7fc00000\n\t"
        "s_branch LPOLL%=\n\t"
        ".p2align 6\n\t"
        "LENTER%=:\n\t"
        "s_cmp_lg_u32 s86, s19\n\t"
        "s_mov_b32 s19, s86\n\t"
        "s_cbranch_scc1 LBLKL0%=\n\t"
        "s_cmp_ge_u32 s77, s34\n\t"
        "s_cbranch_scc1 LBLKF0%=\n\t"
        "LBLKP0%=:\n\t"
        "v_mfma_i32_16x16x64_i8 v[140:143], a[0:3], v[64:67], v[148:151]\n\t"
        "s_mul_i32 s78, s21, 8448\n\t"
        "v_add_u32 v25, 8320, v25\n\t"
        "v_mfma_i32_16x16x64_i8 v[140:143], a[32:35], v[68:71], v[140:143]\n\t"
        "buffer_load_dword v128, v17, s[36:39], s78 offen sc0 sc1\n\t"
        "v_mfma_i32_16x16x64_i8 v[140:143], a[64:67], v[72:75], v[140:143]\n\t"
        "buffer_load_dwordx4 v[96:99], v16, s[36:39], s78 offen offset:0 sc0 sc1\n\t"
        "v_mfma_i32_16x16x64_i8 v[140:143], a[96:99], v[76:79], v[140:143]\n\t"
        "buffer_load_dwordx4 v[100:103], v16, s[36:39], s78 offen offset:1024 sc0 sc1\n\t"
        "v_mfma_i32_16x16x64_i8 v[140:143], a[128:131], v[80:83], v[140:143]\n\t"
        "s_mul_i32 s83, s77, 8320\n\t"
        "ds_add_u64 v25, v[144:145] offset:0\n\t"
        "v_mfma_i32_16x16x64_i8 v[140:143], a[160:163], v[84:87], v[140:143]\n\t"
        "buffer_load_dwordx4 v[104:107], v16, s[36:39], s78 offen offset:2048 sc0 sc1\n\t"
        "v_mfma_i32_16x16x64_i8 v[140:143], a[192:195], v[88:91], v[140:143]\n\t"
        "s_mov_b32 m0, s33\n\t"
        "ds_add_u64 v25, v[146:147] offset:512\n\t"
        "v_mfma_i32_16x16x64_i8 v[140:143], a[224:227], v[92:95], v[140:143]\n\t"
        "buffer_load_dwordx4 v[108:111], v16, s[36:39], s78 offen offset:3072 sc0 sc1\n\t"
        "v_mfma_i32_16x16x64_i8 v[144:147], a[4:7], v[64:67], v[148:151]\n\t"
        "buffer_load_dwordx4 v[112:115], v30, s[36:39], s78 offen offset:0 sc0 sc1\n\t"
        "v_mfma_i32_16x16x64_i8 v[144:147], a[36:39], v[68:71], v[144:147]\n\t"
        "buffer_load_dwordx4 v[116:119], v30, s[36:39], s78 offen offset:1024 sc0 sc1\n\t"
        "v_mfma_i32_16x16x64_i8 v[144:147], a[68:71], v[72:75], v[144:147]\n\t"
        "buffer_load_dwordx4 v[120:123], v30, s[36:39], s78 offen offset:2048 sc0 sc1\n\t"
        "v_mfma_i32_16x16x64_i8 v[144:147], a[100:103], v[76:79], v[144:147]\n\t"
        "buffer_load_dwordx4 v[124:127], v30, s[36:39], s78 offen offset:3072 sc0 sc1\n\t"
        "v_mfma_i32_16x16x64_i8 v[144:147], a[132:135], v[80:83], v[144:147]\n\t"
        "s_lshr_b64 s[28:29], s[28:29], 6\n\t"
        "ds_add_u64 v25, v[140:141] offset:1040\n\t"
        "v_mfma_i32_16x16x64_i8 v[144:147], a[164:167], v[84:87], v[144:147]\n\t"
        "s_and_b32 s21, s28, 63\n\t"
        "ds_write_b32 v24, v53\n\t"
        "v_mfma_i32_16x16x64_i8 v[144:147], a[196:199], v[88:91], v[144:147]\n\t"
        "s_lshl_b32 s81, s77, 13\n\t"
        "ds_add_u64 v25, v[142:143] offset:1552\n\t"
        "v_mfma_i32_16x16x64_i8 v[144:147], a[228:231], v[92:95], v[144:147]\n\t"
        "s_add_u32 s76, s76, 1\n\t"
        "v_add_u32 v135, s83, v21\n\t"
        "v_add_u32 v53, 1, v53\n\t"
        "v_mfma_i32_16x16x64_i8 v[140:143], a[8:11], v[64:67], v[148:151]\n\t"
        "v_mfma_i32_16x16x64_i8 v[140:143], a[40:43], v[68:71], v[140:143]\n\t"
        "v_mfma_i32_16x16x64_i8 v[140:143], a[72:75], v[72:75], v[140:143]\n\t"
        "v_mfma_i32_16x16x64_i8 v[140:143], a[104:107], v[76:79], v[140:143]\n\t"
        "v_mfma_i32_16x16x64_i8 v[140:143], a[136:139], v[80:83], v[140:143]\n\t"
        "ds_add_u64 v25, v[144:145] offset:2080\n\t"
        "v_mfma_i32_16x16x64_i8 v[140:143], a[168:171], v[84:87], v[140:143]\n\t"
        "v_mfma_i32_16x16x64_i8 v[140:143], a[200:203], v[88:91], v[140:143]\n\t"
        "ds_add_u64 v25, v[146:147] offset:2592\n\t"
        "v_mfma_i32_16x16x64_i8 v[140:143], a[232:235], v[92:95], v[140:143]\n\t"
        "ds_read_b32 v132, v23\n\t"
        "ds_read2_b64 v[152:155], v135 offset0:0 offset1:64\n\t"
        "v_mfma_i32_16x16x64_i8 v[144:147], a[12:15], v[64:67], v[148:151]\n\t"
        "ds_read2_b64 v[156:159], v135 offset0:2 offset1:66\n\t"
        "v_mfma_i32_16x16x64_i8 v[144:147], a[44:47], v[68:71], v[144:147]\n\t"
        "v_mfma_i32_16x16x64_i8 v[144:147], a[76:79], v[72:75], v[144:147]\n\t"
        "v_mfma_i32_16x16x64_i8 v[144:147], a[108:111], v[76:79], v[144:147]\n\t"
        "v_mfma_i32_16x16x64_i8 v[144:147], a[140:143], v[80:83], v[144:147]\n\t"
        "ds_add_u64 v25, v[140:141] offset:3120\n\t"
        "v_mfma_i32_16x16x64_i8 v[144:147], a[172:175], v[84:87], v[144:147]\n\t"
        "s_waitcnt   vmcnt(9)\n\t"
        "v_xor_b32 v55, s62, v55\n\t"
        "v_cmp_gt_u32 s[94:95], s87, v55\n\t"
        "v_mfma_i32_16x16x64_i8 v[144:147], a[204:207], v[88:91], v[144:147]\n\t"
        "ds_add_u64 v25, v[142:143] offset:3632\n\t"
        "v_mfma_i32_16x16x64_i8 v[144:147], a[236:239], v[92:95], v[144:147]\n\t"
        "s_cmp_lg_u64 s[94:95], 0\n\t"
        "s_cselect_b32 s9, s89, 0x40000000\n\t"
        "s_cselect_b32 s10, 2048, 0\n\t"
        "v_mfma_i32_16x16x64_i8 v[140:143], a[16:19], v[64:67], v[148:151]\n\t"
        "buffer_load_dwordx4 v18, s[40:43], s9 offen sc0 sc1 lds\n\t"
        "v_mfma_i32_16x16x64_i8 v[140:143], a[48:51], v[68:71], v[140:143]\n\t"
        "buffer_load_dwordx4 v18, s[40:43], s9 offen offset:1024 sc0 sc1 lds\n\t"
        "v_mfma_i32_16x16x64_i8 v[140:143], a[80:83], v[72:75], v[140:143]\n\t"
        "s_addc_u32 s20, s20, 0\n\t"
        "v_mfma_i32_16x16x64_i8 v[140:143], a[112:115], v[76:79], v[140:143]\n\t"
        "s_add_u32 s33, s33, s10\n\t"
        "v_mfma_i32_16x16x64_i8 v[140:143], a[144:147], v[80:83], v[140:143]\n\t"
        "s_lshl2_add_u32 s89, s10, s89\n\t"
        "ds_add_u64 v25, v[144:145] offset:4160\n\t"
        "v_mfma_i32_16x16x64_i8 v[140:143], a[176:179], v[84:87], v[140:143]\n\t"
        "s_waitcnt lgkmcnt(3)\n\t"
        "v_cmp_ge_i32 vcc, s77, v132\n\t"
        "v_mfma_i32_16x16x64_i8 v[140:143], a[208:211], v[88:91], v[140:143]\n\t"
        "s_cmp_eq_u64 vcc, 0\n\t"
        "ds_add_u64 v25, v[146:147] offset:4672\n\t"
        "v_mfma_i32_16x16x64_i8 v[140:143], a[240:243], v[92:95], v[140:143]\n\t"
        "s_cselect_b32 s80, s81, 0x40000000\n\t"
        "s_addc_u32 s77, s77, 0\n\t"
        "s_cmp_ge_u32 s77, s34\n\t"
        "v_mfma_i32_16x16x64_i8 v[144:147], a[20:23], v[64:67], v[148:151]\n\t"
        "s_cselect_b64 vcc, -1, 0\n\t"
        "v_mfma_i32_16x16x64_i8 v[144:147], a[52:55], v[68:71], v[144:147]\n\t"
        "v_mfma_i32_16x16x64_i8 v[144:147], a[84:87], v[72:75], v[144:147]\n\t"
        "v_mfma_i32_16x16x64_i8 v[144:147], a[116:119], v[76:79], v[144:147]\n\t"
        "v_mfma_i32_16x16x64_i8 v[144:147], a[148:151], v[80:83], v[144:147]\n\t"
        "buffer_store_dwordx4 v[152:155], v18, s[44:47], s80 offen offset:0 sc0 sc1\n\t"
        "ds_add_u64 v25, v[140:141] offset:5200\n\t"
        "v_mfma_i32_16x16x64_i8 v[144:147], a[180:183], v[84:87], v[144:147]\n\t"
        "v_mfma_i32_16x16x64_i8 v[144:147], a[212:215], v[88:91], v[144:147]\n\t"
        "buffer_store_dwordx4 v[156:159], v18, s[44:47], s80 offen offset:1024 sc0 sc1\n\t"
        "ds_add_u64 v25, v[142:143] offset:5712\n\t"
        "v_mfma_i32_16x16x64_i8 v[144:147], a[244:247], v[92:95], v[144:147]\n\t"
        "buffer_load_dword v55, v56, s[40:43], s89 offen sc0 sc1\n\t"
        "v_mfma_i32_16x16x64_i8 v[140:143], a[24:27], v[64:67], v[148:151]\n\t"
        "v_mfma_i32_16x16x64_i8 v[140:143], a[56:59], v[68:71], v[140:143]\n\t"
        "v_mfma_i32_16x16x64_i8 v[140:143], a[88:91], v[72:75], v[140:143]\n\t"
        "s_waitcnt vmcnt(9)\n\t"
        "v_bitop3_b32 v130, v99, v103, s60 bitop3:0x7e\n\t"
        "v_bitop3_b32 v131, v107, v111, s60 bitop3:0x7e\n\t"
        "v_mfma_i32_16x16x64_i8 v[140:143], a[120:123], v[76:79], v[140:143]\n\t"
        "v_xor_b32 v135, s61, v128\n\t"
        "v_lshlrev_b32 v129, 3, v128\n\t"
        "v_mfma_i32_16x16x64_i8 v[140:143], a[152:155], v[80:83], v[140:143]\n\t"
        "ds_add_u64 v25, v[144:145] offset:6240\n\t"
        "v_mfma_i32_16x16x64_i8 v[140:143], a[184:187], v[84:87], v[140:143]\n\t"
        "v_xor_b32_sdwa v99, v99, v128 dst_sel:BYTE_3 dst_unused:UNUSED_PRESERVE src0_sel:BYTE_3 src1_sel:BYTE_0\n\t"
        "v_xor_b32_sdwa v103, v103, v128 dst_sel:BYTE_3 dst_unused:UNUSED_PRESERVE src0_sel:BYTE_3 src1_sel:BYTE_1\n\t"
        "v_mfma_i32_16x16x64_i8 v[140:143], a[216:219], v[88:91], v[140:143]\n\t"
        "ds_add_u64 v25, v[146:147] offset:6752\n\t"
        "v_mfma_i32_16x16x64_i8 v[140:143], a[248:251], v[92:95], v[140:143]\n\t"
        "s_waitcnt  vmcnt(7)\n\t"
        "v_bitop3_b32 v133, v115, v119, s60 bitop3:0x7e\n\t"
        "v_xor_b32_sdwa v107, v107, v128 dst_sel:BYTE_3 dst_unused:UNUSED_PRESERVE src0_sel:BYTE_3 src1_sel:BYTE_2\n\t"
        "v_mfma_i32_16x16x64_i8 v[144:147], a[28:31], v[64:67], v[148:151]\n\t"
        "v_or3_b32 v130, v130, v131, v133\n\t"
        "v_xor_b32_sdwa v111, v111, v128 dst_sel:BYTE_3 dst_unused:UNUSED_PRESERVE src0_sel:BYTE_3 src1_sel:BYTE_3\n\t"
        "v_mfma_i32_16x16x64_i8 v[144:147], a[60:63], v[68:71], v[144:147]\n\t"
        "s_waitcnt  vmcnt(5)\n\t"
        "v_bitop3_b32 v134, v123, v127, s60 bitop3:0x7e\n\t"
        "v_bitop3_b32 v130, v130, v134, s90 bitop3:0xa8\n\t"
        "v_mfma_i32_16x16x64_i8 v[144:147], a[92:95], v[72:75], v[144:147]\n\t"
        "v_and_or_b32 v130, v135, s91, v130\n\t"
        "v_cmp_ne_u32 s[94:95], 0, v130\n\t"
        "v_mfma_i32_16x16x64_i8 v[144:147], a[124:127], v[76:79], v[144:147]\n\t"
        "v_xor_b32_sdwa v115, v115, v129 dst_sel:BYTE_3 dst_unused:UNUSED_PRESERVE src0_sel:BYTE_3 src1_sel:BYTE_0\n\t"
        "v_xor_b32_sdwa v119, v119, v129 dst_sel:BYTE_3 dst_unused:UNUSED_PRESERVE src0_sel:BYTE_3 src1_sel:BYTE_1\n\t"
        "v_mfma_i32_16x16x64_i8 v[144:147], a[156:159], v[80:83], v[144:147]\n\t"
        "s_cmp_eq_u64 s[94:95], 0\n\t"
        "ds_add_u64 v25, v[140:141] offset:7280\n\t"
        "v_mfma_i32_16x16x64_i8 v[144:147], a[188:191], v[84:87], v[144:147]\n\t"
        "v_xor_b32_sdwa v123, v123, v129 dst_sel:BYTE_3 dst_unused:UNUSED_PRESERVE src0_sel:BYTE_3 src1_sel:BYTE_2\n\t"
        "v_xor_b32_sdwa v127, v127, v129 dst_sel:BYTE_3 dst_unused:UNUSED_PRESERVE src0_sel:BYTE_3 src1_sel:BYTE_3\n\t"
        "v_mfma_i32_16x16x64_i8 v[144:147], a[220:223], v[88:91], v[144:147]\n\t"
        "ds_add_u64 v25, v[142:143] offset:7792\n\t"
        "v_mfma_i32_16x16x64_i8 v[144:147], a[252:255], v[92:95], v[144:147]\n\t"
        "s_cbranch_scc0 LSERV%=\n\t"
        "s_cbranch_vccnz LBLKF1%=\n\t"
        "LBLKP1%=:\n\t"
        "v_mfma_i32_16x16x64_i8 v[140:143], a[0:3], v[96:99], v[148:151]\n\t"
        "s_mul_i32 s78, s21, 8448\n\t"
        "v_add_u32 v25, 8320, v25\n\t"
        "v_mfma_i32_16x16x64_i8 v[140:143], a[32:35], v[100:103], v[140:143]\n\t"
        "buffer_load_dword v128, v17, s[36:39], s78 offen sc0 sc1\n\t"
        "v_mfma_i32_16x16x64_i8 v[140:143], a[64:67], v[104:107], v[140:143]\n\t"
        "buffer_load_dwordx4 v[64:67], v16, s[36:39], s78 offen offset:0 sc0 sc1\n\t"
        "v_mfma_i32_16x16x64_i8 v[140:143], a[96:99], v[108:111], v[140:143]\n\t"
        "buffer_load_dwordx4 v[68:71], v16, s[36:39], s78 offen offset:1024 sc0 sc1\n\t"
        "v_mfma_i32_16x16x64_i8 v[140:143], a[128:131], v[112:115], v[140:143]\n\t"
        "s_mul_i32 s83, s77, 8320\n\t"
        "ds_add_u64 v25, v[144:145] offset:0\n\t"
        "v_mfma_i32_16x16x64_i8 v[140:143], a[160:163], v[116:119], v[140:143]\n\t"
        "buffer_load_dwordx4 v[72:75], v16, s[36:39], s78 offen offset:2048 sc0 sc1\n\t"
        "v_mfma_i32_16x16x64_i8 v[140:143], a[192:195], v[120:123], v[140:143]\n\t"
        "s_mov_b32 m0, s33\n\t"
        "ds_add_u64 v25, v[146:147] offset:512\n\t"
        "v_mfma_i32_16x16x64_i8 v[140:143], a[224:227], v[124:127], v[140:143]\n\t"
        "buffer_load_dwordx4 v[76:79], v16, s[36:39], s78 offen offset:3072 sc0 sc1\n\t"
        "v_mfma_i32_16x16x64_i8 v[144:147], a[4:7], v[96:99], v[148:151]\n\t"
        "buffer_load_dwordx4 v[80:83], v30, s[36:39], s78 offen offset:0 sc0 sc1\n\t"
        "v_mfma_i32_16x16x64_i8 v[144:147], a[36:39], v[100:103], v[144:147]\n\t"
        "buffer_load_dwordx4 v[84:87], v30, s[36:39], s78 offen offset:1024 sc0 sc1\n\t"
        "v_mfma_i32_16x16x64_i8 v[144:147], a[68:71], v[104:107], v[144:147]\n\t"
        "buffer_load_dwordx4 v[88:91], v30, s[36:39], s78 offen offset:2048 sc0 sc1\n\t"
        "v_mfma_i32_16x16x64_i8 v[144:147], a[100:103], v[108:111], v[144:147]\n\t"
        "buffer_load_dwordx4 v[92:95], v30, s[36:39], s78 offen offset:3072 sc0 sc1\n\t"
        "v_mfma_i32_16x16x64_i8 v[144:147], a[132:135], v[112:115], v[144:147]\n\t"
        "s_lshr_b64 s[28:29], s[28:29], 6\n\t"
        "ds_add_u64 v25, v[140:141] offset:1040\n\t"
        "v_mfma_i32_16x16x64_i8 v[144:147], a[164:167], v[116:119], v[144:147]\n\t"
        "s_and_b32 s21, s28, 63\n\t"
        "ds_write_b32 v24, v53\n\t"
        "v_mfma_i32_16x16x64_i8 v[144:147], a[196:199], v[120:123], v[144:147]\n\t"
        "s_lshl_b32 s81, s77, 13\n\t"
        "ds_add_u64 v25, v[142:143] offset:1552\n\t"
        "v_mfma_i32_16x16x64_i8 v[144:147], a[228:231], v[124:127], v[144:147]\n\t"
        "s_add_u32 s76, s76, 1\n\t"
        "v_add_u32 v135, s83, v21\n\t"
        "v_add_u32 v53, 1, v53\n\t"
        "v_mfma_i32_16x16x64_i8 v[140:143], a[8:11], v[96:99], v[148:151]\n\t"
        "v_mfma_i32_16x16x64_i8 v[140:143], a[40:43], v[100:103], v[140:143]\n\t"
        "v_mfma_i32_16x16x64_i8 v[140:143], a[72:75], v[104:107], v[140:143]\n\t"
        "v_mfma_i32_16x16x64_i8 v[140:143], a[104:107], v[108:111], v[140:143]\n\t"
        "v_mfma_i32_16x16x64_i8 v[140:143], a[136:139], v[112:115], v[140:143]\n\t"
        "ds_add_u64 v25, v[144:145] offset:2080\n\t"
        "v_mfma_i32_16x16x64_i8 v[140:143], a[168:171], v[116:119], v[140:143]\n\t"
        "v_mfma_i32_16x16x64_i8 v[140:143], a[200:203], v[120:123], v[140:143]\n\t"
        "ds_add_u64 v25, v[146:147] offset:2592\n\t"
        "v_mfma_i32_16x16x64_i8 v[140:143], a[232:235], v[124:127], v[140:143]\n\t"
        "ds_read_b32 v132, v23\n\t"
        "ds_read2_b64 v[152:155], v135 offset0:0 offset1:64\n\t"
        "v_mfma_i32_16x16x64_i8 v[144:147], a[12:15], v[96:99], v[148:151]\n\t"
        "ds_read2_b64 v[156:159], v135 offset0:2 offset1:66\n\t"
        "v_mfma_i32_16x16x64_i8 v[144:147], a[44:47], v[100:103], v[144:147]\n\t"
        "v_mfma_i32_16x16x64_i8 v[144:147], a[76:79], v[104:107], v[144:147]\n\t"
        "v_mfma_i32_16x16x64_i8 v[144:147], a[108:111], v[108:111], v[144:147]\n\t"
        "v_mfma_i32_16x16x64_i8 v[144:147], a[140:143], v[112:115], v[144:147]\n\t"
        "ds_add_u64 v25, v[140:141] offset:3120\n\t"
        "v_mfma_i32_16x16x64_i8 v[144:147], a[172:175], v[116:119], v[144:147]\n\t"
        "s_waitcnt   vmcnt(9)\n\t"
        "v_xor_b32 v55, s62, v55\n\t"
        "v_cmp_gt_u32 s[94:95], s87, v55\n\t"
        "v_mfma_i32_16x16x64_i8 v[144:147], a[204:207], v[120:123], v[144:147]\n\t"
        "ds_add_u64 v25, v[142:143] offset:3632\n\t"
        "v_mfma_i32_16x16x64_i8 v[144:147], a[236:239], v[124:127], v[144:147]\n\t"
        "s_cmp_lg_u64 s[94:95], 0\n\t"
        "s_cselect_b32 s9, s89, 0x40000000\n\t"
        "s_cselect_b32 s10, 2048, 0\n\t"
        "v_mfma_i32_16x16x64_i8 v[140:143], a[16:19], v[96:99], v[148:151]\n\t"
        "buffer_load_dwordx4 v18, s[40:43], s9 offen sc0 sc1 lds\n\t"
        "v_mfma_i32_16x16x64_i8 v[140:143], a[48:51], v[100:103], v[140:143]\n\t"
        "buffer_load_dwordx4 v18, s[40:43], s9 offen offset:1024 sc0 sc1 lds\n\t"
        "v_mfma_i32_16x16x64_i8 v[140:143], a[80:83], v[104:107], v[140:143]\n\t"
        "s_addc_u32 s20, s20, 0\n\t"
        "v_mfma_i32_16x16x64_i8 v[140:143], a[112:115], v[108:111], v[140:143]\n\t"
        "s_add_u32 s33, s33, s10\n\t"
        "v_mfma_i32_16x16x64_i8 v[140:143], a[144:147], v[112:115], v[140:143]\n\t"
        "s_lshl2_add_u32 s89, s10, s89\n\t"
        "ds_add_u64 v25, v[144:145] offset:4160\n\t"
        "v_mfma_i32_16x16x64_i8 v[140:143], a[176:179], v[116:119], v[140:143]\n\t"
        "s_waitcnt lgkmcnt(3)\n\t"
        "v_cmp_ge_i32 vcc, s77, v132\n\t"
        "v_mfma_i32_16x16x64_i8 v[140:143], a[208:211], v[120:123], v[140:143]\n\t"
        "s_cmp_eq_u64 vcc, 0\n\t"
        "ds_add_u64 v25, v[146:147] offset:4672\n\t"
        "v_mfma_i32_16x16x64_i8 v[140:143], a[240:243], v[124:127], v[140:143]\n\t"
        "s_cselect_b32 s80, s81, 0x40000000\n\t"
        "s_addc_u32 s77, s77, 0\n\t"
        "s_cmp_ge_u32 s77, s34\n\t"
        "v_mfma_i32_16x16x64_i8 v[144:147], a[20:23], v[96:99], v[148:151]\n\t"
        "s_cselect_b64 vcc, -1, 0\n\t"
        "v_mfma_i32_16x16x64_i8 v[144:147], a[52:55], v[100:103], v[144:147]\n\t"
        "v_mfma_i32_16x16x64_i8 v[144:147], a[84:87], v[104:107], v[144:147]\n\t"
        "v_mfma_i32_16x16x64_i8 v[144:147], a[116:119], v[108:111], v[144:147]\n\t"
        "v_mfma_i32_16x16x64_i8 v[144:147], a[148:151], v[112:115], v[144:147]\n\t"
        "buffer_store_dwordx4 v[152:155], v18, s[44:47], s80 offen offset:0 sc0 sc1\n\t"
        "ds_add_u64 v25, v[140:141] offset:5200\n\t"
        "v_mfma_i32_16x16x64_i8 v[144:147], a[180:183], v[116:119], v[144:147]\n\t"
        "v_mfma_i32_16x16x64_i8 v[144:147], a[212:215], v[120:123], v[144:147]\n\t"
        "buffer_store_dwordx4 v[156:159], v18, s[44:47], s80 offen offset:1024 sc0 sc1\n\t"
        "ds_add_u64 v25, v[142:143] offset:5712\n\t"
        "v_mfma_i32_16x16x64_i8 v[144:147], a[244:247], v[124:127], v[144:147]\n\t"
        "buffer_load_dword v55, v56, s[40:43], s89 offen sc0 sc1\n\t"
        "v_mfma_i32_16x16x64_i8 v[140:143], a[24:27], v[96:99], v[148:151]\n\t"
        "v_mfma_i32_16x16x64_i8 v[140:143], a[56:59], v[100:103], v[140:143]\n\t"
        "v_mfma_i32_16x16x64_i8 v[140:143], a[88:91], v[104:107], v[140:143]\n\t"
        "s_waitcnt vmcnt(9)\n\t"
        "v_bitop3_b32 v130, v67, v71, s60 bitop3:0x7e\n\t"
        "v_bitop3_b32 v131, v75, v79, s60 bitop3:0x7e\n\t"
        "v_mfma_i32_16x16x64_i8 v[140:143], a[120:123], v[108:111], v[140:143]\n\t"
        "v_xor_b32 v135, s61, v128\n\t"
        "v_lshlrev_b32 v129, 3, v128\n\t"
        "v_mfma_i32_16x16x64_i8 v[140:143], a[152:155], v[112:115], v[140:143]\n\t"
        "ds_add_u64 v25, v[144:145] offset:6240\n\t"
        "v_mfma_i32_16x16x64_i8 v[140:143], a[184:187], v[116:119], v[140:143]\n\t"
        "v_xor_b32_sdwa v67, v67, v128 dst_sel:BYTE_3 dst_unused:UNUSED_PRESERVE src0_sel:BYTE_3 src1_sel:BYTE_0\n\t"
        "v_xor_b32_sdwa v71, v71, v128 dst_sel:BYTE_3 dst_unused:UNUSED_PRESERVE src0_sel:BYTE_3 src1_sel:BYTE_1\n\t"
        "v_mfma_i32_16x16x64_i8 v[140:143], a[216:219], v[120:123], v[140:143]\n\t"
        "ds_add_u64 v25, v[146:147] offset:6752\n\t"
        "v_mfma_i32_16x16x64_i8 v[140:143], a[248:251], v[124:127], v[140:143]\n\t"
        "s_waitcnt  vmcnt(7)\n\t"
        "v_bitop3_b32 v133, v83, v87, s60 bitop3:0x7e\n\t"
        "v_xor_b32_sdwa v75, v75, v128 dst_sel:BYTE_3 dst_unused:UNUSED_PRESERVE src0_sel:BYTE_3 src1_sel:BYTE_2\n\t"
        "v_mfma_i32_16x16x64_i8 v[144:147], a[28:31], v[96:99], v[148:151]\n\t"
        "v_or3_b32 v130, v130, v131, v133\n\t"
        "v_xor_b32_sdwa v79, v79, v128 dst_sel:BYTE_3 dst_unused:UNUSED_PRESERVE src0_sel:BYTE_3 src1_sel:BYTE_3\n\t"
        "v_mfma_i32_16x16x64_i8 v[144:147], a[60:63], v[100:103], v[144:147]\n\t"
        "s_waitcnt  vmcnt(5)\n\t"
        "v_bitop3_b32 v134, v91, v95, s60 bitop3:0x7e\n\t"
        "v_bitop3_b32 v130, v130, v134, s90 bitop3:0xa8\n\t"
        "v_mfma_i32_16x16x64_i8 v[144:147], a[92:95], v[104:107], v[144:147]\n\t"
        "v_and_or_b32 v130, v135, s91, v130\n\t"
        "v_cmp_ne_u32 s[94:95], 0, v130\n\t"
        "v_mfma_i32_16x16x64_i8 v[144:147], a[124:127], v[108:111], v[144:147]\n\t"
        "v_xor_b32_sdwa v83, v83, v129 dst_sel:BYTE_3 dst_unused:UNUSED_PRESERVE src0_sel:BYTE_3 src1_sel:BYTE_0\n\t"
        "v_xor_b32_sdwa v87, v87, v129 dst_sel:BYTE_3 dst_unused:UNUSED_PRESERVE src0_sel:BYTE_3 src1_sel:BYTE_1\n\t"
        "v_mfma_i32_16x16x64_i8 v[144:147], a[156:159], v[112:115], v[144:147]\n\t"
        "s_cmp_eq_u64 s[94:95], 0\n\t"
        "ds_add_u64 v25, v[140:141] offset:7280\n\t"
        "v_mfma_i32_16x16x64_i8 v[144:147], a[188:191], v[116:119], v[144:147]\n\t"
        "v_xor_b32_sdwa v91, v91, v129 dst_sel:BYTE_3 dst_unused:UNUSED_PRESERVE src0_sel:BYTE_3 src1_sel:BYTE_2\n\t"
        "v_xor_b32_sdwa v95, v95, v129 dst_sel:BYTE_3 dst_unused:UNUSED_PRESERVE src0_sel:BYTE_3 src1_sel:BYTE_3\n\t"
        "v_mfma_i32_16x16x64_i8 v[144:147], a[220:223], v[120:123], v[144:147]\n\t"
        "ds_add_u64 v25, v[142:143] offset:7792\n\t"
        "v_mfma_i32_16x16x64_i8 v[144:147], a[252:255], v[124:127], v[144:147]\n\t"
        "s_cbranch_scc0 LSERV%=\n\t"
        "s_cbranch_vccz LBLKP0%=\n\t"
        "LBLKF0%=:\n\t"
        "v_mfma_i32_16x16x64_i8 v[140:143], a[0:3], v[64:67], v[148:151]\n\t"
        "s_mul_i32 s78, s21, 8448\n\t"
        "v_add_u32 v25, 8320, v25\n\t"
        "v_lshl_add_u32 v134, s77, 11, v22\n\t"
        "v_mfma_i32_16x16x64_i8 v[140:143], a[32:35], v[68:71], v[140:143]\n\t"
        "buffer_load_dword v128, v17, s[36:39], s78 offen sc0 sc1\n\t"
        "v_mfma_i32_16x16x64_i8 v[140:143], a[64:67], v[72:75], v[140:143]\n\t"
        "buffer_load_dwordx4 v[96:99], v16, s[36:39], s78 offen offset:0 sc0 sc1\n\t"
        "v_mfma_i32_16x16x64_i8 v[140:143], a[96:99], v[76:79], v[140:143]\n\t"
        "buffer_load_dwordx4 v[100:103], v16, s[36:39], s78 offen offset:1024 sc0 sc1\n\t"
        "v_mfma_i32_16x16x64_i8 v[140:143], a[128:131], v[80:83], v[140:143]\n\t"
        "s_mul_i32 s83, s77, 8320\n\t"
        "ds_add_u64 v25, v[144:145] offset:0\n\t"
        "v_mfma_i32_16x16x64_i8 v[140:143], a[160:163], v[84:87], v[140:143]\n\t"
        "buffer_load_dwordx4 v[104:107], v16, s[36:39], s78 offen offset:2048 sc0 sc1\n\t"
        "v_mfma_i32_16x16x64_i8 v[140:143], a[192:195], v[88:91], v[140:143]\n\t"
        "s_mov_b32 m0, s33\n\t"
        "ds_add_u64 v25, v[146:147] offset:512\n\t"
        "v_mfma_i32_16x16x64_i8 v[140:143], a[224:227], v[92:95], v[140:143]\n\t"
        "buffer_load_dwordx4 v[108:111], v16, s[36:39], s78 offen offset:3072 sc0 sc1\n\t"
        "v_mfma_i32_16x16x64_i8 v[144:147], a[4:7], v[64:67], v[148:151]\n\t"
        "buffer_load_dwordx4 v[112:115], v30, s[36:39], s78 offen offset:0 sc0 sc1\n\t"
        "v_mfma_i32_16x16x64_i8 v[144:147], a[36:39], v[68:71], v[144:147]\n\t"
        "buffer_load_dwordx4 v[116:119], v30, s[36:39], s78 offen offset:1024 sc0 sc1\n\t"
        "v_mfma_i32_16x16x64_i8 v[144:147], a[68:71], v[72:75], v[144:147]\n\t"
        "buffer_load_dwordx4 v[120:123], v30, s[36:39], s78 offen offset:2048 sc0 sc1\n\t"
        "v_mfma_i32_16x16x64_i8 v[144:147], a[100:103], v[76:79], v[144:147]\n\t"
        "buffer_load_dwordx4 v[124:127], v30, s[36:39], s78 offen offset:3072 sc0 sc1\n\t"
        "v_mfma_i32_16x16x64_i8 v[144:147], a[132:135], v[80:83], v[144:147]\n\t"
        "s_cmp_lt_u32 s77, s20\n\t"
        "ds_add_u64 v25, v[140:141] offset:1040\n\t"
        "v_mfma_i32_16x16x64_i8 v[144:147], a[164:167], v[84:87], v[144:147]\n\t"
        "s_cselect_b32 s8, s77, 64\n\t"
        "ds_write_b32 v24, v53\n\t"
        "v_mfma_i32_16x16x64_i8 v[144:147], a[196:199], v[88:91], v[144:147]\n\t"
        "s_lshr_b64 s[28:29], s[28:29], 6\n\t"
        "ds_add_u64 v25, v[142:143] offset:1552\n\t"
        "v_mfma_i32_16x16x64_i8 v[144:147], a[228:231], v[92:95], v[144:147]\n\t"
        "s_and_b32 s21, s28, 63\n\t"
        "v_add_u32 v135, s83, v21\n\t"
        "v_add_u32 v53, 1, v53\n\t"
        "v_mfma_i32_16x16x64_i8 v[140:143], a[8:11], v[64:67], v[148:151]\n\t"
        "s_waitcnt  vmcnt(12)\n\t"
        "ds_read_b128 v[160:163], v134\n\t"
        "ds_read_b128 v[164:167], v134 offset:1024\n\t"
        "v_mfma_i32_16x16x64_i8 v[140:143], a[40:43], v[68:71], v[140:143]\n\t"
        "s_lshl3_add_u32 s84, s77, s63\n\t"
        "ds_read_b32 v132, v23\n\t"
        "ds_read2_b64 v[152:155], v135 offset0:0 offset1:64\n\t"
        "v_mfma_i32_16x16x64_i8 v[140:143], a[72:75], v[72:75], v[140:143]\n\t"
        "s_bfe_u64 s[84:85], s[30:31], s84\n\t"
        "ds_read2_b64 v[156:159], v135 offset0:2 offset1:66\n\t"
        "v_mfma_i32_16x16x64_i8 v[140:143], a[104:107], v[76:79], v[140:143]\n\t"
        "s_lshl_b32 s80, s84, 18\n\t"
        "v_mfma_i32_16x16x64_i8 v[140:143], a[136:139], v[80:83], v[140:143]\n\t"
        "s_add_u32 s76, s76, 1\n\t"
        "ds_add_u64 v25, v[144:145] offset:2080\n\t"
        "v_mfma_i32_16x16x64_i8 v[140:143], a[168:171], v[84:87], v[140:143]\n\t"
        "s_waitcnt lgkmcnt(4)\n\t"
        "v_bitop3_b32 v133, v160, v162, s62 bitop3:0x7e\n\t"
        "v_bitop3_b32 v134, v164, v166, s62 bitop3:0x7e\n\t"
        "v_mfma_i32_16x16x64_i8 v[140:143], a[200:203], v[88:91], v[140:143]\n\t"
        "ds_add_u64 v25, v[146:147] offset:2592\n\t"
        "v_mfma_i32_16x16x64_i8 v[140:143], a[232:235], v[92:95], v[140:143]\n\t"
        "v_or_b32 v133, v133, v134\n\t"
        "v_cmp_le_u32 s[92:93], s87, v133\n\t"
        "v_mfma_i32_16x16x64_i8 v[144:147], a[12:15], v[64:67], v[148:151]\n\t"
        "s_waitcnt lgkmcnt(2)\n\t"
        "v_cmp_ge_i32 vcc, s8, v132\n\t"
        "v_add3_u32 v160, v152, v160, s88\n\t"
        "v_mfma_i32_16x16x64_i8 v[144:147], a[44:47], v[68:71], v[144:147]\n\t"
        "v_add_u32 v161, v161, v153\n\t"
        "v_add3_u32 v162, v154, v162, s88\n\t"
        "v_mfma_i32_16x16x64_i8 v[144:147], a[76:79], v[72:75], v[144:147]\n\t"
        "v_add_u32 v163, v163, v155\n\t"
        "v_cvt_f32_i32 v168, v160\n\t"
        "v_mfma_i32_16x16x64_i8 v[144:147], a[108:111], v[76:79], v[144:147]\n\t"
        "v_cvt_f32_i32 v169, v161\n\t"
        "v_cvt_f32_i32 v170, v162\n\t"
        "v_mfma_i32_16x16x64_i8 v[144:147], a[140:143], v[80:83], v[144:147]\n\t"
        "ds_add_u64 v25, v[140:141] offset:3120\n\t"
        "v_mfma_i32_16x16x64_i8 v[144:147], a[172:175], v[84:87], v[144:147]\n\t"
        "s_waitcnt   vmcnt(9)\n\t"
        "v_xor_b32 v55, s62, v55\n\t"
        "v_cmp_gt_u32 s[94:95], s87, v55\n\t"
        "v_mfma_i32_16x16x64_i8 v[144:147], a[204:207], v[88:91], v[144:147]\n\t"
        "ds_add_u64 v25, v[142:143] offset:3632\n\t"
        "v_mfma_i32_16x16x64_i8 v[144:147], a[236:239], v[92:95], v[144:147]\n\t"
        "s_cmp_lg_u64 s[94:95], 0\n\t"
        "s_cselect_b32 s9, s89, 0x40000000\n\t"
        "s_cselect_b32 s10, 2048, 0\n\t"
        "v_mfma_i32_16x16x64_i8 v[140:143], a[16:19], v[64:67], v[148:151]\n\t"
        "buffer_load_dwordx4 v18, s[40:43], s9 offen sc0 sc1 lds\n\t"
        "v_mfma_i32_16x16x64_i8 v[140:143], a[48:51], v[68:71], v[140:143]\n\t"
        "buffer_load_dwordx4 v18, s[40:43], s9 offen offset:1024 sc0 sc1 lds\n\t"
        "v_mfma_i32_16x16x64_i8 v[140:143], a[80:83], v[72:75], v[140:143]\n\t"
        "s_addc_u32 s20, s20, 0\n\t"
        "v_cvt_f32_i32 v171, v163\n\t"
        "v_mul_f32 v168, v52, v168\n\t"
        "v_mfma_i32_16x16x64_i8 v[140:143], a[112:115], v[76:79], v[140:143]\n\t"
        "s_add_u32 s33, s33, s10\n\t"
        "v_mul_f32 v169, v52, v169\n\t"
        "v_mul_f32 v170, v52, v170\n\t"
        "v_mfma_i32_16x16x64_i8 v[140:143], a[144:147], v[80:83], v[140:143]\n\t"
        "s_lshl2_add_u32 s89, s10, s89\n\t"
        "ds_add_u64 v25, v[144:145] offset:4160\n\t"
        "v_mfma_i32_16x16x64_i8 v[140:143], a[176:179], v[84:87], v[140:143]\n\t"
        "s_or_b64 s[92:93], s[92:93], vcc\n\t"
        "v_mul_f32 v171, v52, v171\n\t"
        "v_add3_u32 v164, v156, v164, s88\n\t"
        "v_mfma_i32_16x16x64_i8 v[140:143], a[208:211], v[88:91], v[140:143]\n\t"
        "s_cselect_b32 s81, 0x40000000, s80\n\t"
        "ds_add_u64 v25, v[146:147] offset:4672\n\t"
        "v_mfma_i32_16x16x64_i8 v[140:143], a[240:243], v[92:95], v[140:143]\n\t"
        "s_subb_u32 s77, s77, -1\n\t"
        "v_add_u32 v165, v165, v157\n\t"
        "v_add3_u32 v166, v158, v166, s88\n\t"
        "v_mfma_i32_16x16x64_i8 v[144:147], a[20:23], v[64:67], v[148:151]\n\t"
        "buffer_store_dwordx4 v[168:171], v19, s[48:51], s81 offen sc0 sc1\n\t"
        "v_add_u32 v167, v167, v159\n\t"
        "v_cvt_f32_i32 v172, v164\n\t"
        "v_cvt_f32_i32 v173, v165\n\t"
        "v_mfma_i32_16x16x64_i8 v[144:147], a[52:55], v[68:71], v[144:147]\n\t"
        "v_cvt_f32_i32 v174, v166\n\t"
        "v_cvt_f32_i32 v175, v167\n\t"
        "v_mfma_i32_16x16x64_i8 v[144:147], a[84:87], v[72:75], v[144:147]\n\t"
        "v_mul_f32 v172, v52, v172\n\t"
        "v_mul_f32 v173, v52, v173\n\t"
        "v_mfma_i32_16x16x64_i8 v[144:147], a[116:119], v[76:79], v[144:147]\n\t"
        "v_mul_f32 v174, v52, v174\n\t"
        "v_mul_f32 v175, v52, v175\n\t"
        "v_mfma_i32_16x16x64_i8 v[144:147], a[148:151], v[80:83], v[144:147]\n\t"
        "buffer_store_dwordx4 v[172:175], v20, s[48:51], s81 offen sc0 sc1\n\t"
        "ds_add_u64 v25, v[140:141] offset:5200\n\t"
        "v_mfma_i32_16x16x64_i8 v[144:147], a[180:183], v[84:87], v[144:147]\n\t"
        "buffer_load_dword v55, v56, s[40:43], s89 offen sc0 sc1\n\t"
        "v_mfma_i32_16x16x64_i8 v[144:147], a[212:215], v[88:91], v[144:147]\n\t"
        "ds_add_u64 v25, v[142:143] offset:5712\n\t"
        "v_mfma_i32_16x16x64_i8 v[144:147], a[244:247], v[92:95], v[144:147]\n\t"
        "v_mfma_i32_16x16x64_i8 v[140:143], a[24:27], v[64:67], v[148:151]\n\t"
        "v_mfma_i32_16x16x64_i8 v[140:143], a[56:59], v[68:71], v[140:143]\n\t"
        "v_mfma_i32_16x16x64_i8 v[140:143], a[88:91], v[72:75], v[140:143]\n\t"
        "s_waitcnt vmcnt(9)\n\t"
        "v_bitop3_b32 v130, v99, v103, s60 bitop3:0x7e\n\t"
        "v_bitop3_b32 v131, v107, v111, s60 bitop3:0x7e\n\t"
        "v_mfma_i32_16x16x64_i8 v[140:143], a[120:123], v[76:79], v[140:143]\n\t"
        "v_xor_b32 v135, s61, v128\n\t"
        "v_lshlrev_b32 v129, 3, v128\n\t"
        "v_mfma_i32_16x16x64_i8 v[140:143], a[152:155], v[80:83], v[140:143]\n\t"
        "ds_add_u64 v25, v[144:145] offset:6240\n\t"
        "v_mfma_i32_16x16x64_i8 v[140:143], a[184:187], v[84:87], v[140:143]\n\t"
        "v_xor_b32_sdwa v99, v99, v128 dst_sel:BYTE_3 dst_unused:UNUSED_PRESERVE src0_sel:BYTE_3 src1_sel:BYTE_0\n\t"
        "v_xor_b32_sdwa v103, v103, v128 dst_sel:BYTE_3 dst_unused:UNUSED_PRESERVE src0_sel:BYTE_3 src1_sel:BYTE_1\n\t"
        "v_mfma_i32_16x16x64_i8 v[140:143], a[216:219], v[88:91], v[140:143]\n\t"
        "ds_add_u64 v25, v[146:147] offset:6752\n\t"
        "v_mfma_i32_16x16x64_i8 v[140:143], a[248:251], v[92:95], v[140:143]\n\t"
        "s_waitcnt  vmcnt(7)\n\t"
        "v_bitop3_b32 v133, v115, v119, s60 bitop3:0x7e\n\t"
        "v_xor_b32_sdwa v107, v107, v128 dst_sel:BYTE_3 dst_unused:UNUSED_PRESERVE src0_sel:BYTE_3 src1_sel:BYTE_2\n\t"
        "v_mfma_i32_16x16x64_i8 v[144:147], a[28:31], v[64:67], v[148:151]\n\t"
        "v_or3_b32 v130, v130, v131, v133\n\t"
        "v_xor_b32_sdwa v111, v111, v128 dst_sel:BYTE_3 dst_unused:UNUSED_PRESERVE src0_sel:BYTE_3 src1_sel:BYTE_3\n\t"
        "v_mfma_i32_16x16x64_i8 v[144:147], a[60:63], v[68:71], v[144:147]\n\t"
        "s_waitcnt  vmcnt(5)\n\t"
        "v_bitop3_b32 v134, v123, v127, s60 bitop3:0x7e\n\t"
        "v_bitop3_b32 v130, v130, v134, s90 bitop3:0xa8\n\t"
        "v_mfma_i32_16x16x64_i8 v[144:147], a[92:95], v[72:75], v[144:147]\n\t"
        "v_and_or_b32 v130, v135, s91, v130\n\t"
        "v_cmp_ne_u32 s[94:95], 0, v130\n\t"
        "v_mfma_i32_16x16x64_i8 v[144:147], a[124:127], v[76:79], v[144:147]\n\t"
        "v_xor_b32_sdwa v115, v115, v129 dst_sel:BYTE_3 dst_unused:UNUSED_PRESERVE src0_sel:BYTE_3 src1_sel:BYTE_0\n\t"
        "v_xor_b32_sdwa v119, v119, v129 dst_sel:BYTE_3 dst_unused:UNUSED_PRESERVE src0_sel:BYTE_3 src1_sel:BYTE_1\n\t"
        "v_mfma_i32_16x16x64_i8 v[144:147], a[156:159], v[80:83], v[144:147]\n\t"
        "s_cmp_eq_u64 s[94:95], 0\n\t"
        "ds_add_u64 v25, v[140:141] offset:7280\n\t"
        "v_mfma_i32_16x16x64_i8 v[144:147], a[188:191], v[84:87], v[144:147]\n\t"
        "v_xor_b32_sdwa v123, v123, v129 dst_sel:BYTE_3 dst_unused:UNUSED_PRESERVE src0_sel:BYTE_3 src1_sel:BYTE_2\n\t"
        "v_xor_b32_sdwa v127, v127, v129 dst_sel:BYTE_3 dst_unused:UNUSED_PRESERVE src0_sel:BYTE_3 src1_sel:BYTE_3\n\t"
        "v_mfma_i32_16x16x64_i8 v[144:147], a[220:223], v[88:91], v[144:147]\n\t"
        "ds_add_u64 v25, v[142:143] offset:7792\n\t"
        "v_mfma_i32_16x16x64_i8 v[144:147], a[252:255], v[92:95], v[144:147]\n\t"
        "s_cbranch_scc0 LSERV%=\n\t"
        "LBLKF1%=:\n\t"
        "v_mfma_i32_16x16x64_i8 v[140:143], a[0:3], v[96:99], v[148:151]\n\t"
        "s_mul_i32 s78, s21, 8448\n\t"
        "v_add_u32 v25, 8320, v25\n\t"
        "v_lshl_add_u32 v134, s77, 11, v22\n\t"
        "v_mfma_i32_16x16x64_i8 v[140:143], a[32:35], v[100:103], v[140:143]\n\t"
        "buffer_load_dword v128, v17, s[36:39], s78 offen sc0 sc1\n\t"
        "v_mfma_i32_16x16x64_i8 v[140:143], a[64:67], v[104:107], v[140:143]\n\t"
        "buffer_load_dwordx4 v[64:67], v16, s[36:39], s78 offen offset:0 sc0 sc1\n\t"
        "v_mfma_i32_16x16x64_i8 v[140:143], a[96:99], v[108:111], v[140:143]\n\t"
        "buffer_load_dwordx4 v[68:71], v16, s[36:39], s78 offen offset:1024 sc0 sc1\n\t"
        "v_mfma_i32_16x16x64_i8 v[140:143], a[128:131], v[112:115], v[140:143]\n\t"
        "s_mul_i32 s83, s77, 8320\n\t"
        "ds_add_u64 v25, v[144:145] offset:0\n\t"
        "v_mfma_i32_16x16x64_i8 v[140:143], a[160:163], v[116:119], v[140:143]\n\t"
        "buffer_load_dwordx4 v[72:75], v16, s[36:39], s78 offen offset:2048 sc0 sc1\n\t"
        "v_mfma_i32_16x16x64_i8 v[140:143], a[192:195], v[120:123], v[140:143]\n\t"
        "s_mov_b32 m0, s33\n\t"
        "ds_add_u64 v25, v[146:147] offset:512\n\t"
        "v_mfma_i32_16x16x64_i8 v[140:143], a[224:227], v[124:127], v[140:143]\n\t"
        "buffer_load_dwordx4 v[76:79], v16, s[36:39], s78 offen offset:3072 sc0 sc1\n\t"
        "v_mfma_i32_16x16x64_i8 v[144:147], a[4:7], v[96:99], v[148:151]\n\t"
        "buffer_load_dwordx4 v[80:83], v30, s[36:39], s78 offen offset:0 sc0 sc1\n\t"
        "v_mfma_i32_16x16x64_i8 v[144:147], a[36:39], v[100:103], v[144:147]\n\t"
        "buffer_load_dwordx4 v[84:87], v30, s[36:39], s78 offen offset:1024 sc0 sc1\n\t"
        "v_mfma_i32_16x16x64_i8 v[144:147], a[68:71], v[104:107], v[144:147]\n\t"
        "buffer_load_dwordx4 v[88:91], v30, s[36:39], s78 offen offset:2048 sc0 sc1\n\t"
        "v_mfma_i32_16x16x64_i8 v[144:147], a[100:103], v[108:111], v[144:147]\n\t"
        "buffer_load_dwordx4 v[92:95], v30, s[36:39], s78 offen offset:3072 sc0 sc1\n\t"
        "v_mfma_i32_16x16x64_i8 v[144:147], a[132:135], v[112:115], v[144:147]\n\t"
        "s_cmp_lt_u32 s77, s20\n\t"
        "ds_add_u64 v25, v[140:141] offset:1040\n\t"
        "v_mfma_i32_16x16x64_i8 v[144:147], a[164:167], v[116:119], v[144:147]\n\t"
        "s_cselect_b32 s8, s77, 64\n\t"
        "ds_write_b32 v24, v53\n\t"
        "v_mfma_i32_16x16x64_i8 v[144:147], a[196:199], v[120:123], v[144:147]\n\t"
        "s_lshr_b64 s[28:29], s[28:29], 6\n\t"
        "ds_add_u64 v25, v[142:143] offset:1552\n\t"
        "v_mfma_i32_16x16x64_i8 v[144:147], a[228:231], v[124:127], v[144:147]\n\t"
        "s_and_b32 s21, s28, 63\n\t"
        "v_add_u32 v135, s83, v21\n\t"
        "v_add_u32 v53, 1, v53\n\t"
        "v_mfma_i32_16x16x64_i8 v[140:143], a[8:11], v[96:99], v[148:151]\n\t"
        "s_waitcnt  vmcnt(12)\n\t"
        "ds_read_b128 v[160:163], v134\n\t"
        "ds_read_b128 v[164:167], v134 offset:1024\n\t"
        "v_mfma_i32_16x16x64_i8 v[140:143], a[40:43], v[100:103], v[140:143]\n\t"
        "s_lshl3_add_u32 s84, s77, s63\n\t"
        "ds_read_b32 v132, v23\n\t"
        "ds_read2_b64 v[152:155], v135 offset0:0 offset1:64\n\t"
        "v_mfma_i32_16x16x64_i8 v[140:143], a[72:75], v[104:107], v[140:143]\n\t"
        "s_bfe_u64 s[84:85], s[30:31], s84\n\t"
        "ds_read2_b64 v[156:159], v135 offset0:2 offset1:66\n\t"
        "v_mfma_i32_16x16x64_i8 v[140:143], a[104:107], v[108:111], v[140:143]\n\t"
        "s_lshl_b32 s80, s84, 18\n\t"
        "v_mfma_i32_16x16x64_i8 v[140:143], a[136:139], v[112:115], v[140:143]\n\t"
        "s_add_u32 s76, s76, 1\n\t"
        "ds_add_u64 v25, v[144:145] offset:2080\n\t"
        "v_mfma_i32_16x16x64_i8 v[140:143], a[168:171], v[116:119], v[140:143]\n\t"
        "s_waitcnt lgkmcnt(4)\n\t"
        "v_bitop3_b32 v133, v160, v162, s62 bitop3:0x7e\n\t"
        "v_bitop3_b32 v134, v164, v166, s62 bitop3:0x7e\n\t"
        "v_mfma_i32_16x16x64_i8 v[140:143], a[200:203], v[120:123], v[140:143]\n\t"
        "ds_add_u64 v25, v[146:147] offset:2592\n\t"
        "v_mfma_i32_16x16x64_i8 v[140:143], a[232:235], v[124:127], v[140:143]\n\t"
        "v_or_b32 v133, v133, v134\n\t"
        "v_cmp_le_u32 s[92:93], s87, v133\n\t"
        "v_mfma_i32_16x16x64_i8 v[144:147], a[12:15], v[96:99], v[148:151]\n\t"
        "s_waitcnt lgkmcnt(2)\n\t"
        "v_cmp_ge_i32 vcc, s8, v132\n\t"
        "v_add3_u32 v160, v152, v160, s88\n\t"
        "v_mfma_i32_16x16x64_i8 v[144:147], a[44:47], v[100:103], v[144:147]\n\t"
        "v_add_u32 v161, v161, v153\n\t"
        "v_add3_u32 v162, v154, v162, s88\n\t"
        "v_mfma_i32_16x16x64_i8 v[144:147], a[76:79], v[104:107], v[144:147]\n\t"
        "v_add_u32 v163, v163, v155\n\t"
        "v_cvt_f32_i32 v168, v160\n\t"
        "v_mfma_i32_16x16x64_i8 v[144:147], a[108:111], v[108:111], v[144:147]\n\t"
        "v_cvt_f32_i32 v169, v161\n\t"
        "v_cvt_f32_i32 v170, v162\n\t"
        "v_mfma_i32_16x16x64_i8 v[144:147], a[140:143], v[112:115], v[144:147]\n\t"
        "ds_add_u64 v25, v[140:141] offset:3120\n\t"
        "v_mfma_i32_16x16x64_i8 v[144:147], a[172:175], v[116:119], v[144:147]\n\t"
        "s_waitcnt   vmcnt(9)\n\t"
        "v_xor_b32 v55, s62, v55\n\t"
        "v_cmp_gt_u32 s[94:95], s87, v55\n\t"
        "v_mfma_i32_16x16x64_i8 v[144:147], a[204:207], v[120:123], v[144:147]\n\t"
        "ds_add_u64 v25, v[142:143] offset:3632\n\t"
        "v_mfma_i32_16x16x64_i8 v[144:147], a[236:239], v[124:127], v[144:147]\n\t"
        "s_cmp_lg_u64 s[94:95], 0\n\t"
        "s_cselect_b32 s9, s89, 0x40000000\n\t"
        "s_cselect_b32 s10, 2048, 0\n\t"
        "v_mfma_i32_16x16x64_i8 v[140:143], a[16:19], v[96:99], v[148:151]\n\t"
        "buffer_load_dwordx4 v18, s[40:43], s9 offen sc0 sc1 lds\n\t"
        "v_mfma_i32_16x16x64_i8 v[140:143], a[48:51], v[100:103], v[140:143]\n\t"
        "buffer_load_dwordx4 v18, s[40:43], s9 offen offset:1024 sc0 sc1 lds\n\t"
        "v_mfma_i32_16x16x64_i8 v[140:143], a[80:83], v[104:107], v[140:143]\n\t"
        "s_addc_u32 s20, s20, 0\n\t"
        "v_cvt_f32_i32 v171, v163\n\t"
        "v_mul_f32 v168, v52, v168\n\t"
        "v_mfma_i32_16x16x64_i8 v[140:143], a[112:115], v[108:111], v[140:143]\n\t"
        "s_add_u32 s33, s33, s10\n\t"
        "v_mul_f32 v169, v52, v169\n\t"
        "v_mul_f32 v170, v52, v170\n\t"
        "v_mfma_i32_16x16x64_i8 v[140:143], a[144:147], v[112:115], v[140:143]\n\t"
        "s_lshl2_add_u32 s89, s10, s89\n\t"
        "ds_add_u64 v25, v[144:145] offset:4160\n\t"
        "v_mfma_i32_16x16x64_i8 v[140:143], a[176:179], v[116:119], v[140:143]\n\t"
        "s_or_b64 s[92:93], s[92:93], vcc\n\t"
        "v_mul_f32 v171, v52, v171\n\t"
        "v_add3_u32 v164, v156, v164, s88\n\t"
        "v_mfma_i32_16x16x64_i8 v[140:143], a[208:211], v[120:123], v[140:143]\n\t"
        "s_cselect_b32 s81, 0x40000000, s80\n\t"
        "ds_add_u64 v25, v[146:147] offset:4672\n\t"
        "v_mfma_i32_16x16x64_i8 v[140:143], a[240:243], v[124:127], v[140:143]\n\t"
        "s_subb_u32 s77, s77, -1\n\t"
        "v_add_u32 v165, v165, v157\n\t"
        "v_add3_u32 v166, v158, v166, s88\n\t"
        "v_mfma_i32_16x16x64_i8 v[144:147], a[20:23], v[96:99], v[148:151]\n\t"
        "buffer_store_dwordx4 v[168:171], v19, s[48:51], s81 offen sc0 sc1\n\t"
        "v_add_u32 v167, v167, v159\n\t"
        "v_cvt_f32_i32 v172, v164\n\t"
        "v_cvt_f32_i32 v173, v165\n\t"
        "v_mfma_i32_16x16x64_i8 v[144:147], a[52:55], v[100:103], v[144:147]\n\t"
        "v_cvt_f32_i32 v174, v166\n\t"
        "v_cvt_f32_i32 v175, v167\n\t"
        "v_mfma_i32_16x16x64_i8 v[144:147], a[84:87], v[104:107], v[144:147]\n\t"
        "v_mul_f32 v172, v52, v172\n\t"
        "v_mul_f32 v173, v52, v173\n\t"
        "v_mfma_i32_16x16x64_i8 v[144:147], a[116:119], v[108:111], v[144:147]\n\t"
        "v_mul_f32 v174, v52, v174\n\t"
        "v_mul_f32 v175, v52, v175\n\t"
        "v_mfma_i32_16x16x64_i8 v[144:147], a[148:151], v[112:115], v[144:147]\n\t"
        "buffer_store_dwordx4 v[172:175], v20, s[48:51], s81 offen sc0 sc1\n\t"
        "ds_add_u64 v25, v[140:141] offset:5200\n\t"
        "v_mfma_i32_16x16x64_i8 v[144:147], a[180:183], v[116:119], v[144:147]\n\t"
        "buffer_load_dword v55, v56, s[40:43], s89 offen sc0 sc1\n\t"
        "v_mfma_i32_16x16x64_i8 v[144:147], a[212:215], v[120:123], v[144:147]\n\t"
        "ds_add_u64 v25, v[142:143] offset:5712\n\t"
        "v_mfma_i32_16x16x64_i8 v[144:147], a[244:247], v[124:127], v[144:147]\n\t"
        "v_mfma_i32_16x16x64_i8 v[140:143], a[24:27], v[96:99], v[148:151]\n\t"
        "v_mfma_i32_16x16x64_i8 v[140:143], a[56:59], v[100:103], v[140:143]\n\t"
        "v_mfma_i32_16x16x64_i8 v[140:143], a[88:91], v[104:107], v[140:143]\n\t"
        "s_waitcnt vmcnt(9)\n\t"
        "v_bitop3_b32 v130, v67, v71, s60 bitop3:0x7e\n\t"
        "v_bitop3_b32 v131, v75, v79, s60 bitop3:0x7e\n\t"
        "v_mfma_i32_16x16x64_i8 v[140:143], a[120:123], v[108:111], v[140:143]\n\t"
        "v_xor_b32 v135, s61, v128\n\t"
        "v_lshlrev_b32 v129, 3, v128\n\t"
        "v_mfma_i32_16x16x64_i8 v[140:143], a[152:155], v[112:115], v[140:143]\n\t"
        "ds_add_u64 v25, v[144:145] offset:6240\n\t"
        "v_mfma_i32_16x16x64_i8 v[140:143], a[184:187], v[116:119], v[140:143]\n\t"
        "v_xor_b32_sdwa v67, v67, v128 dst_sel:BYTE_3 dst_unused:UNUSED_PRESERVE src0_sel:BYTE_3 src1_sel:BYTE_0\n\t"
        "v_xor_b32_sdwa v71, v71, v128 dst_sel:BYTE_3 dst_unused:UNUSED_PRESERVE src0_sel:BYTE_3 src1_sel:BYTE_1\n\t"
        "v_mfma_i32_16x16x64_i8 v[140:143], a[216:219], v[120:123], v[140:143]\n\t"
        "ds_add_u64 v25, v[146:147] offset:6752\n\t"
        "v_mfma_i32_16x16x64_i8 v[140:143], a[248:251], v[124:127], v[140:143]\n\t"
        "s_waitcnt  vmcnt(7)\n\t"
        "v_bitop3_b32 v133, v83, v87, s60 bitop3:0x7e\n\t"
        "v_xor_b32_sdwa v75, v75, v128 dst_sel:BYTE_3 dst_unused:UNUSED_PRESERVE src0_sel:BYTE_3 src1_sel:BYTE_2\n\t"
        "v_mfma_i32_16x16x64_i8 v[144:147], a[28:31], v[96:99], v[148:151]\n\t"
        "v_or3_b32 v130, v130, v131, v133\n\t"
        "v_xor_b32_sdwa v79, v79, v128 dst_sel:BYTE_3 dst_unused:UNUSED_PRESERVE src0_sel:BYTE_3 src1_sel:BYTE_3\n\t"
        "v_mfma_i32_16x16x64_i8 v[144:147], a[60:63], v[100:103], v[144:147]\n\t"
        "s_waitcnt  vmcnt(5)\n\t"
        "v_bitop3_b32 v134, v91, v95, s60 bitop3:0x7e\n\t"
        "v_bitop3_b32 v130, v130, v134, s90 bitop3:0xa8\n\t"
        "v_mfma_i32_16x16x64_i8 v[144:147], a[92:95], v[104:107], v[144:147]\n\t"
        "v_and_or_b32 v130, v135, s91, v130\n\t"
        "v_cmp_ne_u32 s[94:95], 0, v130\n\t"
        "v_mfma_i32_16x16x64_i8 v[144:147], a[124:127], v[108:111], v[144:147]\n\t"
        "v_xor_b32_sdwa v83, v83, v129 dst_sel:BYTE_3 dst_unused:UNUSED_PRESERVE src0_sel:BYTE_3 src1_sel:BYTE_0\n\t"
        "v_xor_b32_sdwa v87, v87, v129 dst_sel:BYTE_3 dst_unused:UNUSED_PRESERVE src0_sel:BYTE_3 src1_sel:BYTE_1\n\t"
        "v_mfma_i32_16x16x64_i8 v[144:147], a[156:159], v[112:115], v[144:147]\n\t"
        "s_cmp_eq_u64 s[94:95], 0\n\t"
        "ds_add_u64 v25, v[140:141] offset:7280\n\t"
        "v_mfma_i32_16x16x64_i8 v[144:147], a[188:191], v[116:119], v[144:147]\n\t"
        "v_xor_b32_sdwa v91, v91, v129 dst_sel:BYTE_3 dst_unused:UNUSED_PRESERVE src0_sel:BYTE_3 src1_sel:BYTE_2\n\t"
        "v_xor_b32_sdwa v95, v95, v129 dst_sel:BYTE_3 dst_unused:UNUSED_PRESERVE src0_sel:BYTE_3 src1_sel:BYTE_3\n\t"
        "v_mfma_i32_16x16x64_i8 v[144:147], a[220:223], v[120:123], v[144:147]\n\t"
        "ds_add_u64 v25, v[142:143] offset:7792\n\t"
        "v_mfma_i32_16x16x64_i8 v[144:147], a[252:255], v[124:127], v[144:147]\n\t"
        "s_cbranch_scc1 LBLKF0%=\n\t"
        "s_branch LSERV%=\n\t"
        "LEND%=:\n\t"
        "s_branch LEXIT%=\n\t"
        "LSM%=:\n\t"
        "s_and_b32 s27, s2, 7\n\t"
        "s_lshl2_add_u32 s26, s27, s26\n\t"
        "s_lshl_b32 s27, s20, 5\n\t"
        "s_add_u32 s26, s26, s27\n\t"
        "s_mul_i32 s81, s26, 8448\n\t"
        "s_lshl_b32 s26, s26, 13\n\t"
        "s_mov_b32 s56, s8\n\t"
        "s_and_b32 s57, s9, 0xffff\n\t"
        "s_mov_b32 s58, 2097152\n\t"
        "s_mov_b32 s59, 0x20000\n\t"
        "s_add_u32 m0, s64, 8192\n\t"
        "v_lshrrev_b32 v5, 5, v1\n\t"
        "v_xor_b32 v6, v5, v1\n\t"
        "v_and_b32 v6, 31, v6\n\t"
        "v_lshlrev_b32 v5, 9, v5\n\t"
        "v_lshl_or_b32 v32, v6, 4, v5\n\t"
        "v_xor_b32 v33, 32, v32\n\t"
        "v_xor_b32 v34, 64, v32\n\t"
        "v_xor_b32 v35, 96, v32\n\t"
        "v_xor_b32 v36, 128, v32\n\t"
        "v_xor_b32 v37, 160, v32\n\t"
        "v_xor_b32 v38, 192, v32\n\t"
        "v_xor_b32 v39, 224, v32\n\t"
        "buffer_load_dwordx4 v32, s[56:59], s26 offen offset:0 lds\n\t"
        "buffer_load_dwordx4 v33, s[56:59], s26 offen offset:1024 lds\n\t"
        "buffer_load_dwordx4 v34, s[56:59], s26 offen offset:2048 lds\n\t"
        "buffer_load_dwordx4 v35, s[56:59], s26 offen offset:3072 lds\n\t"
        "s_add_u32 m0, s64, 12288\n\t"
        "s_add_u32 s26, s26, 4096\n\t"
        "buffer_load_dwordx4 v36, s[56:59], s26 offen offset:0 lds\n\t"
        "buffer_load_dwordx4 v37, s[56:59], s26 offen offset:1024 lds\n\t"
        "buffer_load_dwordx4 v38, s[56:59], s26 offen offset:2048 lds\n\t"
        "buffer_load_dwordx4 v39, s[56:59], s26 offen offset:3072 lds\n\t"
        "v_lshl_or_b32 v5, v2, 2, v2\n\t"
        "v_and_b32 v5, 6, v5\n\t"
        "v_xor_b32 v5, v3, v5\n\t"
        "v_lshlrev_b32 v5, 4, v5\n\t"
        "v_lshl_or_b32 v5, v3, 9, v5\n\t"
        "v_add_u32 v40, s64, v5\n\t"
        "v_xor_b32 v41, 16, v40\n\t"
        "v_xor_b32 v42, 128, v40\n\t"
        "v_xor_b32 v43, 144, v40\n\t"
        "v_mov_b32 v15, 0xe0000000\n\t"
        "s_add_u32 s56, s12, 1048576\n\t"
        "s_addc_u32 s57, s13, 0\n\t"
        "s_and_b32 s57, s57, 0xffff\n\t"
        "s_mov_b32 s58, 4325376\n\t"
        "s_mov_b32 m0, s64\n\t"
        "s_branch LSM1%=\n\t"
        "LSM2%=:\n\t"
        "s_add_u32 s81, s81, s27\n\t"
        "v_mov_b32 v5, s60\n\t"
        "s_waitcnt vmcnt(8)\n\t"
        "ds_read_b128 v[64:67], v40 offset:8192\n\t"
        "ds_read_b128 v[68:71], v41 offset:8192\n\t"
        "ds_read_b128 v[72:75], v42 offset:8192\n\t"
        "ds_read_b128 v[76:79], v43 offset:8192\n\t"
        "ds_read_b128 v[80:83], v40 offset:8448\n\t"
        "ds_read_b128 v[84:87], v41 offset:8448\n\t"
        "ds_read_b128 v[88:91], v42 offset:8448\n\t"
        "ds_read_b128 v[92:95], v43 offset:8448\n\t"
        "v_mov_b32 v58, s61\n\t"
        "s_waitcnt lgkmcnt(7)\n\t"
        "v_bitop3_b32 v8, v67, v5, v15 bitop3:0x28\n\t"
        "v_lshrrev_b32 v8, 24, v8\n\t"
        "s_waitcnt lgkmcnt(6)\n\t"
        "v_bitop3_b32 v9, v71, v5, v15 bitop3:0x28\n\t"
        "v_lshrrev_b32 v9, 16, v9\n\t"
        "v_or3_b32 v58, v8, v9, v58\n\t"
        "s_waitcnt lgkmcnt(5)\n\t"
        "v_bitop3_b32 v8, v75, v5, v15 bitop3:0x28\n\t"
        "v_lshrrev_b32 v8, 8, v8\n\t"
        "s_waitcnt lgkmcnt(4)\n\t"
        "v_bitop3_b32 v9, v79, v5, v15 bitop3:0x28\n\t"
        "v_or3_b32 v58, v8, v9, v58\n\t"
        "s_waitcnt lgkmcnt(3)\n\t"
        "v_bitop3_b32 v8, v83, v5, v15 bitop3:0x28\n\t"
        "v_lshrrev_b32 v8, 27, v8\n\t"
        "s_waitcnt lgkmcnt(2)\n\t"
        "v_bitop3_b32 v9, v87, v5, v15 bitop3:0x28\n\t"
        "v_lshrrev_b32 v9, 19, v9\n\t"
        "v_or3_b32 v58, v8, v9, v58\n\t"
        "s_waitcnt lgkmcnt(1)\n\t"
        "v_bitop3_b32 v8, v91, v5, v15 bitop3:0x28\n\t"
        "v_lshrrev_b32 v8, 11, v8\n\t"
        "s_waitcnt lgkmcnt(0)\n\t"
        "v_bitop3_b32 v9, v95, v5, v15 bitop3:0x28\n\t"
        "v_lshrrev_b32 v9, 3, v9\n\t"
        "v_or3_b32 v58, v8, v9, v58\n\t"
        "v_lshlrev_b32 v8, 3, v58\n\t"
        "v_xor_b32_sdwa v67, v67, v58 dst_sel:BYTE_3 dst_unused:UNUSED_PRESERVE src0_sel:BYTE_3 src1_sel:BYTE_0\n\t"
        "v_xor_b32_sdwa v71, v71, v58 dst_sel:BYTE_3 dst_unused:UNUSED_PRESERVE src0_sel:BYTE_3 src1_sel:BYTE_1\n\t"
        "buffer_store_dwordx4 v[64:67], v16, s[56:59], s81 offen offset:0 sc0 sc1\n\t"
        "v_xor_b32_sdwa v75, v75, v58 dst_sel:BYTE_3 dst_unused:UNUSED_PRESERVE src0_sel:BYTE_3 src1_sel:BYTE_2\n\t"
        "buffer_store_dwordx4 v[68:71], v16, s[56:59], s81 offen offset:1024 sc0 sc1\n\t"
        "v_xor_b32_sdwa v79, v79, v58 dst_sel:BYTE_3 dst_unused:UNUSED_PRESERVE src0_sel:BYTE_3 src1_sel:BYTE_3\n\t"
        "buffer_store_dwordx4 v[72:75], v16, s[56:59], s81 offen offset:2048 sc0 sc1\n\t"
        "v_xor_b32_sdwa v83, v83, v8 dst_sel:BYTE_3 dst_unused:UNUSED_PRESERVE src0_sel:BYTE_3 src1_sel:BYTE_0\n\t"
        "buffer_store_dwordx4 v[76:79], v16, s[56:59], s81 offen offset:3072 sc0 sc1\n\t"
        "v_xor_b32_sdwa v87, v87, v8 dst_sel:BYTE_3 dst_unused:UNUSED_PRESERVE src0_sel:BYTE_3 src1_sel:BYTE_1\n\t"
        "buffer_store_dwordx4 v[80:83], v30, s[56:59], s81 offen offset:0 sc0 sc1\n\t"
        "v_xor_b32_sdwa v91, v91, v8 dst_sel:BYTE_3 dst_unused:UNUSED_PRESERVE src0_sel:BYTE_3 src1_sel:BYTE_2\n\t"
        "buffer_store_dwordx4 v[84:87], v30, s[56:59], s81 offen offset:1024 sc0 sc1\n\t"
        "v_xor_b32_sdwa v95, v95, v8 dst_sel:BYTE_3 dst_unused:UNUSED_PRESERVE src0_sel:BYTE_3 src1_sel:BYTE_3\n\t"
        "buffer_store_dwordx4 v[88:91], v30, s[56:59], s81 offen offset:2048 sc0 sc1\n\t"
        "s_nop 0\n\t"
        "buffer_store_dwordx4 v[92:95], v30, s[56:59], s81 offen offset:3072 sc0 sc1\n\t"
        "buffer_store_dword v58, v17, s[56:59], s81 offen sc0 sc1\n\t"
        "s_branch LSM3%=\n\t"
        "LXQ2_%=:\n\t"
        "s_swappc_b64 s[96:97], s[98:99]\n\t"
        "s_branch LYQ2_%=\n\t"
        "LXQ3_%=:\n\t"
        "s_swappc_b64 s[96:97], s[98:99]\n\t"
        "s_branch LYQ3_%=\n\t"
        "LXQ4_%=:\n\t"
        "s_swappc_b64 s[96:97], s[98:99]\n\t"
        "s_branch LYQ4_%=\n\t"
        "LXQ5_%=:\n\t"
        "s_swappc_b64 s[96:97], s[98:99]\n\t"
        "s_branch LYQ5_%=\n\t"
        "LXQ6_%=:\n\t"
        "s_swappc_b64 s[96:97], s[98:99]\n\t"
        "s_branch LYQ6_%=\n\t"
        "LXQ7_%=:\n\t"
        "s_swappc_b64 s[96:97], s[98:99]\n\t"
        "s_branch LYQ7_%=\n\t"
        "LXQ8_%=:\n\t"
        "s_swappc_b64 s[96:97], s[98:99]\n\t"
        "s_branch LYQ8_%=\n\t"
        "LXQ9_%=:\n\t"
        "s_swappc_b64 s[96:97], s[98:99]\n\t"
        "s_branch LYQ9_%=\n\t"
        "LXQ10_%=:\n\t"
        "s_swappc_b64 s[96:97], s[98:99]\n\t"
        "s_branch LYQ10_%=\n\t"
        "LXQ11_%=:\n\t"
        "s_swappc_b64 s[96:97], s[98:99]\n\t"
        "s_branch LYQ11_%=\n\t"
        "LXQ12_%=:\n\t"
        "s_swappc_b64 s[96:97], s[98:99]\n\t"
        "s_branch LYQ12_%=\n\t"
        "LXQ13_%=:\n\t"
        "s_swappc_b64 s[96:97], s[98:99]\n\t"
        "s_branch LYQ13_%=\n\t"
        "LXQ14_%=:\n\t"
        "s_swappc_b64 s[96:97], s[98:99]\n\t"
        "s_branch LYQ14_%=\n\t"
        "LXQ15_%=:\n\t"
        "s_swappc_b64 s[96:97], s[98:99]\n\t"
        "s_branch LYQ15_%=\n\t"
        "LXR13_%=:\n\t"
        "s_swappc_b64 s[96:97], s[98:99]\n\t"
        "s_branch LYR13_%=\n\t"
        "LXR14_%=:\n\t"
        "s_swappc_b64 s[96:97], s[98:99]\n\t"
        "s_branch LYR14_%=\n\t"
        "LXR15_%=:\n\t"
        "s_swappc_b64 s[96:97], s[98:99]\n\t"
        "s_branch LYR15_%=\n\t"
        "LBLKL0%=:\n\t"
        "v_mfma_i32_16x16x64_i8 v[140:143], a[0:3], v[64:67], v[148:151]\n\t"
        "v_add_u32 v25, 8320, v25\n\t"
        "v_add_u32 v53, 1, v53\n\t"
        "v_mfma_i32_16x16x64_i8 v[140:143], a[32:35], v[68:71], v[140:143]\n\t"
        "s_mul_i32 s78, s21, 8448\n\t"
        "v_mfma_i32_16x16x64_i8 v[140:143], a[64:67], v[72:75], v[140:143]\n\t"
        "s_lshr_b64 s[28:29], s[28:29], 6\n\t"
        "v_mfma_i32_16x16x64_i8 v[140:143], a[96:99], v[76:79], v[140:143]\n\t"
        "s_and_b32 s21, s28, 63\n\t"
        "v_mfma_i32_16x16x64_i8 v[140:143], a[128:131], v[80:83], v[140:143]\n\t"
        "s_add_u32 s76, s76, 1\n\t"
        "v_mfma_i32_16x16x64_i8 v[140:143], a[160:163], v[84:87], v[140:143]\n\t"
        "s_cmp_ge_u32 s77, s34\n\t"
        "v_mfma_i32_16x16x64_i8 v[140:143], a[192:195], v[88:91], v[140:143]\n\t"
        "s_cselect_b64 vcc, -1, 0\n\t"
        "v_mfma_i32_16x16x64_i8 v[140:143], a[224:227], v[92:95], v[140:143]\n\t"
        "v_mfma_i32_16x16x64_i8 v[144:147], a[4:7], v[64:67], v[148:151]\n\t"
        "v_mfma_i32_16x16x64_i8 v[144:147], a[36:39], v[68:71], v[144:147]\n\t"
        "v_mfma_i32_16x16x64_i8 v[144:147], a[68:71], v[72:75], v[144:147]\n\t"
        "v_mfma_i32_16x16x64_i8 v[144:147], a[100:103], v[76:79], v[144:147]\n\t"
        "v_mfma_i32_16x16x64_i8 v[144:147], a[132:135], v[80:83], v[144:147]\n\t"
        "ds_add_u64 v25, v[140:141] offset:1040\n\t"
        "v_mfma_i32_16x16x64_i8 v[144:147], a[164:167], v[84:87], v[144:147]\n\t"
        "v_mfma_i32_16x16x64_i8 v[144:147], a[196:199], v[88:91], v[144:147]\n\t"
        "ds_add_u64 v25, v[142:143] offset:1552\n\t"
        "v_mfma_i32_16x16x64_i8 v[144:147], a[228:231], v[92:95], v[144:147]\n\t"
        "buffer_load_dword v128, v17, s[36:39], s78 offen sc0 sc1\n\t"
        "v_mfma_i32_16x16x64_i8 v[140:143], a[8:11], v[64:67], v[148:151]\n\t"
        "buffer_load_dwordx4 v[96:99], v16, s[36:39], s78 offen offset:0 sc0 sc1\n\t"
        "v_mfma_i32_16x16x64_i8 v[140:143], a[40:43], v[68:71], v[140:143]\n\t"
        "buffer_load_dwordx4 v[100:103], v16, s[36:39], s78 offen offset:1024 sc0 sc1\n\t"
        "v_mfma_i32_16x16x64_i8 v[140:143], a[72:75], v[72:75], v[140:143]\n\t"
        "buffer_load_dwordx4 v[104:107], v16, s[36:39], s78 offen offset:2048 sc0 sc1\n\t"
        "v_mfma_i32_16x16x64_i8 v[140:143], a[104:107], v[76:79], v[140:143]\n\t"
        "buffer_load_dwordx4 v[108:111], v16, s[36:39], s78 offen offset:3072 sc0 sc1\n\t"
        "v_mfma_i32_16x16x64_i8 v[140:143], a[136:139], v[80:83], v[140:143]\n\t"
        "ds_add_u64 v25, v[144:145] offset:2080\n\t"
        "v_mfma_i32_16x16x64_i8 v[140:143], a[168:171], v[84:87], v[140:143]\n\t"
        "buffer_load_dwordx4 v[112:115], v30, s[36:39], s78 offen offset:0 sc0 sc1\n\t"
        "v_mfma_i32_16x16x64_i8 v[140:143], a[200:203], v[88:91], v[140:143]\n\t"
        "ds_add_u64 v25, v[146:147] offset:2592\n\t"
        "v_mfma_i32_16x16x64_i8 v[140:143], a[232:235], v[92:95], v[140:143]\n\t"
        "buffer_load_dwordx4 v[116:119], v30, s[36:39], s78 offen offset:1024 sc0 sc1\n\t"
        "v_mfma_i32_16x16x64_i8 v[144:147], a[12:15], v[64:67], v[148:151]\n\t"
        "buffer_load_dwordx4 v[120:123], v30, s[36:39], s78 offen offset:2048 sc0 sc1\n\t"
        "v_mfma_i32_16x16x64_i8 v[144:147], a[44:47], v[68:71], v[144:147]\n\t"
        "buffer_load_dwordx4 v[124:127], v30, s[36:39], s78 offen offset:3072 sc0 sc1\n\t"
        "v_mfma_i32_16x16x64_i8 v[144:147], a[76:79], v[72:75], v[144:147]\n\t"
        "v_mfma_i32_16x16x64_i8 v[144:147], a[108:111], v[76:79], v[144:147]\n\t"
        "v_mfma_i32_16x16x64_i8 v[144:147], a[140:143], v[80:83], v[144:147]\n\t"
        "ds_add_u64 v25, v[140:141] offset:3120\n\t"
        "v_mfma_i32_16x16x64_i8 v[144:147], a[172:175], v[84:87], v[144:147]\n\t"
        "v_mfma_i32_16x16x64_i8 v[144:147], a[204:207], v[88:91], v[144:147]\n\t"
        "ds_add_u64 v25, v[142:143] offset:3632\n\t"
        "v_mfma_i32_16x16x64_i8 v[144:147], a[236:239], v[92:95], v[144:147]\n\t"
        "v_mfma_i32_16x16x64_i8 v[140:143], a[16:19], v[64:67], v[148:151]\n\t"
        "v_mfma_i32_16x16x64_i8 v[140:143], a[48:51], v[68:71], v[140:143]\n\t"
        "v_mfma_i32_16x16x64_i8 v[140:143], a[80:83], v[72:75], v[140:143]\n\t"
        "v_mfma_i32_16x16x64_i8 v[140:143], a[112:115], v[76:79], v[140:143]\n\t"
        "v_mfma_i32_16x16x64_i8 v[140:143], a[144:147], v[80:83], v[140:143]\n\t"
        "ds_add_u64 v25, v[144:145] offset:4160\n\t"
        "v_mfma_i32_16x16x64_i8 v[140:143], a[176:179], v[84:87], v[140:143]\n\t"
        "v_mfma_i32_16x16x64_i8 v[140:143], a[208:211], v[88:91], v[140:143]\n\t"
        "ds_add_u64 v25, v[146:147] offset:4672\n\t"
        "v_mfma_i32_16x16x64_i8 v[140:143], a[240:243], v[92:95], v[140:143]\n\t"
        "v_mfma_i32_16x16x64_i8 v[144:147], a[20:23], v[64:67], v[148:151]\n\t"
        "v_mfma_i32_16x16x64_i8 v[144:147], a[52:55], v[68:71], v[144:147]\n\t"
        "v_mfma_i32_16x16x64_i8 v[144:147], a[84:87], v[72:75], v[144:147]\n\t"
        "v_mfma_i32_16x16x64_i8 v[144:147], a[116:119], v[76:79], v[144:147]\n\t"
        "v_mfma_i32_16x16x64_i8 v[144:147], a[148:151], v[80:83], v[144:147]\n\t"
        "ds_add_u64 v25, v[140:141] offset:5200\n\t"
        "v_mfma_i32_16x16x64_i8 v[144:147], a[180:183], v[84:87], v[144:147]\n\t"
        "v_mfma_i32_16x16x64_i8 v[144:147], a[212:215], v[88:91], v[144:147]\n\t"
        "ds_add_u64 v25, v[142:143] offset:5712\n\t"
        "v_mfma_i32_16x16x64_i8 v[144:147], a[244:247], v[92:95], v[144:147]\n\t"
        "buffer_load_dword v55, v56, s[40:43], s89 offen sc0 sc1\n\t"
        "v_mfma_i32_16x16x64_i8 v[140:143], a[24:27], v[64:67], v[148:151]\n\t"
        "v_mfma_i32_16x16x64_i8 v[140:143], a[56:59], v[68:71], v[140:143]\n\t"
        "v_mfma_i32_16x16x64_i8 v[140:143], a[88:91], v[72:75], v[140:143]\n\t"
        "v_mfma_i32_16x16x64_i8 v[140:143], a[120:123], v[76:79], v[140:143]\n\t"
        "v_mfma_i32_16x16x64_i8 v[140:143], a[152:155], v[80:83], v[140:143]\n\t"
        "ds_add_u64 v25, v[144:145] offset:6240\n\t"
        "v_mfma_i32_16x16x64_i8 v[140:143], a[184:187], v[84:87], v[140:143]\n\t"
        "v_mfma_i32_16x16x64_i8 v[140:143], a[216:219], v[88:91], v[140:143]\n\t"
        "ds_add_u64 v25, v[146:147] offset:6752\n\t"
        "v_mfma_i32_16x16x64_i8 v[140:143], a[248:251], v[92:95], v[140:143]\n\t"
        "s_waitcnt vmcnt(5)\n\t"
        "v_bitop3_b32 v130, v99, v103, s60 bitop3:0x7e\n\t"
        "v_bitop3_b32 v131, v107, v111, s60 bitop3:0x7e\n\t"
        "v_mfma_i32_16x16x64_i8 v[144:147], a[28:31], v[64:67], v[148:151]\n\t"
        "v_xor_b32 v135, s61, v128\n\t"
        "v_lshlrev_b32 v129, 3, v128\n\t"
        "v_mfma_i32_16x16x64_i8 v[144:147], a[60:63], v[68:71], v[144:147]\n\t"
        "v_xor_b32_sdwa v99, v99, v128 dst_sel:BYTE_3 dst_unused:UNUSED_PRESERVE src0_sel:BYTE_3 src1_sel:BYTE_0\n\t"
        "v_xor_b32_sdwa v103, v103, v128 dst_sel:BYTE_3 dst_unused:UNUSED_PRESERVE src0_sel:BYTE_3 src1_sel:BYTE_1\n\t"
        "v_mfma_i32_16x16x64_i8 v[144:147], a[92:95], v[72:75], v[144:147]\n\t"
        "v_xor_b32_sdwa v107, v107, v128 dst_sel:BYTE_3 dst_unused:UNUSED_PRESERVE src0_sel:BYTE_3 src1_sel:BYTE_2\n\t"
        "v_xor_b32_sdwa v111, v111, v128 dst_sel:BYTE_3 dst_unused:UNUSED_PRESERVE src0_sel:BYTE_3 src1_sel:BYTE_3\n\t"
        "v_mfma_i32_16x16x64_i8 v[144:147], a[124:127], v[76:79], v[144:147]\n\t"
        "s_waitcnt vmcnt(1)\n\t"
        "v_bitop3_b32 v133, v115, v119, s60 bitop3:0x7e\n\t"
        "v_bitop3_b32 v134, v123, v127, s60 bitop3:0x7e\n\t"
        "v_mfma_i32_16x16x64_i8 v[144:147], a[156:159], v[80:83], v[144:147]\n\t"
        "ds_add_u64 v25, v[140:141] offset:7280\n\t"
        "v_mfma_i32_16x16x64_i8 v[144:147], a[188:191], v[84:87], v[144:147]\n\t"
        "v_or3_b32 v130, v130, v131, v133\n\t"
        "v_bitop3_b32 v130, v130, v134, s90 bitop3:0xa8\n\t"
        "v_mfma_i32_16x16x64_i8 v[144:147], a[220:223], v[88:91], v[144:147]\n\t"
        "ds_add_u64 v25, v[142:143] offset:7792\n\t"
        "v_mfma_i32_16x16x64_i8 v[144:147], a[252:255], v[92:95], v[144:147]\n\t"
        "v_and_or_b32 v130, v135, s91, v130\n\t"
        "v_cmp_ne_u32 s[94:95], 0, v130\n\t"
        "v_xor_b32_sdwa v115, v115, v129 dst_sel:BYTE_3 dst_unused:UNUSED_PRESERVE src0_sel:BYTE_3 src1_sel:BYTE_0\n\t"
        "v_xor_b32_sdwa v119, v119, v129 dst_sel:BYTE_3 dst_unused:UNUSED_PRESERVE src0_sel:BYTE_3 src1_sel:BYTE_1\n\t"
        "v_xor_b32_sdwa v123, v123, v129 dst_sel:BYTE_3 dst_unused:UNUSED_PRESERVE src0_sel:BYTE_3 src1_sel:BYTE_2\n\t"
        "v_xor_b32_sdwa v127, v127, v129 dst_sel:BYTE_3 dst_unused:UNUSED_PRESERVE src0_sel:BYTE_3 src1_sel:BYTE_3\n\t"
        "s_cmp_eq_u64 s[94:95], 0\n\t"
        "s_cbranch_scc0 LSERV%=\n\t"
        "s_cbranch_vccnz LBLKF1%=\n\t"
        "s_branch LBLKP1%=\n\t"
        "LEXIT%=:\n\t"
        :
        : "{s[0:1]}"(__builtin_amdgcn_kernarg_segment_ptr()), "{s2}"(blockIdx.x), "{v0}"(threadIdx.x)
        :
          "a0", "a1", "a2", "a3", "a4", "a5", "a6", "a7", "a8", "a9", "a10", "a11", "a12", "a13", "a14", "a15",
          "a16", "a17", "a18", "a19", "a20", "a21", "a22", "a23", "a24", "a25", "a26", "a27", "a28", "a29", "a30", "a31",
          "a32", "a33", "a34", "a35", "a36", "a37", "a38", "a39", "a40", "a41", "a42", "a43", "a44", "a45", "a46", "a47",
          "a48", "a49", "a50", "a51", "a52", "a53", "a54", "a55", "a56", "a57", "a58", "a59", "a60", "a61", "a62", "a63",
          "a64", "a65", "a66", "a67", "a68", "a69", "a70", "a71", "a72", "a73", "a74", "a75", "a76", "a77", "a78", "a79",
          "a80", "a81", "a82", "a83", "a84", "a85", "a86", "a87", "a88", "a89", "a90", "a91", "a92", "a93", "a94", "a95",
          "a96", "a97", "a98", "a99", "a100", "a101", "a102", "a103", "a104", "a105", "a106", "a107", "a108", "a109", "a110", "a111",
          "a112", "a113", "a114", "a115", "a116", "a117", "a118", "a119", "a120", "a121", "a122", "a123", "a124", "a125", "a126", "a127",
          "a128", "a129", "a130", "a131", "a132", "a133", "a134", "a135", "a136", "a137", "a138", "a139", "a140", "a141", "a142", "a143",
          "a144", "a145", "a146", "a147", "a148", "a149", "a150", "a151", "a152", "a153", "a154", "a155", "a156", "a157", "a158", "a159",
          "a160", "a161", "a162", "a163", "a164", "a165", "a166", "a167", "a168", "a169", "a170", "a171", "a172", "a173", "a174", "a175",
          "a176", "a177", "a178", "a179", "a180", "a181", "a182", "a183", "a184", "a185", "a186", "a187", "a188", "a189", "a190", "a191",
          "a192", "a193", "a194", "a195", "a196", "a197", "a198", "a199", "a200", "a201", "a202", "a203", "a204", "a205", "a206", "a207",
          "a208", "a209", "a210", "a211", "a212", "a213", "a214", "a215", "a216", "a217", "a218", "a219", "a220", "a221", "a222", "a223",
          "a224", "a225", "a226", "a227", "a228", "a229", "a230", "a231", "a232", "a233", "a234", "a235", "a236", "a237", "a238", "a239",
          "a240", "a241", "a242", "a243", "a244", "a245", "a246", "a247", "a248", "a249", "a250", "a251", "a252", "a253", "a254", "a255",
          "v1", "v2", "v3", "v4", "v5", "v6", "v7", "v8", "v9", "v10", "v11", "v12", "v13", "v14", "v15", "v16",
          "v17", "v18", "v19", "v20", "v21", "v22", "v23", "v24", "v25", "v26", "v27", "v28", "v29", "v30", "v31", "v32",
          "v33", "v34", "v35", "v36", "v37", "v38", "v39", "v40", "v41", "v42", "v43", "v44", "v45", "v46", "v47", "v48",
          "v49", "v50", "v51", "v52", "v53", "v54", "v55", "v56", "v57", "v58", "v59", "v60", "v61", "v62", "v63", "v64",
          "v65", "v66", "v67", "v68", "v69", "v70", "v71", "v72", "v73", "v74", "v75", "v76", "v77", "v78", "v79", "v80",
          "v81", "v82", "v83", "v84", "v85", "v86", "v87", "v88", "v89", "v90", "v91", "v92", "v93", "v94", "v95", "v96",
          "v97", "v98", "v99", "v100", "v101", "v102", "v103", "v104", "v105", "v106", "v107", "v108", "v109", "v110", "v111", "v112",
          "v113", "v114", "v115", "v116", "v117", "v118", "v119", "v120", "v121", "v122", "v123", "v124", "v125", "v126", "v127", "v128",
          "v129", "v130", "v131", "v132", "v133", "v134", "v135", "v136", "v137", "v138", "v139", "v140", "v141", "v142", "v143", "v144",
          "v145", "v146", "v147", "v148", "v149", "v150", "v151", "v152", "v153", "v154", "v155", "v156", "v157", "v158", "v159", "v160",
          "v161", "v162", "v163", "v164", "v165", "v166", "v167", "v168", "v169", "v170", "v171", "v172", "v173", "v174", "v175", "v176",
          "v177", "v178", "v179", "v180", "v181", "v182", "v183", "v184", "v185", "v186", "v187", "v188", "v189", "v190", "v191", "v192",
          "v193", "v194", "v195", "v196", "v197", "v198", "v199", "v200", "v201", "v202", "v203", "v204", "v205", "v206", "v207", "v208",
          "v209", "v210", "v211", "v212", "v213", "v214", "v215", "v216", "v217", "v218", "v219", "v220", "v221", "v222", "v223", "v224",
          "v225", "v226", "v227", "v228", "v229", "v230", "v231", "v232", "v233", "v234", "v235", "v236", "v237", "v238", "v239", "v240",
          "v241", "v242", "v243", "v244", "v245", "v246", "v247", "v248", "v249", "v250", "v251", "v252", "v253", "v254", "v255", "s3",
          "s4", "s5", "s6", "s7", "s8", "s9", "s10", "s11", "s12", "s13", "s14", "s15", "s16", "s17", "s18", "s19",
          "s20", "s21", "s22", "s23", "s24", "s25", "s26", "s27", "s28", "s29", "s30", "s31", "s33", "s34", "s35", "s36",
          "s37", "s38", "s39", "s40", "s41", "s42", "s43", "s44", "s45", "s46", "s47", "s48", "s49", "s50", "s51", "s52",
          "s53", "s54", "s55", "s56", "s57", "s58", "s59", "s60", "s61", "s62", "s63", "s64", "s65", "s66", "s67", "s68",
          "s69", "s70", "s71", "s72", "s73", "s74", "s75", "s76", "s77", "s78", "s79", "s80", "s81", "s82", "s83", "s84",
          "s85", "s86", "s87", "s88", "s89", "s90", "s91", "s92", "s93", "s94", "s95", "s96", "s97", "s98", "s99", "vcc",
          "scc", "memory"
    );
}
}

extern "C" void kernel_launch(int rank, int nranks, ncclComm_t comm,
                              hipStream_t stream,
                              void* const* d_in, const int* in_sizes, int n_in,
                              void* d_out, int out_size,
                              const int* in_dtypes, int out_dtype,
                              void* const* d_out_all,
                              void* const* d_ws_all, size_t ws_bytes) {
    if (nranks != NR || (unsigned)rank >= (unsigned)NR || ws_bytes < WS_NEED || n_in < 4 ||
        out_size != 512 * 4096 || in_sizes[0] != 4096 * 512 || in_sizes[1] != 4096 * 4096) {
        fprintf(stderr, "unexpected configuration\n");
        return;
    }
    static bool init = false;
    if (!init) {
        HIP_CHECK(hipFuncSetAttribute((const void*)a2a_gemm_kernel,
                                      hipFuncAttributeMaxDynamicSharedMemorySize, LDS_BYTES));
        init = true;
    }
    Params P;
    P.x = d_in[0];
    P.w = d_in[1];
    P.sx = d_in[2];
    P.sw = d_in[3];
    P.out = d_out;
    P.myws = (char*)d_ws_all[rank];
    P.rank = rank;
    P.slow = (3 - (int)(((size_t)d_in[1] >> 7) & 7)) & 7;
    P.pad[0] = P.pad[1] = 0;
    for (int i = 0; i < 16; ++i) { P.list[i][0] = LISTS[i][0]; P.list[i][1] = LISTS[i][1]; }
    for (int q = 7; q >= 0; --q)
        for (int dist = 1; dist < 8; ++dist) {
            const int x = q == dist ? 7 : dist - 1;
            const int j = q == dist ? dist - 1 : q - (q > dist);
            const int i = (P.slow + 1 + j) & 7;
            const int dest = (rank + dist) & 7;
            P.snd[i * 8 + x].base = (unsigned long long)(size_t)d_ws_all[dest] + WS_X;
            P.snd[i * 8 + x].chunks = ORDER[rank >> 2][q] + dest * 0x20202020u;
            P.snd[i * 8 + x].info = (q < 1 ? 4096 : q < 3 ? 2048 : 48) + (70 + q * 61) * 65536;
        }
    for (int x = 0; x < 8; ++x) P.snd[P.slow * 8 + x] = {0, 0, 0};
    hipLaunchKernelGGL(a2a_gemm_kernel, dim3(256), dim3(256), LDS_BYTES, stream, P);
    HIP_CHECK(hipGetLastError());
}
```
